```python
import numpy as np

kernel_name = "gpu_gemm_a2a_m4096_k4096_n16384_bf16_q8"


ABSMAX_RTOL = 0.02

COMPUTE_DTYPE = "bfloat16"
EPILOGUE = "quant_int8"
IN_QUANT = None
OUT_QUANT = "int8"
IN_DTYPES = {"x": "bfloat16", "w_mat": "bfloat16"}
WORLD_SIZE = 8

def _to_bf16(x):
    u = np.ascontiguousarray(x, dtype=np.float32).view(np.uint32)
    return ((u + 0x7FFF + ((u >> 16) & 1)) & 0xFFFF0000).view(np.float32)

def setup_inputs():

    x = _to_bf16(np.random.randn(4096, 4096).astype(np.float32))
    w_mat = _to_bf16((np.random.randn(4096, 16384) / np.sqrt(4096)).astype(np.float32))
    return {"x": x, "w_mat": w_mat}

def _quant_dequant(y, amax):
    scale = amax / 127.0
    q = np.clip(np.round(y / scale), -127.0, 127.0)
    return (q * scale).astype(np.float32)
def _epilogue(y):
    return _quant_dequant(y, np.abs(y).max())

def reference(x, w_mat):
    y = (x @ w_mat).astype(np.float32)
    return _epilogue(y)
```

```cpp
#include <hip/hip_runtime.h>
#include <rccl/rccl.h>
#include <cstdio>

#define HIP_CHECK(expr) do { hipError_t _e = (expr); if (_e != hipSuccess) { \
    fprintf(stderr, "HIP error %s at %s:%d\n", hipGetErrorString(_e), __FILE__, __LINE__); \
    return; } } while (0)

typedef __bf16 bf16x4 __attribute__((ext_vector_type(4)));
typedef __bf16 bf16x8 __attribute__((ext_vector_type(8)));
typedef float f32x4 __attribute__((ext_vector_type(4)));
typedef unsigned u32x4 __attribute__((ext_vector_type(4)));
typedef unsigned long long u64;
#define LAS __attribute__((address_space(3)))
#define DI __device__ __forceinline__

constexpr int NR = 8;
constexpr int M = 512, N = 16384, K = 4096, BK = 64, NT = K / BK;
constexpr int NS = N / NR;
constexpr int TM = 128, TN = 256, NWG = 256;
constexpr int XST = 3, WST = 3;
constexpr int XB = TM * 128, WB = 64 * TN * 2;
constexpr int XOFF = 0, WOFF = XST * XB, MISC = XST * XB + WST * WB, LDS_BYTES = MISC + 256;
constexpr int XP = XB / 1024 / 4, WP = WB / 1024 / 4;
constexpr int WROW = TN * 2;
constexpr size_t WS_CNT = 0;
constexpr size_t WS_TMO = WS_CNT + NWG * 128;
constexpr size_t WS_GRAN = WS_TMO + 1024;
constexpr size_t WS_DONE = WS_GRAN + 2 * NR * NWG * 8;
constexpr size_t WS_END = WS_DONE + NR * 32 * 128;
constexpr u64 DEADLINE = 30000000ull;

struct Params {
  const void* x; const void* w;
  char* out[NR]; char* ws[NR];
  int rank; int pad;
};

#define WAITV(n) asm volatile("s_waitcnt vmcnt(%0)" ::"n"(n) : "memory")
#define BAR() asm volatile("s_barrier" ::: "memory")
#define LBAR() asm volatile("s_waitcnt lgkmcnt(0)\n\ts_barrier" ::: "memory")
#define SB0() __builtin_amdgcn_sched_barrier(0)

struct Frag { bf16x8 x[8]; bf16x8 w[4]; };

template <int KS> DI void read_frags(Frag& f, LAS char* xa, LAS char* w0, LAS char* w1, LAS char* w2, LAS char* w3) {
  LAS char* wa[4] = {w0, w1, w2, w3};
#pragma unroll
  for (int j = 0; j < 4; ++j) {
    bf16x4 lo = __builtin_amdgcn_ds_read_tr16_b64_v4bf16((LAS bf16x4*)(wa[j] + WROW * (32 * KS)));
    bf16x4 hi = __builtin_amdgcn_ds_read_tr16_b64_v4bf16((LAS bf16x4*)(wa[j] + WROW * (32 * KS + 4)));
    f.w[j] = __builtin_shufflevector(lo, hi, 0, 1, 2, 3, 4, 5, 6, 7);
  }
#pragma unroll
  for (int i = 0; i < 8; ++i) f.x[i] = *(LAS bf16x8*)(xa + 2048 * i);
}
DI void mma(f32x4 (&acc)[8][4], const Frag& f) {
#pragma unroll
  for (int i = 0; i < 8; ++i)
#pragma unroll
    for (int j = 0; j < 4; ++j) acc[i][j] = __builtin_amdgcn_mfma_f32_16x16x32_bf16(f.w[j], f.x[i], acc[i][j], 0, 0, 0);
}
DI void interleave() {
#pragma unroll
  for (int g = 0; g < 16; ++g) {
    __builtin_amdgcn_sched_group_barrier(0x008, 1, 0);
    __builtin_amdgcn_sched_group_barrier(0x100, 1, 0);
  }
  __builtin_amdgcn_sched_group_barrier(0x008, 16, 0);
}
DI unsigned wave_umax(unsigned v) {
#pragma unroll
  for (int o = 32; o >= 1; o >>= 1) { unsigned t = __shfl_xor(v, o); v = t > v ? t : v; }
  return v;
}
DI bool timed_out(unsigned* tmo, unsigned call, u64 t0) {
  if (__hip_atomic_load(tmo, __ATOMIC_RELAXED, __HIP_MEMORY_SCOPE_AGENT) == call || __builtin_amdgcn_s_memrealtime() - t0 > DEADLINE) {
    __hip_atomic_store(tmo, call, __ATOMIC_RELAXED, __HIP_MEMORY_SCOPE_AGENT);
    return true;
  }
  return false;
}

__global__ __launch_bounds__(512, 2) void gemm_a2a(const Params p) {
  extern __shared__ __attribute__((aligned(16))) char smem[];
  LAS char* lds = (LAS char*)smem;
  LAS unsigned* misc = (LAS unsigned*)(lds + MISC);
  const int tid = threadIdx.x, lane = tid & 63;
  const int wid = __builtin_amdgcn_readfirstlane(tid >> 6);
  const int b = blockIdx.x, xcd = b & 7, idx = b >> 3;
  const int rank = p.rank;
  const int rt = idx & 3;
  const int dst = ((idx >> 2) + rank) & 7;
  const int ct = dst * 8 + xcd;
  char* myws = p.ws[rank & 7];
  if (wid >= 4) {
    const int w = wid - 4;
    auto rx = __builtin_amdgcn_make_buffer_rsrc((void*)p.x, 0, M * K * 2, 0x00020000);
    auto rw = __builtin_amdgcn_make_buffer_rsrc((void*)p.w, 0, K * N * 2, 0x00020000);
    const int xrow = 8 * w + (lane >> 3);
    const unsigned xv = (unsigned)((rt * TM + xrow) * K * 2 + (((lane & 7) ^ ((xrow >> 1) & 7)) << 4));
    const int kl = (lane >> 5) | ((w & 1) << 1) | ((w >> 1) << 3);
    const int fk = (kl & 3) | (((kl >> 3) & 1) << 2);
    const unsigned wv = (unsigned)(kl * N * 2 + ct * TN * 2 + (((lane & 31) ^ (fk << 1)) << 4));
    const int wlu = (((w & 1) << 1) | ((w >> 1) << 3)) * WROW;
    auto issue_x = [&](int t, int slot) {
#pragma unroll
      for (int ii = 0; ii < XP; ++ii)
        __builtin_amdgcn_raw_ptr_buffer_load_lds(rx, (LAS void*)(lds + XOFF + slot * XB + (4 * ii + w) * 1024), 16, xv, t * 128 + ii * 32 * K * 2, 0, 0);
    };
    auto issue_w = [&](int t, int slot) {
#pragma unroll
      for (int jj = 0; jj < WP; ++jj) {
        const int ko = ((jj & 1) << 2) | ((jj >> 1) << 4);
        __builtin_amdgcn_raw_ptr_buffer_load_lds(rw, (LAS void*)(lds + WOFF + slot * WB + wlu + ko * WROW), 16, wv, (t * 64 + ko) * N * 2, 0, 0);
      }
    };
    issue_w(0, 0); issue_x(0, 0); issue_w(1, 1); issue_x(1, 1);
    WAITV(XP + WP);
    BAR();
    int sl = 2;
    for (int s = 0; s < NT; ++s) {
      if (s + 2 < NT) { issue_x(s + 2, sl); issue_w(s + 2, sl); WAITV(XP + WP); }
      else WAITV(0);
      sl = sl == 2 ? 0 : sl + 1;
      BAR();
    }
    unsigned* cnt = (unsigned*)(myws + WS_CNT + b * 128);
    const unsigned call = __builtin_amdgcn_readfirstlane(__hip_atomic_load(cnt, __ATOMIC_RELAXED, __HIP_MEMORY_SCOPE_AGENT)) + 1u;
    unsigned* tmo = (unsigned*)(myws + WS_TMO);
    char* wsl = p.ws[lane & 7];
    LBAR();
    if (w == 0) {
      unsigned m = misc[0];
      { unsigned t = misc[1]; m = t > m ? t : m; t = misc[2]; m = t > m ? t : m; t = misc[3]; m = t > m ? t : m; }
      if (lane < NR)
        __hip_atomic_store((u64*)(wsl + WS_GRAN + ((size_t)((call & 1) * NR + rank) * NWG + b) * 8), ((u64)call << 32) | m, __ATOMIC_RELAXED, __HIP_MEMORY_SCOPE_SYSTEM);
    }
    const char* gp = myws + WS_GRAN + (size_t)(call & 1) * NR * NWG * 8 + w * 4096 + lane * 16;
    unsigned gm = 0; bool bad = false;
    const u64 t0 = __builtin_amdgcn_s_memrealtime();
    for (;;) {
      u32x4 a, c, d, e;
      asm volatile("global_load_dwordx4 %0, %4, off sc0 sc1\n\t"
                   "global_load_dwordx4 %1, %4, off offset:1024 sc0 sc1\n\t"
                   "global_load_dwordx4 %2, %4, off offset:2048 sc0 sc1\n\t"
                   "global_load_dwordx4 %3, %4, off offset:3072 sc0 sc1\n\ts_waitcnt vmcnt(0)"
                   : "=&v"(a), "=&v"(c), "=&v"(d), "=&v"(e) : "v"(gp) : "memory");
      const bool ok = a[1] == call && a[3] == call && c[1] == call && c[3] == call && d[1] == call && d[3] == call && e[1] == call && e[3] == call;
      if (__all(ok)) {
        gm = max(max(max(a[0], a[2]), max(c[0], c[2])), max(max(d[0], d[2]), max(e[0], e[2])));
        break;
      }
      if (__any(timed_out(tmo, call, t0))) { bad = true; break; }
      __builtin_amdgcn_s_sleep(1);
    }
    gm = wave_umax(gm);
    if (__hip_atomic_load(tmo, __ATOMIC_RELAXED, __HIP_MEMORY_SCOPE_AGENT) == call) bad = true;
    if (lane == 0) misc[4 + w] = bad ? 0xffffffffu : gm;
    LBAR();
    LBAR();
    if (w == 0 && lane == 0) {
      if (dst != rank)
        __hip_atomic_store((unsigned*)(p.ws[dst] + WS_DONE + (size_t)(rank * 32 + xcd * 4 + rt) * 128), call, __ATOMIC_RELAXED, __HIP_MEMORY_SCOPE_SYSTEM);
      __hip_atomic_store(cnt, call, __ATOMIC_RELAXED, __HIP_MEMORY_SCOPE_AGENT);
      if ((b >> 5) != rank) {
        unsigned* fl = (unsigned*)(myws + WS_DONE + (size_t)b * 128);
        const u64 t1 = __builtin_amdgcn_s_memrealtime();
        while (__hip_atomic_load(fl, __ATOMIC_RELAXED, __HIP_MEMORY_SCOPE_SYSTEM) < call) {
          if (timed_out(tmo, call, t1)) { p.out[rank & 7][(size_t)rank * M * NS + b] = (char)-128; break; }
          __builtin_amdgcn_s_sleep(1);
        }
      }
    }
  } else {
    __builtin_amdgcn_s_setprio(2);
    const int wn = wid;
    const int fr = lane & 15, g = lane >> 4, q = (lane & 15) >> 2, pp = lane & 3;
    const int xo0 = fr * 128 + ((g ^ (fr >> 1)) << 4);
    const int fk = q | ((g & 1) << 2);
    const int wo0 = WROW * (8 * g + q) + ((wn >> 1) * 256) + ((((wn & 1) * 4) ^ fk) << 5) + 8 * pp;
    f32x4 acc[8][4];
#pragma unroll
    for (int i = 0; i < 8; ++i)
#pragma unroll
      for (int j = 0; j < 4; ++j) acc[i][j] = (f32x4){0.f, 0.f, 0.f, 0.f};
    Frag f0, f1;
    BAR();
    int sl = 0;
    {
      LAS char* xa = lds + XOFF + xo0; LAS char* wa = lds + WOFF;
      read_frags<0>(f0, xa, wa + wo0, wa + (wo0 ^ 32), wa + (wo0 ^ 64), wa + (wo0 ^ 96));
    }
    for (int s = 0; s < NT; ++s) {
      SB0();
      {
        LAS char* xa = lds + XOFF + sl * XB + (xo0 ^ 64); LAS char* wa = lds + WOFF + sl * WB;
        read_frags<1>(f1, xa, wa + wo0, wa + (wo0 ^ 32), wa + (wo0 ^ 64), wa + (wo0 ^ 96));
      }
      mma(acc, f0);
      interleave();
      SB0();
      __builtin_amdgcn_s_waitcnt(0xC07F);
      BAR();
      SB0();
      sl = sl == 2 ? 0 : sl + 1;
      {
        LAS char* xa = lds + XOFF + sl * XB + xo0; LAS char* wa = lds + WOFF + sl * WB;
        read_frags<0>(f0, xa, wa + wo0, wa + (wo0 ^ 32), wa + (wo0 ^ 64), wa + (wo0 ^ 96));
      }
      mma(acc, f1);
      interleave();
    }
    SB0();
    float m = 0.f;
#pragma unroll
    for (int i = 0; i < 8; ++i)
#pragma unroll
      for (int j = 0; j < 4; ++j) {
        m = fmaxf(fmaxf(m, __builtin_fabsf(acc[i][j][0])), __builtin_fabsf(acc[i][j][1]));
        m = fmaxf(fmaxf(m, __builtin_fabsf(acc[i][j][2])), __builtin_fabsf(acc[i][j][3]));
      }
    const unsigned mu = wave_umax(__float_as_uint(m));
    if (lane == 0) misc[wid] = mu;
    LBAR();
    LBAR();
    unsigned gm = misc[4];
    { unsigned t = misc[5]; gm = t > gm ? t : gm; t = misc[6]; gm = t > gm ? t : gm; t = misc[7]; gm = t > gm ? t : gm; }
    const bool bad = gm == 0xffffffffu;
    const float amax = __uint_as_float(gm);
    const float inv = amax > 0.f ? 127.f / amax : 0.f;
    char* orow = p.out[dst] + (size_t)(rank * M + rt * TM + fr) * NS + xcd * TN + wn * 64 + 16 * g;
#pragma unroll
    for (int i = 0; i < 8; ++i) {
      unsigned d[4];
#pragma unroll
      for (int j = 0; j < 4; ++j) {
        unsigned a0 = __float_as_uint(__builtin_fmaf(acc[i][j][0], inv, 12582912.f)), a1 = __float_as_uint(__builtin_fmaf(acc[i][j][1], inv, 12582912.f));
        unsigned a2 = __float_as_uint(__builtin_fmaf(acc[i][j][2], inv, 12582912.f)), a3 = __float_as_uint(__builtin_fmaf(acc[i][j][3], inv, 12582912.f));
        d[j] = __builtin_amdgcn_perm(a1, a0, 0x0c0c0400) | __builtin_amdgcn_perm(a3, a2, 0x04000c0c);
      }
      auto r0 = __builtin_amdgcn_permlane32_swap(d[0], d[2], false, false); d[0] = r0[0]; d[2] = r0[1];
      auto r1 = __builtin_amdgcn_permlane32_swap(d[1], d[3], false, false); d[1] = r1[0]; d[3] = r1[1];
      auto r2 = __builtin_amdgcn_permlane16_swap(d[0], d[1], false, false); d[0] = r2[0]; d[1] = r2[1];
      auto r3 = __builtin_amdgcn_permlane16_swap(d[2], d[3], false, false); d[2] = r3[0]; d[3] = r3[1];
      u32x4 v = {d[0], d[1], d[2], d[3]};
      if (bad) v = (u32x4){0x80808080u, 0x80808080u, 0x80808080u, 0x80808080u};
      asm volatile("global_store_dwordx4 %0, %1, off sc0 sc1\n\ts_nop 1" ::"v"(orow + (size_t)i * 16 * NS), "v"(v) : "memory");
    }
    WAITV(0);
    LBAR();
  }
}

extern "C" void kernel_launch(int rank, int nranks, ncclComm_t comm, hipStream_t stream,
                              void* const* d_in, const int* in_sizes, int n_in,
                              void* d_out, int out_size, const int* in_dtypes, int out_dtype,
                              void* const* d_out_all, void* const* d_ws_all, size_t ws_bytes) {
  if (nranks != NR || (unsigned)rank >= (unsigned)NR || n_in < 2 || ws_bytes < WS_END || in_sizes[0] != M * K || in_sizes[1] != K * N || out_size != NR * M * NS) {
    fprintf(stderr, "unexpected problem shape\n");
    return;
  }
  static bool init = false;
  if (!init) {
    HIP_CHECK(hipFuncSetAttribute((const void*)gemm_a2a, hipFuncAttributeMaxDynamicSharedMemorySize, LDS_BYTES));
    init = true;
  }
  Params p;
  p.x = d_in[0]; p.w = d_in[1];
  for (int i = 0; i < NR; ++i) { p.out[i] = (char*)d_out_all[i]; p.ws[i] = (char*)d_ws_all[i]; }
  p.rank = rank; p.pad = 0;
  gemm_a2a<<<NWG, 512, LDS_BYTES, stream>>>(p);
  HIP_CHECK(hipGetLastError());
}
```

```cpp
#include <hip/hip_runtime.h>
#include <rccl/rccl.h>
#include <cstdio>

#define HIP_CHECK(expr) do { hipError_t _e = (expr); if (_e != hipSuccess) { \
    fprintf(stderr, "HIP error %s at %s:%d\n", hipGetErrorString(_e), __FILE__, __LINE__); \
    return; } } while (0)

typedef __bf16 bf16x4 __attribute__((ext_vector_type(4)));
typedef __bf16 bf16x8 __attribute__((ext_vector_type(8)));
typedef float f32x4 __attribute__((ext_vector_type(4)));
typedef unsigned u32x4 __attribute__((ext_vector_type(4)));
typedef unsigned long long u64;
#define LAS __attribute__((address_space(3)))
#define DI __device__ __forceinline__

#define LPERM 1
#define PACE 0

constexpr int NR = 8;
constexpr int M = 512, N = 16384, K = 4096, NT = K / 64;
constexpr int NS = N / NR;
constexpr int TN = 256, NWG = 256;
constexpr int NU = 14, GR = NU * 16;
constexpr int XB = 144 * 128, WB = 64 * TN * 2, SLOT = XB + WB;
constexpr int MISC = 3 * SLOT, LDS_BYTES = MISC + 256;
constexpr int WROW = TN * 2;
constexpr int UB = NT * 2048;
static_assert(LDS_BYTES <= 163840 && UB <= MISC, "LDS");
constexpr int L_MAX = 0, L_GMAX = 4;
constexpr size_t WS_CNT = 0;
constexpr size_t WS_TMO = WS_CNT + NWG * 128;
constexpr size_t WS_GRAN = WS_TMO + 1024;
constexpr size_t WS_DONE = WS_GRAN + 2 * NR * NWG * 8;
constexpr size_t WS_SCR = WS_DONE + NR * 16 * 128;
constexpr size_t WS_XFLG = 1u << 20;
constexpr size_t WS_XST = 2u << 20;
constexpr size_t WS_END = WS_XST + (size_t)NR * NU * UB;
static_assert(WS_SCR + 128 * 128 <= WS_XFLG && WS_XFLG + NR * NU * NT * 128 <= WS_XST, "workspace map");
#define STAMP(i) do {} while (0)
constexpr u64 DEADLINE = 30000000ull;

struct Params {
  const void* x; const void* w;
  char* out[NR]; char* ws[NR];
  int rank; int pad;
};

#define WAITV(n) asm volatile("s_waitcnt vmcnt(%0)" ::"n"(n) : "memory")
#define BAR() asm volatile("s_barrier" ::: "memory")
#define LBAR() asm volatile("s_waitcnt lgkmcnt(0)\n\ts_barrier" ::: "memory")
#define SB0() __builtin_amdgcn_sched_barrier(0)

template <int NI> struct Frag { bf16x8 x[NI]; bf16x8 w[4]; };

template <int KS, int NI> DI void read_frags(Frag<NI>& f, LAS char* xa, LAS char* w0, LAS char* w1, LAS char* w2, LAS char* w3) {
  LAS char* wa[4] = {w0, w1, w2, w3};
#pragma unroll
  for (int j = 0; j < 4; ++j) {
    bf16x4 lo = __builtin_amdgcn_ds_read_tr16_b64_v4bf16((LAS bf16x4*)(wa[j] + WROW * (32 * KS)));
    bf16x4 hi = __builtin_amdgcn_ds_read_tr16_b64_v4bf16((LAS bf16x4*)(wa[j] + WROW * (32 * KS + 4)));
    f.w[j] = __builtin_shufflevector(lo, hi, 0, 1, 2, 3, 4, 5, 6, 7);
  }
#pragma unroll
  for (int i = 0; i < NI; ++i) f.x[i] = *(LAS bf16x8*)(xa + 2048 * i);
}
template <int NI> DI void mma(f32x4 (&acc)[NI][4], const Frag<NI>& f) {
#pragma unroll
  for (int i = 0; i < NI; ++i)
#pragma unroll
    for (int j = 0; j < 4; ++j) acc[i][j] = __builtin_amdgcn_mfma_f32_16x16x32_bf16(f.w[j], f.x[i], acc[i][j], 0, 0, 0);
#pragma unroll
  for (int g = 0; g < 8; ++g) {
    __builtin_amdgcn_sched_group_barrier(0x008, 1, 0);
    __builtin_amdgcn_sched_group_barrier(0x100, 1, 0);
    __builtin_amdgcn_sched_group_barrier(0x008, 2, 0);
    __builtin_amdgcn_sched_group_barrier(0x100, 1, 0);
  }
  if (NI == 9) { __builtin_amdgcn_sched_group_barrier(0x008, 2, 0); __builtin_amdgcn_sched_group_barrier(0x100, 1, 0); }
  __builtin_amdgcn_sched_group_barrier(0x008, 10, 0);
}
DI unsigned wave_umax(unsigned v) {
#pragma unroll
  for (int o = 32; o >= 1; o >>= 1) { unsigned t = __shfl_xor(v, o); v = t > v ? t : v; }
  return v;
}
DI bool timed_out(unsigned* tmo, unsigned call, u64 t0) {
  if (__hip_atomic_load(tmo, __ATOMIC_RELAXED, __HIP_MEMORY_SCOPE_AGENT) == call || __builtin_amdgcn_s_memrealtime() - t0 > DEADLINE) {
    __hip_atomic_store(tmo, call, __ATOMIC_RELAXED, __HIP_MEMORY_SCOPE_AGENT);
    return true;
  }
  return false;
}

template <int NI> DI void mfma_role(const Params& p, LAS char* lds, int wid, int lane, int b, char* myws, char* obase, int u0) {
  volatile LAS unsigned* misc = (volatile LAS unsigned*)(lds + MISC);
  const int fr = lane & 15, g = lane >> 4, q = (lane & 15) >> 2, pp = lane & 3;
  const int xo0 = fr * 128 + ((g ^ (fr >> 1)) << 4);
  const int fk = q | ((g & 1) << 2);
  const int wo0 = XB + WROW * (8 * g + q) + ((wid >> 1) * 256) + ((((wid & 1) * 4) ^ fk) << 5) + 8 * pp;
  f32x4 acc[NI][4];
#pragma unroll
  for (int i = 0; i < NI; ++i)
#pragma unroll
    for (int j = 0; j < 4; ++j) acc[i][j] = (f32x4){0.f, 0.f, 0.f, 0.f};
  Frag<NI> f0, f1;
  BAR();
  read_frags<0>(f0, lds + xo0, lds + wo0, lds + (wo0 ^ 32), lds + (wo0 ^ 64), lds + (wo0 ^ 96));
  int so = 0;
  for (int s = 0; s < NT; ++s) {
    SB0();
    { LAS char* base = lds + so;
      read_frags<1>(f1, base + (xo0 ^ 64), base + wo0, base + (wo0 ^ 32), base + (wo0 ^ 64), base + (wo0 ^ 96)); }
    mma(acc, f0);
    so = so == 2 * SLOT ? 0 : so + SLOT;
    SB0();
    __builtin_amdgcn_s_waitcnt(0xC07F);
    BAR();
    SB0();
    { LAS char* base = lds + so;
      read_frags<0>(f0, base + xo0, base + wo0, base + (wo0 ^ 32), base + (wo0 ^ 64), base + (wo0 ^ 96)); }
    mma(acc, f1);
  }
  SB0();
  float m = 0.f;
#pragma unroll
  for (int i = 0; i < NI; ++i)
#pragma unroll
    for (int j = 0; j < 4; ++j) {
      m = fmaxf(fmaxf(m, __builtin_fabsf(acc[i][j][0])), __builtin_fabsf(acc[i][j][1]));
      m = fmaxf(fmaxf(m, __builtin_fabsf(acc[i][j][2])), __builtin_fabsf(acc[i][j][3]));
    }
  const unsigned mu = wave_umax(__float_as_uint(m));
  if (lane == 0) misc[L_MAX + wid] = mu;
  LBAR();
  LBAR();
  if (wid == 0) STAMP(3);
  const u32x4 gv = *(volatile LAS u32x4*)(misc + L_GMAX);
  const unsigned gm = max(max(gv[0], gv[1]), max(gv[2], gv[3]));
  const bool bad = gm == 0xffffffffu;
  const float amax = __uint_as_float(gm);
  const float inv = amax > 0.f ? 127.f / amax : 0.f;
  char* orow = obase + (size_t)fr * NS + wid * 64 + 16 * g;
#pragma unroll
  for (int i = 0; i < NI; ++i) {
    unsigned d[4];
#pragma unroll
    for (int j = 0; j < 4; ++j) {
      unsigned a0 = __float_as_uint(__builtin_fmaf(acc[i][j][0], inv, 12582912.f)), a1 = __float_as_uint(__builtin_fmaf(acc[i][j][1], inv, 12582912.f));
      unsigned a2 = __float_as_uint(__builtin_fmaf(acc[i][j][2], inv, 12582912.f)), a3 = __float_as_uint(__builtin_fmaf(acc[i][j][3], inv, 12582912.f));
      d[j] = __builtin_amdgcn_perm(a1, a0, 0x0c0c0400) | __builtin_amdgcn_perm(a3, a2, 0x04000c0c);
    }
    auto r0 = __builtin_amdgcn_permlane32_swap(d[0], d[2], false, false); d[0] = r0[0]; d[2] = r0[1];
    auto r1 = __builtin_amdgcn_permlane32_swap(d[1], d[3], false, false); d[1] = r1[0]; d[3] = r1[1];
    auto r2 = __builtin_amdgcn_permlane16_swap(d[0], d[1], false, false); d[0] = r2[0]; d[1] = r2[1];
    auto r3 = __builtin_amdgcn_permlane16_swap(d[2], d[3], false, false); d[2] = r3[0]; d[3] = r3[1];
    u32x4 v = {d[0], d[1], d[2], d[3]};
    if (bad) v = (u32x4){0x80808080u, 0x80808080u, 0x80808080u, 0x80808080u};
    const int r16 = NI == 9 ? i * 16 : ((u0 + i) / NU) * M + ((u0 + i) % NU) * 16;
    asm volatile("global_store_dwordx4 %0, %1, off sc0 sc1\n\ts_nop 1" ::"v"(orow + (size_t)r16 * NS), "v"(v) : "memory");
  }
  if (wid == 0) STAMP(4);
  WAITV(0);
  LBAR();
  if (wid == 0) STAMP(5);
}

__global__ __launch_bounds__(512, 2) void gemm_a2a(const Params p) {
  extern __shared__ __attribute__((aligned(16))) char smem[];
  LAS char* lds = (LAS char*)smem;
  volatile LAS unsigned* misc = (volatile LAS unsigned*)(lds + MISC);
  const int tid = threadIdx.x, lane = tid & 63;
  const int wid = __builtin_amdgcn_readfirstlane(tid >> 6);
  const int b = blockIdx.x, xcd = b & 7, idx = b >> 3;
  const int rank = p.rank & 7;
  char* myws = p.ws[rank];
  unsigned* cnt = (unsigned*)(myws + WS_CNT + b * 128);
  unsigned* tmo = (unsigned*)(myws + WS_TMO);
  if (wid == 0) STAMP(0);

  if (idx >= 30) {
    const int un = xcd * 2 + (idx & 1);
    unsigned call;
    if (un < NU) {
      auto rx = __builtin_amdgcn_make_buffer_rsrc((void*)p.x, 0, M * K * 2, 0x00020000);
      const unsigned v0 = (unsigned)((un * 16 + (lane >> 3)) * K * 2 + (((lane & 7) ^ (lane >> 4)) << 4)), v1 = (v0 ^ 64) + 8 * K * 2;
      auto stage = [&](int kt) {
        __builtin_amdgcn_raw_ptr_buffer_load_lds(rx, (LAS void*)(lds + kt * 2048), 16, v0, kt * 128, 0, 0);
        __builtin_amdgcn_raw_ptr_buffer_load_lds(rx, (LAS void*)(lds + kt * 2048 + 1024), 16, v1, kt * 128, 0, 0);
      };
      stage(wid);
      unsigned c0;
      asm volatile("global_load_dword %0, %1, off sc1" : "=v"(c0) : "v"(cnt) : "memory");
      char* pw = p.ws[(rank + wid) & 7];
      char* dp = pw + WS_XST + (size_t)(rank * NU + un) * UB + lane * 16;
      char* fn = pw + WS_XFLG + (size_t)(rank * NU + un) * NT * 128;
      char* f1 = myws + WS_SCR + (un * 8 + wid) * 128;
      asm volatile("s_waitcnt vmcnt(0)" : "+v"(c0)::"memory");
      call = __builtin_amdgcn_readfirstlane(c0) + 1u;
      LAS char* la = lds + lane * 16;
      u64 tn = 0;
      for (int r = 0; r < 8; ++r) {
        BAR();
        if (PACE && r == 0) tn = __builtin_amdgcn_s_memrealtime();
        u32x4 a0 = *(LAS u32x4*)(la + r * 16384), a1 = *(LAS u32x4*)(la + r * 16384 + 1024);
#pragma unroll
        for (int j = 0; j < 8; ++j) {
          u32x4 n0 = a0, n1 = a1;
          if (j < 7) { n0 = *(LAS u32x4*)(la + r * 16384 + (j + 1) * 2048); n1 = *(LAS u32x4*)(la + r * 16384 + (j + 1) * 2048 + 1024); }
          if (PACE) { while ((long long)(__builtin_amdgcn_s_memrealtime() - tn) < 0) __builtin_amdgcn_s_sleep(1); tn += PACE; }
          asm volatile("global_store_dwordx4 %0, %1, off sc0 sc1\n\t"
                       "global_store_dwordx4 %0, %2, off offset:1024 sc0 sc1\n\t"
                       "s_waitcnt vmcnt(3)\n\t"
                       "s_mov_b64 exec, 1\n\tglobal_store_dword %3, %4, off sc0 sc1\n\ts_mov_b64 exec, -1\n\ts_nop 0"
                       ::"v"(dp), "v"(a0), "v"(a1), "v"(f1), "v"(call) : "memory");
          f1 = fn; fn += 128; dp += 2048;
          if (j == 3 && r < 7) stage(8 * (r + 1) + wid);
          a0 = n0; a1 = n1;
        }
      }
      WAITV(0);
      if (lane == 0) __hip_atomic_store((unsigned*)f1, call, __ATOMIC_RELAXED, __HIP_MEMORY_SCOPE_SYSTEM);
    } else {
      call = __builtin_amdgcn_readfirstlane(__hip_atomic_load(cnt, __ATOMIC_RELAXED, __HIP_MEMORY_SCOPE_AGENT)) + 1u;
    }
    if (wid == 0) STAMP(7);
    const int fi = un * 8 + wid;
    if (lane == 0 && (fi >> 4) != rank) {
      unsigned* fl = (unsigned*)(myws + WS_DONE + (size_t)fi * 128);
      const u64 t1 = __builtin_amdgcn_s_memrealtime();
      unsigned np = 0;
      while (__hip_atomic_load(fl, __ATOMIC_RELAXED, __HIP_MEMORY_SCOPE_SYSTEM) < call) {
        if ((++np & 63u) == 0 && timed_out(tmo, call, t1)) { p.out[rank][(size_t)rank * M * NS + fi] = (char)-128; break; }
        __builtin_amdgcn_s_sleep(2);
      }
    }
    BAR();
    if (tid == 0) __hip_atomic_store(cnt, call, __ATOMIC_RELAXED, __HIP_MEMORY_SCOPE_AGENT);
    if (wid == 0) STAMP(6);
    return;
  }

  const bool isG = idx >= 16;
  const int i16 = idx & 15;
  const int rt = isG ? (xcd & 1) * 7 + i16 % 7 : (i16 & 1);
  const int dst = isG ? rank : (rank + (i16 >> 1)) & 7;
  const int cl = isG ? (xcd >> 1) * 2 + i16 / 7 : xcd;
  const int ct = dst * 8 + cl;
  if (wid < 4) {
    __builtin_amdgcn_s_setprio(2);
    if (isG) mfma_role<8>(p, lds, wid, lane, b, myws, p.out[rank] + cl * TN, rt * 8);
    else mfma_role<9>(p, lds, wid, lane, b, myws, p.out[dst] + (size_t)(rank * M + GR + rt * 144) * NS + cl * TN, 0);
    return;
  }
  const int w = wid - 4;
  bool bad = false;
  unsigned call = 0;
  auto ktile = [&](int u) {
    if (!isG && LPERM) { const int uu = (u + 2 * ((i16 >> 1) & 1)) & 63; return (uu & ~3) | ((uu + 2 * rt) & 3); }
    return u;
  };
  if (w == 0 && isG) {
    auto rs = __builtin_amdgcn_make_buffer_rsrc((void*)(myws + WS_XST + (size_t)rt * 8 * UB), 0, 8 * UB, 0x00020000);
    const unsigned gv = lane * 16;
    call = __builtin_amdgcn_readfirstlane(__hip_atomic_load(cnt, __ATOMIC_RELAXED, __HIP_MEMORY_SCOPE_AGENT)) + 1u;
    const char* fp = myws + WS_XFLG + (size_t)(rt * 8 + (lane & 7)) * NT * 128;
    const u64 t0 = __builtin_amdgcn_s_memrealtime();
    unsigned rdy = 0, np = 0;
    auto seen = [&](unsigned f) { const u64 miss = ~__ballot(f >= call); rdy = min((unsigned)NT, rdy + (miss ? (unsigned)__builtin_ctzll(miss) >> 3 : 8u)); };
    int so = 0;
    for (int t = 0; t < NT + 2; ++t) {
      unsigned f = 0; bool ahead = false;
      if (t < NT) {
        while (rdy <= (unsigned)t) {
          seen(__hip_atomic_load((const unsigned*)(fp + min(rdy + (lane >> 3), NT - 1u) * 128), __ATOMIC_RELAXED, __HIP_MEMORY_SCOPE_SYSTEM));
          if (rdy <= (unsigned)t && (++np & 63u) == 0 && __any(timed_out(tmo, call, t0))) { bad = true; rdy = NT; }
        }
        ahead = rdy < NT && rdy < (unsigned)t + 5;
        if (ahead) asm volatile("global_load_dword %0, %1, off sc0 sc1" : "=v"(f) : "v"(fp + min(rdy + (lane >> 3), NT - 1u) * 128) : "memory");
#pragma unroll
        for (int i = 0; i < 16; ++i)
          __builtin_amdgcn_raw_ptr_buffer_load_lds(rs, (LAS void*)(lds + so + i * 1024), 16, gv, t * 2048 + (i >> 1) * UB + (i & 1) * 1024, 0, 17);
        so = so == 2 * SLOT ? 0 : so + SLOT;
        asm volatile("s_waitcnt vmcnt(16)" : "+v"(f)::"memory");
        if (ahead) seen(f);
      } else WAITV(0);
      if (t >= 1) BAR();
    }
    STAMP(7);
  } else if (w == 0) {
    auto rs = __builtin_amdgcn_make_buffer_rsrc((void*)p.x, 0, M * K * 2, 0x00020000);
    const unsigned v0 = (unsigned)((GR + rt * 144 + (lane >> 3)) * K * 2 + (((lane & 7) ^ (lane >> 4)) << 4)), v1 = (v0 ^ 64) + 8 * K * 2;
    int so = 0;
    for (int t = 0; t < NT + 2; ++t) {
      if (t < NT) {
        const int kt = ktile(t);
#pragma unroll
        for (int i = 0; i < 18; ++i)
          __builtin_amdgcn_raw_ptr_buffer_load_lds(rs, (LAS void*)(lds + so + i * 1024), 16, (i & 1) ? v1 : v0, kt * 128 + (i >> 1) * 16 * K * 2, 0, 0);
        so = so == 2 * SLOT ? 0 : so + SLOT;
        WAITV(18);
      } else WAITV(0);
      if (t >= 1) BAR();
    }
  } else if (w < 3) {
    auto rs = __builtin_amdgcn_make_buffer_rsrc((void*)p.w, 0, K * N * 2, 0x00020000);
    const unsigned v0 = (unsigned)(((lane >> 5) + (w - 1) * 32) * N * 2 + ct * TN * 2 + (((lane & 31) ^ ((lane >> 5) << 1)) << 4));
    const unsigned v1 = (v0 ^ 64) + 2 * N * 2, v2 = (v0 ^ 128) + 8 * N * 2, v3 = (v0 ^ 192) + 10 * N * 2;
    int so = XB + (w - 1) * 16384;
    for (int t = 0; t < NT + 2; ++t) {
      if (t < NT) {
        const int kt = ktile(t);
#pragma unroll
        for (int j = 0; j < 16; ++j) {
          const unsigned v = (j & 4) ? ((j & 1) ? v3 : v2) : ((j & 1) ? v1 : v0);
          __builtin_amdgcn_raw_ptr_buffer_load_lds(rs, (LAS void*)(lds + so + j * 1024), 16, v, (kt * 64 + ((j & 2) << 1) + ((j & 8) << 1)) * N * 2, 0, 0);
        }
        so = so >= 2 * SLOT ? so - 2 * SLOT : so + SLOT;
        WAITV(16);
      } else WAITV(0);
      if (t >= 1) BAR();
    }
  } else {
    if (isG) {
      auto rs = __builtin_amdgcn_make_buffer_rsrc((void*)p.x, 0, M * K * 2, 0x00020000);
      unsigned t0_, t1_, t2_, t3_, t4_;
      const unsigned pv = (unsigned)(((i16 * 8 + xcd) * 5) * K * 2 + lane * 128);
      asm volatile("buffer_load_dword %0, %5, %6, 0 offen\n\tbuffer_load_dword %1, %5, %6, %7 offen\n\t"
                   "buffer_load_dword %2, %5, %6, %8 offen\n\tbuffer_load_dword %3, %5, %6, %9 offen\n\t"
                   "buffer_load_dword %4, %5, %6, %10 offen\n\ts_waitcnt vmcnt(0)"
                   : "=&v"(t0_), "=&v"(t1_), "=&v"(t2_), "=&v"(t3_), "=&v"(t4_) : "v"(pv), "s"(rs), "s"(K * 2), "s"(K * 4), "s"(K * 6), "s"(K * 8) : "memory");
    }
    for (int t = 0; t <= NT; ++t) BAR();
  }
  if (!(w == 0 && isG)) call = __builtin_amdgcn_readfirstlane(__hip_atomic_load(cnt, __ATOMIC_RELAXED, __HIP_MEMORY_SCOPE_AGENT)) + 1u;
  char* wsl = p.ws[lane & 7];
  LBAR();
  if (w == 0) {
    const u32x4 mv = *(volatile LAS u32x4*)(misc + L_MAX);
    const unsigned m = max(max(mv[0], mv[1]), max(mv[2], mv[3]));
    if (lane < NR)
      __hip_atomic_store((u64*)(wsl + WS_GRAN + ((size_t)((call & 1) * NR + rank) * NWG + b) * 8), ((u64)call << 32) | m, __ATOMIC_RELAXED, __HIP_MEMORY_SCOPE_SYSTEM);
  }
  const char* gp = myws + WS_GRAN + (size_t)(call & 1) * NR * NWG * 8 + w * 4096 + lane * 16;
  const bool skip = lane >= 56;
  unsigned gm = 0;
  const u64 t0 = __builtin_amdgcn_s_memrealtime();
  for (;;) {
    u32x4 a, c, d, e; unsigned tm;
    asm volatile("global_load_dwordx4 %0, %5, off sc0 sc1\n\t"
                 "global_load_dwordx4 %1, %5, off offset:1024 sc0 sc1\n\t"
                 "global_load_dwordx4 %2, %5, off offset:2048 sc0 sc1\n\t"
                 "global_load_dwordx4 %3, %5, off offset:3072 sc0 sc1\n\t"
                 "global_load_dword %4, %6, off sc1\n\ts_waitcnt vmcnt(0)"
                 : "=&v"(a), "=&v"(c), "=&v"(d), "=&v"(e), "=&v"(tm) : "v"(gp), "v"(tmo) : "memory");
    if (__any(tm == call)) { bad = true; break; }
    const bool ok = a[1] == call && a[3] == call && d[1] == call && d[3] == call && (skip || (c[1] == call && c[3] == call && e[1] == call && e[3] == call));
    if (__all(ok)) {
      gm = max(max(a[0], a[2]), max(d[0], d[2]));
      if (!skip) gm = max(gm, max(max(c[0], c[2]), max(e[0], e[2])));
      break;
    }
    if (__builtin_amdgcn_s_memrealtime() - t0 > DEADLINE) { __hip_atomic_store(tmo, call, __ATOMIC_RELAXED, __HIP_MEMORY_SCOPE_AGENT); bad = true; break; }
    __builtin_amdgcn_s_sleep(1);
  }
  gm = wave_umax(gm);
  if (lane == 0) misc[L_GMAX + w] = bad ? 0xffffffffu : gm;
  LBAR();
  LBAR();
  if (w == 0 && lane == 0) {
    if (dst != rank)
      __hip_atomic_store((unsigned*)(p.ws[dst] + WS_DONE + (size_t)(rank * 16 + xcd * 2 + rt) * 128), call, __ATOMIC_RELAXED, __HIP_MEMORY_SCOPE_SYSTEM);
    __hip_atomic_store(cnt, call, __ATOMIC_RELAXED, __HIP_MEMORY_SCOPE_AGENT);
    STAMP(6);
  }
}

extern "C" void kernel_launch(int rank, int nranks, ncclComm_t comm, hipStream_t stream,
                              void* const* d_in, const int* in_sizes, int n_in,
                              void* d_out, int out_size, const int* in_dtypes, int out_dtype,
                              void* const* d_out_all, void* const* d_ws_all, size_t ws_bytes) {
  if (nranks != NR || (unsigned)rank >= (unsigned)NR || n_in < 2 || ws_bytes < WS_END || in_sizes[0] != M * K || in_sizes[1] != K * N || out_size != NR * M * NS) {
    fprintf(stderr, "unexpected problem shape\n");
    return;
  }
  static bool init = false;
  if (!init) {
    HIP_CHECK(hipFuncSetAttribute((const void*)gemm_a2a, hipFuncAttributeMaxDynamicSharedMemorySize, LDS_BYTES));
    init = true;
  }
  Params p;
  p.x = d_in[0]; p.w = d_in[1];
  for (int i = 0; i < NR; ++i) { p.out[i] = (char*)d_out_all[i]; p.ws[i] = (char*)d_ws_all[i]; }
  p.rank = rank; p.pad = 0;
  gemm_a2a<<<NWG, 512, LDS_BYTES, stream>>>(p);
  HIP_CHECK(hipGetLastError());
}
```

```cpp
#include <hip/hip_runtime.h>
#include <rccl/rccl.h>
#include <cstdio>
#include <utility>

#define HIP_CHECK(expr) do { hipError_t _e = (expr); if (_e != hipSuccess) { \
    fprintf(stderr, "HIP error %s at %s:%d\n", hipGetErrorString(_e), __FILE__, __LINE__); \
    return; } } while (0)

typedef __bf16 bf16x4 __attribute__((ext_vector_type(4)));
typedef __bf16 bf16x8 __attribute__((ext_vector_type(8)));
typedef float f32x4 __attribute__((ext_vector_type(4)));
typedef unsigned u32x4 __attribute__((ext_vector_type(4)));
typedef unsigned long long u64;
#define LAS __attribute__((address_space(3)))
#define DI __device__ __forceinline__

#define LPERM 1
#define GPERM 1
#define MPRIO 2
#define LPRIO 0
#define PDEPTH 2
#define ASLEEP 2
#define DSLEEP 4
#define WPF 0

constexpr int NR = 8;
constexpr int M = 512, N = 16384, K = 4096, NT = K / 64;
constexpr int NS = N / NR;
constexpr int TN = 256, NWG = 256;
constexpr int NU = 16, GR = NU * 16;
constexpr int XB = 144 * 128, WB = 64 * TN * 2, SLOT = XB + WB;
constexpr int MISC = 3 * SLOT, LDS_BYTES = MISC + 512;
constexpr int WROW = TN * 2;
constexpr int UB = NT * 2048;
static_assert(LDS_BYTES <= 163840 && UB <= MISC, "LDS");
constexpr int L_MAX = 0, L_GMAX = 4;
constexpr size_t WS_CNT = 0;
constexpr size_t WS_TMO = WS_CNT + NWG * 128;
constexpr size_t WS_GRAN = WS_TMO + 1024;
constexpr size_t WS_GMAX = WS_GRAN + NWG * 8;
constexpr size_t WS_DONE = WS_GMAX + 2 * 8 * 128;
constexpr size_t WS_SCR = WS_DONE + NR * 16 * 128;
constexpr size_t WS_XFLG = 1u << 20;
constexpr size_t WS_XST = 2u << 20;
constexpr size_t WS_END = WS_XST + (size_t)(NR - 1) * NU * UB;
static_assert(WS_SCR + 128 * 128 <= WS_XFLG && WS_XFLG + (NR - 1) * NU * NT * 128 <= WS_XST, "workspace map");
#define STAMP(i) do {} while (0)
#define TSTAMP() do {} while (0)
#define TSAVE() do {} while (0)
constexpr u64 DEADLINE = 30000000ull;

struct Params {
  const void* x; const void* w;
  char* out[NR]; char* ws[NR];
  int rank; int pad;
};

#define WAITV(n) asm volatile("s_waitcnt vmcnt(%0)" ::"n"(n) : "memory")
#define BAR() asm volatile("s_barrier" ::: "memory")
#define LBAR() asm volatile("s_waitcnt lgkmcnt(0)\n\ts_barrier" ::: "memory")
#define SB0() __builtin_amdgcn_sched_barrier(0)
#define DMA(rs, dst, voff, soff, aux) __builtin_amdgcn_raw_ptr_buffer_load_lds(rs, (LAS void*)(dst), 16, voff, soff, 0, aux)

template <int NI> struct Frag { bf16x8 x[NI]; bf16x4 wl[4], wh[4]; };
struct Addr { LAS char* x0; LAS char* x1; LAS char* w[4]; };

template <int KS, int R, int NI> DI void rd(Frag<NI>& f, const Addr& a) {
  if constexpr (R < 8) {
    const bf16x4 v = __builtin_amdgcn_ds_read_tr16_b64_v4bf16((LAS bf16x4*)(a.w[R >> 1] + WROW * (32 * KS + 4 * (R & 1))));
    if constexpr (R & 1) f.wh[R >> 1] = v; else f.wl[R >> 1] = v;
  } else if constexpr (R < 8 + NI) f.x[R - 8] = *(LAS bf16x8*)((KS ? a.x1 : a.x0) + 2048 * (R - 8));
}
template <int Q, int NI> DI void mm(f32x4 (&acc)[NI][4], const Frag<NI>& f) {
  acc[Q >> 2][Q & 3] = __builtin_amdgcn_mfma_f32_16x16x32_bf16(__builtin_shufflevector(f.wl[Q & 3], f.wh[Q & 3], 0, 1, 2, 3, 4, 5, 6, 7), f.x[Q >> 2], acc[Q >> 2][Q & 3], 0, 0, 0);
}
template <int Q, int KN, int NI> DI void gap(f32x4 (&acc)[NI][4], const Frag<NI>& fc, Frag<NI>& fn, Addr& a, int& sl, int& d) {
  mm<Q>(acc, fc);
  SB0();
  if constexpr (Q % 3 == 0) rd<KN, 2 * (Q / 3)>(fn, a);
  if constexpr (Q % 3 == 2) rd<KN, 2 * (Q / 3) + 1>(fn, a);
  if constexpr (KN == 1) {
    if constexpr (Q == 13) { a.w[0] += d; asm volatile("" : "+v"(a.w[0])); }
    if constexpr (Q == 16) { a.w[1] += d; asm volatile("" : "+v"(a.w[1])); }
    if constexpr (Q == 19) { a.w[2] += d; asm volatile("" : "+v"(a.w[2])); }
    if constexpr (Q == 22) { a.w[3] += d; asm volatile("" : "+v"(a.w[3])); }
    if constexpr (Q == 25) { a.x0 += d; asm volatile("" : "+v"(a.x0)); }
    if constexpr (Q == 26) { a.x1 += d; asm volatile("" : "+v"(a.x1)); }
  } else {
    if constexpr (Q == 25) { sl = sl == 2 ? 0 : sl + 1; asm volatile("" : "+s"(sl)); }
    if constexpr (Q == 27) { d = sl == 2 ? -2 * SLOT : SLOT; asm volatile("" : "+s"(d)); }
  }
  if constexpr (Q == 4 * NI - 2) __builtin_amdgcn_s_waitcnt(0xC07F);
  SB0();
}
template <int KN, int NI, int... Qs> DI void kstep(f32x4 (&acc)[NI][4], const Frag<NI>& fc, Frag<NI>& fn, Addr& a, int& sl, int& d, std::integer_sequence<int, Qs...>) {
  (gap<Qs, KN>(acc, fc, fn, a, sl, d), ...);
}
template <int NI, int... Rs> DI void rd_all(Frag<NI>& f, const Addr& a, std::integer_sequence<int, Rs...>) { (rd<0, Rs>(f, a), ...); }

DI unsigned wave_umax(unsigned v) {
#pragma unroll
  for (int o = 32; o >= 1; o >>= 1) { unsigned t = __shfl_xor(v, o); v = t > v ? t : v; }
  return v;
}
DI bool timed_out(unsigned* tmo, unsigned call, u64 t0) {
  if (__hip_atomic_load(tmo, __ATOMIC_RELAXED, __HIP_MEMORY_SCOPE_AGENT) == call || __builtin_amdgcn_s_memrealtime() - t0 > DEADLINE) {
    __hip_atomic_store(tmo, call, __ATOMIC_RELAXED, __HIP_MEMORY_SCOPE_AGENT);
    return true;
  }
  return false;
}
DI int gperm(int u, int rt) { return GPERM ? (u & ~7) | ((u + rt) & 7) : u; }

template <int NI> DI void mfma_role(LAS char* lds, int wid, int lane, int b, char* myws, char* obase, int rank, int u0) {
  volatile LAS unsigned* misc = (volatile LAS unsigned*)(lds + MISC);
  const int fr = lane & 15, g = lane >> 4, q = (lane & 15) >> 2, pp = lane & 3;
  const int xo0 = fr * 128 + ((g ^ (fr >> 1)) << 4);
  const int fk = q | ((g & 1) << 2);
  const int wo0 = XB + WROW * (8 * g + q) + ((wid >> 1) * 256) + ((((wid & 1) * 4) ^ fk) << 5) + 8 * pp;
  f32x4 acc[NI][4];
#pragma unroll
  for (int i = 0; i < NI; ++i)
#pragma unroll
    for (int j = 0; j < 4; ++j) acc[i][j] = (f32x4){0.f, 0.f, 0.f, 0.f};
  Frag<NI> f0, f1;
  Addr a = {lds + xo0, lds + (xo0 ^ 64), {lds + wo0, lds + (wo0 ^ 32), lds + (wo0 ^ 64), lds + (wo0 ^ 96)}};
  SB0();
  BAR();
  rd_all(f0, a, std::make_integer_sequence<int, 8 + NI>{});
  __builtin_amdgcn_s_waitcnt(0xC07F);
  int sl = 0, d = SLOT;
  for (int s = 0; s < NT; ++s) {
    SB0();
    kstep<1>(acc, f0, f1, a, sl, d, std::make_integer_sequence<int, 4 * NI>{});
    BAR();
    SB0();
    kstep<0>(acc, f1, f0, a, sl, d, std::make_integer_sequence<int, 4 * NI>{});
  }
  SB0();
  float m = 0.f;
#pragma unroll
  for (int i = 0; i < NI; ++i)
#pragma unroll
    for (int j = 0; j < 4; ++j) {
      m = fmaxf(fmaxf(m, __builtin_fabsf(acc[i][j][0])), __builtin_fabsf(acc[i][j][1]));
      m = fmaxf(fmaxf(m, __builtin_fabsf(acc[i][j][2])), __builtin_fabsf(acc[i][j][3]));
    }
  const unsigned mu = wave_umax(__float_as_uint(m));
  if (lane == 0) misc[L_MAX + wid] = mu;
  LBAR();
  LBAR();
  if (wid == 0) STAMP(3);
  const unsigned gm = misc[L_GMAX];
  const bool bad = gm == 0xffffffffu;
  const float amax = __uint_as_float(gm);
  const float inv = amax > 0.f ? 127.f / amax : 0.f;
  char* orow = obase + (size_t)fr * NS + wid * 64 + 16 * g;
#pragma unroll
  for (int i = 0; i < NI; ++i) {
    unsigned d4[4];
#pragma unroll
    for (int j = 0; j < 4; ++j) {
      unsigned a0 = __float_as_uint(__builtin_fmaf(acc[i][j][0], inv, 12582912.f)), a1 = __float_as_uint(__builtin_fmaf(acc[i][j][1], inv, 12582912.f));
      unsigned a2 = __float_as_uint(__builtin_fmaf(acc[i][j][2], inv, 12582912.f)), a3 = __float_as_uint(__builtin_fmaf(acc[i][j][3], inv, 12582912.f));
      d4[j] = __builtin_amdgcn_perm(a1, a0, 0x0c0c0400) | __builtin_amdgcn_perm(a3, a2, 0x04000c0c);
    }
    auto r0 = __builtin_amdgcn_permlane32_swap(d4[0], d4[2], false, false); d4[0] = r0[0]; d4[2] = r0[1];
    auto r1 = __builtin_amdgcn_permlane32_swap(d4[1], d4[3], false, false); d4[1] = r1[0]; d4[3] = r1[1];
    auto r2 = __builtin_amdgcn_permlane16_swap(d4[0], d4[1], false, false); d4[0] = r2[0]; d4[1] = r2[1];
    auto r3 = __builtin_amdgcn_permlane16_swap(d4[2], d4[3], false, false); d4[2] = r3[0]; d4[3] = r3[1];
    u32x4 v = {d4[0], d4[1], d4[2], d4[3]};
    if (bad) v = (u32x4){0x80808080u, 0x80808080u, 0x80808080u, 0x80808080u};
    const int U = u0 + i;
    const int r16 = NI == 8 ? i * 16 : U < 32 ? rank * M + U * 16 : ((rank + (U >> 4) - 1) & 7) * M + (U & 15) * 16;
    asm volatile("global_store_dwordx4 %0, %1, off sc0 sc1\n\ts_nop 1" ::"v"(orow + (size_t)r16 * NS), "v"(v) : "memory");
  }
  if (wid == 0) STAMP(4);
  WAITV(0);
  LBAR();
  if (wid == 0) STAMP(5);
}

__global__ __launch_bounds__(512, 2) void gemm_a2a(const Params p) {
  extern __shared__ __attribute__((aligned(16))) char smem[];
  LAS char* lds = (LAS char*)smem;
  volatile LAS unsigned* misc = (volatile LAS unsigned*)(lds + MISC);
  const int tid = threadIdx.x, lane = tid & 63;
  const int wid = __builtin_amdgcn_readfirstlane(tid >> 6);
  const int b = blockIdx.x, xcd = b & 7, idx = b >> 3;
  const int rank = p.rank & 7;
  char* myws = p.ws[rank];
  unsigned* cnt = (unsigned*)(myws + WS_CNT + b * 128);
  unsigned* tmo = (unsigned*)(myws + WS_TMO);
  if (wid == 0) STAMP(0);
  auto rx = __builtin_amdgcn_make_buffer_rsrc((void*)p.x, 0, M * K * 2, 0x00020000);
  const unsigned xv0 = (unsigned)((lane >> 3) * K * 2 + (((lane & 7) ^ (lane >> 4)) << 4)), xv1 = (xv0 ^ 64) + 8 * K * 2;

  if (idx >= 30) {
    const int un = xcd * 2 + (idx & 1);
    auto stage = [&](int kt) { DMA(rx, lds + kt * 2048, xv0, un * UB + kt * 128, 0); DMA(rx, lds + kt * 2048 + 1024, xv1, un * UB + kt * 128, 0); };
    stage(wid);
    unsigned c0;
    asm volatile("global_load_dword %0, %1, off sc1\n\ts_waitcnt vmcnt(0)" : "=&v"(c0) : "v"(cnt) : "memory");
    const unsigned call = __builtin_amdgcn_readfirstlane(c0) + 1u;
    const int R = (wid - 1) * NU + un;
    const int rot = GPERM ? (32 + R) / 9 : 0;
    char* pw = p.ws[(rank - wid) & 7];
    char* dp = pw + WS_XST + (size_t)R * UB + lane * 16;
    char* fn = pw + WS_XFLG + (size_t)R * NT * 128;
    char* f1 = myws + WS_SCR + (un * 8 + wid) * 128;
    char* f2 = f1;
    LAS char* la = lds + lane * 16;
    for (int r = 0; r < 8; ++r) {
      BAR();
      if (wid == 0) {
        if (r < 7) { stage(8 * (r + 1)); WAITV(0); }
        continue;
      }
      u32x4 a0 = *(LAS u32x4*)(la + r * 16384 + (rot & 7) * 2048), a1 = *(LAS u32x4*)(la + r * 16384 + (rot & 7) * 2048 + 1024);
#pragma unroll
      for (int j = 0; j < 8; ++j) {
        u32x4 n0 = a0, n1 = a1;
        if (j < 7) { n0 = *(LAS u32x4*)(la + r * 16384 + ((rot + j + 1) & 7) * 2048); n1 = *(LAS u32x4*)(la + r * 16384 + ((rot + j + 1) & 7) * 2048 + 1024); }
        asm volatile("global_store_dwordx4 %0, %1, off sc0 sc1\n\t"
                     "global_store_dwordx4 %0, %2, off offset:1024 sc0 sc1\n\t"
                     "s_waitcnt vmcnt(%5)\n\t"
                     "s_mov_b64 exec, 1\n\tglobal_store_dword %3, %4, off sc0 sc1\n\ts_mov_b64 exec, -1\n\ts_nop 0"
                     ::"v"(dp), "v"(a0), "v"(a1), "v"(PDEPTH == 3 ? f2 : f1), "v"(call), "n"(PDEPTH == 3 ? 6 : 3) : "memory");
        f2 = f1; f1 = fn; fn += 128; dp += 2048;
        if (j == 2 && r < 7) stage(8 * (r + 1) + wid);
        a0 = n0; a1 = n1;
      }
    }
    WAITV(0);
    if (lane == 0 && wid) {
      if (PDEPTH == 3) __hip_atomic_store((unsigned*)f2, call, __ATOMIC_RELAXED, __HIP_MEMORY_SCOPE_SYSTEM);
      __hip_atomic_store((unsigned*)f1, call, __ATOMIC_RELAXED, __HIP_MEMORY_SCOPE_SYSTEM);
    }
    if (wid == 1) STAMP(7);
    if (un == 0 && wid == 0) {
      const char* gp = myws + WS_GRAN + lane * 16;
      const bool skip = lane >= 56;
      unsigned gm = 0xffffffffu;
      const u64 t0 = __builtin_amdgcn_s_memrealtime();
      for (unsigned np = 1;; ++np) {
        u32x4 a, c;
        asm volatile("global_load_dwordx4 %0, %2, off sc1\n\tglobal_load_dwordx4 %1, %2, off offset:1024 sc1\n\ts_waitcnt vmcnt(0)"
                     : "=&v"(a), "=&v"(c) : "v"(gp) : "memory");
        if (__all(a[1] == call && a[3] == call && (skip || (c[1] == call && c[3] == call)))) {
          gm = wave_umax(skip ? max(a[0], a[2]) : max(max(a[0], a[2]), max(c[0], c[2])));
          break;
        }
        if ((np & 63u) == 0 && __any(timed_out(tmo, call, t0))) break;
      }
      __hip_atomic_store((u64*)(p.ws[lane & 7] + WS_GMAX + ((call & 1) * 8 + (lane >> 3)) * 128 + rank * 8), ((u64)call << 32) | gm, __ATOMIC_RELAXED, __HIP_MEMORY_SCOPE_SYSTEM);
      STAMP(8);
    }
    const int fi = un * 8 + wid;
    if (lane == 0 && (fi >> 4) != rank) {
      unsigned* fl = (unsigned*)(myws + WS_DONE + (size_t)fi * 128);
      const u64 t1 = __builtin_amdgcn_s_memrealtime();
      unsigned np = 0;
      while (__hip_atomic_load(fl, __ATOMIC_RELAXED, __HIP_MEMORY_SCOPE_SYSTEM) < call) {
        if ((++np & 63u) == 0 && timed_out(tmo, call, t1)) { p.out[rank][(size_t)rank * M * NS + fi] = (char)-128; break; }
        __builtin_amdgcn_s_sleep(DSLEEP);
      }
    }
    if (tid == 0) __hip_atomic_store(cnt, call, __ATOMIC_RELAXED, __HIP_MEMORY_SCOPE_AGENT);
    if (wid == 0) STAMP(6);
    return;
  }

  const bool isG = idx < 16;
  const int il = idx - 16;
  const int rt = isG ? (xcd & 1) * 8 + (idx & 7) : (il & 1);
  const int dst = isG ? rank : (rank + 1 + (il >> 1)) & 7;
  const int cl = isG ? (xcd >> 1) * 2 + (idx >> 3) : xcd;
  const int ct = dst * 8 + cl;
  if (wid < 4) {
    __builtin_amdgcn_s_setprio(MPRIO);
    if (isG) mfma_role<9>(lds, wid, lane, b, myws, p.out[rank] + cl * TN, rank, rt * 9);
    else mfma_role<8>(lds, wid, lane, b, myws, p.out[dst] + (size_t)(rank * M + GR + rt * 128) * NS + cl * TN, rank, 0);
    return;
  }
  const int w = wid - 4;
  bool bad = false;
  unsigned call = 0;
  auto ktile = [&](int u) {
    const int uu = (u + ((il >> 1) & 1)) & 63;
    const int kl = LPERM ? ((uu & 1) ? ((uu - 3) & 63) + 1 - rt : uu + rt) : u;
    return isG ? gperm(u, rt) : kl;
  };
  __builtin_amdgcn_s_setprio(LPRIO);
  if (w == 0 && isG) {
    const int u0 = rt * 9;
    auto rg = __builtin_amdgcn_make_buffer_rsrc((void*)(myws + WS_XST), 0, (NR - 1) * NU * UB, 0x00020000);
    const unsigned gv = lane * 16, gv1 = gv + 1024;
    const int lu = u0 + lane % 9, lp = lane / 9;
    const char* fp = myws + WS_XFLG + (size_t)max(lu - 32, 0) * NT * 128;
    unsigned rdy = u0 + 8 < 32 ? NT : 0, np = 0;
    if (rdy == 0) call = __builtin_amdgcn_readfirstlane(__hip_atomic_load(cnt, __ATOMIC_RELAXED, __HIP_MEMORY_SCOPE_AGENT)) + 1u;
    const u64 t0 = __builtin_amdgcn_s_memrealtime();
    auto seen = [&](unsigned f) { const u64 miss = ~__ballot((f >= call || lu < 32) && lane < 63); rdy = min((unsigned)NT, rdy + (unsigned)__builtin_ctzll(miss) / 9u); };
    int so = 0;
    for (int t = 0; t < NT + 2; ++t) {
      if (t < NT) {
        while (rdy <= (unsigned)t) {
          seen(__hip_atomic_load((const unsigned*)(fp + min(rdy + lp, NT - 1u) * 128), __ATOMIC_RELAXED, __HIP_MEMORY_SCOPE_SYSTEM));
          if (rdy <= (unsigned)t && (++np & 63u) == 0 && __any(timed_out(tmo, call, t0))) { bad = true; rdy = NT; }
        }
        const bool ahead = rdy < NT && rdy < (unsigned)t + 4;
        if (ahead) __builtin_amdgcn_global_load_lds((const unsigned*)(fp + min(rdy + lp, NT - 1u) * 128), (LAS unsigned*)(lds + MISC + 256), 4, 0, 17);
        const int kt = gperm(t, rt);
        const unsigned ox = u0 * UB + kt * 128, og = (u0 - 32) * UB + t * 2048;
#define XUNITS(NOWN) _Pragma("unroll") for (int i = 0; i < 9; ++i) { \
          if (i < NOWN) { DMA(rx, lds + so + i * 2048, xv0, ox + i * UB, 0); DMA(rx, lds + so + i * 2048 + 1024, xv1, ox + i * UB, 0); } \
          else { DMA(rg, lds + so + i * 2048, gv, og + i * UB, 17); DMA(rg, lds + so + i * 2048 + 1024, gv1, og + i * UB, 17); } }
        if (rt > 3) { XUNITS(0) } else if (rt == 3) { XUNITS(5) } else { XUNITS(9) }
        so = so == 2 * SLOT ? 0 : so + SLOT;
        WAITV(18);
        if (ahead) seen(*(volatile LAS unsigned*)(lds + MISC + 256 + lane * 4));
      } else WAITV(0);
      TSTAMP();
      if (t >= 1) BAR();
    }
    TSAVE();
  } else if (w == 0) {
    int so = 0;
    for (int t = 0; t < NT + 2; ++t) {
      if (t < NT) {
        const unsigned go = (unsigned)(NU + rt * 8) * UB + ktile(t) * 128;
#pragma unroll
        for (int i = 0; i < 8; ++i) { DMA(rx, lds + so + i * 2048, xv0, go + i * UB, 0); DMA(rx, lds + so + i * 2048 + 1024, xv1, go + i * UB, 0); }
        so = so == 2 * SLOT ? 0 : so + SLOT;
        WAITV(16);
      } else WAITV(0);
      TSTAMP();
      if (t >= 1) BAR();
    }
    TSAVE();
  } else if (w < 3) {
    auto rs = __builtin_amdgcn_make_buffer_rsrc((void*)p.w, 0, K * N * 2, 0x00020000);
    const unsigned v0 = (unsigned)(((lane >> 5) + (w - 1) * 32) * N * 2 + ct * TN * 2 + (((lane & 31) ^ ((lane >> 5) << 1)) << 4));
    const unsigned v1 = (v0 ^ 64) + 2 * N * 2, v2 = (v0 ^ 128) + 8 * N * 2, v3 = (v0 ^ 192) + 10 * N * 2;
    int so = XB + (w - 1) * 16384;
    for (int t = 0; t < NT + 2; ++t) {
      if (t < NT) {
        const int kt = ktile(t);
#pragma unroll
        for (int j = 0; j < 16; ++j)
          DMA(rs, lds + so + j * 1024, (j & 4) ? ((j & 1) ? v3 : v2) : ((j & 1) ? v1 : v0), (kt * 64 + ((j & 2) << 1) + ((j & 8) << 1)) * N * 2, 0);
        so = so >= 2 * SLOT ? so - 2 * SLOT : so + SLOT;
        WAITV(16);
      } else WAITV(0);
      TSTAMP();
      if (t >= 1) BAR();
    }
    TSAVE();
  } else {
    if (isG && rt >= 3) {
      const int gi = (rt - 3) * 8 + cl;
      unsigned t0_, t1_, t2_, t3_, t4_;
      const unsigned pv = (unsigned)(gi * 5 * K * 2 + lane * 128);
      asm volatile("buffer_load_dword %0, %5, %6, 0 offen\n\tbuffer_load_dword %1, %5, %6, %7 offen\n\t"
                   "buffer_load_dword %2, %5, %6, %8 offen\n\tbuffer_load_dword %3, %5, %6, %9 offen\n\t"
                   "buffer_load_dword %4, %5, %6, %10 offen\n\ts_waitcnt vmcnt(0)"
                   : "=&v"(t0_), "=&v"(t1_), "=&v"(t2_), "=&v"(t3_), "=&v"(t4_) : "v"(pv), "s"(rx), "s"(K * 2), "s"(K * 4), "s"(K * 6), "s"(K * 8) : "memory");
      if (WPF) {
        auto rs = __builtin_amdgcn_make_buffer_rsrc((void*)p.w, 0, K * N * 2, 0x00020000);
        for (int i = 0; i < WPF; ++i) {
          const unsigned wv = (unsigned)((256 + i * 104 + gi) * N * 2 + lane * 128);
          asm volatile("buffer_load_dword %0, %4, %5, 0 offen\n\tbuffer_load_dword %1, %4, %5, %6 offen\n\t"
                       "buffer_load_dword %2, %4, %5, %7 offen\n\tbuffer_load_dword %3, %4, %5, %8 offen"
                       : "+v"(t0_), "+v"(t1_), "+v"(t2_), "+v"(t3_) : "v"(wv), "s"(rs), "s"(8192), "s"(16384), "s"(24576) : "memory");
        }
        asm volatile("s_waitcnt vmcnt(0)" : "+v"(t0_), "+v"(t1_), "+v"(t2_), "+v"(t3_)::"memory");
      }
    }
    for (int t = 0; t <= NT; ++t) BAR();
  }
  if (w == 0) call = __builtin_amdgcn_readfirstlane(__hip_atomic_load(cnt, __ATOMIC_RELAXED, __HIP_MEMORY_SCOPE_AGENT)) + 1u;
  LBAR();
  if (w == 0) {
    const u32x4 mv = *(volatile LAS u32x4*)(misc + L_MAX);
    const unsigned m = bad ? 0xffffffffu : max(max(mv[0], mv[1]), max(mv[2], mv[3]));
    if (lane == 0) __hip_atomic_store((u64*)(myws + WS_GRAN + b * 8), ((u64)call << 32) | m, __ATOMIC_RELAXED, __HIP_MEMORY_SCOPE_AGENT);
    const char* gp = myws + WS_GMAX + ((call & 1) * 8 + xcd) * 128 + (lane & 7) * 8;
    unsigned gm = 0xffffffffu;
    const u64 t0 = __builtin_amdgcn_s_memrealtime();
    for (;;) {
      u64 gq; unsigned tm;
      asm volatile("global_load_dwordx2 %0, %2, off sc0 sc1\n\tglobal_load_dword %1, %3, off sc1\n\ts_waitcnt vmcnt(0)"
                   : "=&v"(gq), "=&v"(tm) : "v"(gp), "v"(tmo) : "memory");
      if (__any(tm == call)) break;
      if (__all((unsigned)(gq >> 32) == call)) {
        gm = (unsigned)gq;
        gm = max(gm, (unsigned)__shfl_xor(gm, 1)); gm = max(gm, (unsigned)__shfl_xor(gm, 2)); gm = max(gm, (unsigned)__shfl_xor(gm, 4));
        break;
      }
      if (__builtin_amdgcn_s_memrealtime() - t0 > DEADLINE) { __hip_atomic_store(tmo, call, __ATOMIC_RELAXED, __HIP_MEMORY_SCOPE_AGENT); break; }
      __builtin_amdgcn_s_sleep(ASLEEP);
    }
    if (lane == 0) misc[L_GMAX] = gm;
  }
  LBAR();
  LBAR();
  if (w == 0 && lane == 0) {
    if (!isG)
      __hip_atomic_store((unsigned*)(p.ws[dst] + WS_DONE + (size_t)(rank * 16 + xcd * 2 + rt) * 128), call, __ATOMIC_RELAXED, __HIP_MEMORY_SCOPE_SYSTEM);
    __hip_atomic_store(cnt, call, __ATOMIC_RELAXED, __HIP_MEMORY_SCOPE_AGENT);
    STAMP(6);
  }
}

extern "C" void kernel_launch(int rank, int nranks, ncclComm_t comm, hipStream_t stream,
                              void* const* d_in, const int* in_sizes, int n_in,
                              void* d_out, int out_size, const int* in_dtypes, int out_dtype,
                              void* const* d_out_all, void* const* d_ws_all, size_t ws_bytes) {
  if (nranks != NR || (unsigned)rank >= (unsigned)NR || n_in < 2 || ws_bytes < WS_END || in_sizes[0] != M * K || in_sizes[1] != K * N || out_size != NR * M * NS) {
    fprintf(stderr, "unexpected problem shape\n");
    return;
  }
  static bool init = false;
  if (!init) {
    HIP_CHECK(hipFuncSetAttribute((const void*)gemm_a2a, hipFuncAttributeMaxDynamicSharedMemorySize, LDS_BYTES));
    init = true;
  }
  Params p;
  p.x = d_in[0]; p.w = d_in[1];
  for (int i = 0; i < NR; ++i) { p.out[i] = (char*)d_out_all[i]; p.ws[i] = (char*)d_ws_all[i]; }
  p.rank = rank; p.pad = 0;
  gemm_a2a<<<NWG, 512, LDS_BYTES, stream>>>(p);
  HIP_CHECK(hipGetLastError());
}
```

```cpp
#include <hip/hip_runtime.h>
#include <rccl/rccl.h>
#include <cstdio>
#include <utility>

#define HIP_CHECK(expr) do { hipError_t _e = (expr); if (_e != hipSuccess) { \
    fprintf(stderr, "HIP error %s at %s:%d\n", hipGetErrorString(_e), __FILE__, __LINE__); \
    return; } } while (0)

typedef __bf16 bf16x4 __attribute__((ext_vector_type(4)));
typedef __bf16 bf16x8 __attribute__((ext_vector_type(8)));
typedef float f32x4 __attribute__((ext_vector_type(4)));
typedef unsigned u32x4 __attribute__((ext_vector_type(4)));
typedef unsigned long long u64;
#define LAS __attribute__((address_space(3)))
#define DI __device__ __forceinline__

#define LPERM 0
#define GPERM 2
#define MPRIO 2
#define LPRIO 0
#define PDEPTH 2
#define ASLEEP 2
#define DSLEEP 4
#define CMIX 0
#define CHELP 0
#define CH0 6
#define CWIN 4
#define PACE 0
#define NOCOMM 0
#define SLOW 0
#define CUTG 0
#define SOLO 0
#define XAUX 17
#define XSPEC 1
#define G44 0
#define LDIST 3
#define LNT 2
#define NTAUX 2
#define GNT 0
#define HELP 0

constexpr int NR = 8;
constexpr int M = 512, N = 16384, K = 4096, NT = K / 64;
constexpr int NS = N / NR;
constexpr int TN = 256, NWG = 256;
constexpr int NU = 16, GR = NU * 16;
constexpr int XB = 144 * 128, WB = 64 * TN * 2, SLOT = XB + WB;
constexpr int MISC = 3 * SLOT, LDS_BYTES = MISC + 512;
constexpr int WROW = TN * 2;
constexpr int UB = NT * 2048;
static_assert(LDS_BYTES <= 163840 && UB <= MISC, "LDS");
constexpr int L_MAX = 0, L_GMAX = 4;
constexpr size_t WS_CNT = 0;
constexpr size_t WS_TMO = WS_CNT + NWG * 128;
constexpr size_t WS_GRAN = WS_TMO + 1024;
constexpr size_t WS_GMAX = WS_GRAN + NWG * 8;
constexpr size_t WS_DONE = WS_GMAX + 2 * 8 * 128;
constexpr size_t WS_SCR = WS_DONE + NR * 16 * 128;
constexpr size_t WS_XFLG = 1u << 20;
constexpr size_t WS_XST = 2u << 20;
constexpr size_t WS_END = WS_XST + (size_t)(NR - 1) * NU * UB;
static_assert(WS_SCR + 128 * 128 <= WS_XFLG && WS_XFLG + (NR - 1) * NU * 128 <= WS_XST, "workspace map");
#define STAMP(i) do {} while (0)
#define TSTAMP() do {} while (0)
#define TSAVE() do {} while (0)
constexpr u64 DEADLINE = 30000000ull;

struct Params {
  const void* x; const void* w;
  char* out[NR]; char* ws[NR];
  int rank; int pad;
};

#define WAITV(n) asm volatile("s_waitcnt vmcnt(%0)" ::"n"(n) : "memory")
#define BAR() asm volatile("s_barrier" ::: "memory")
#define LBAR() asm volatile("s_waitcnt lgkmcnt(0)\n\ts_barrier" ::: "memory")
#define SB0() __builtin_amdgcn_sched_barrier(0)
#define DMA(rs, dst, voff, soff, aux) __builtin_amdgcn_raw_ptr_buffer_load_lds(rs, (LAS void*)(dst), 16, voff, soff, 0, aux)

template <int NI> struct Frag { bf16x8 x[NI]; bf16x4 wl[4], wh[4]; };
struct Addr { LAS char* x0; LAS char* x1; LAS char* w[4]; };

template <int KS, int R, int NI> DI void rd(Frag<NI>& f, const Addr& a) {
  if constexpr (R < 8) {
    const bf16x4 v = __builtin_amdgcn_ds_read_tr16_b64_v4bf16((LAS bf16x4*)(a.w[R >> 1] + WROW * (32 * KS + 4 * (R & 1))));
    if constexpr (R & 1) f.wh[R >> 1] = v; else f.wl[R >> 1] = v;
  } else if constexpr (R < 8 + NI) f.x[R - 8] = *(LAS bf16x8*)((KS ? a.x1 : a.x0) + 2048 * (R - 8));
}
template <int Q, int NI> DI void mm(f32x4 (&acc)[NI][4], const Frag<NI>& f) {
  acc[Q >> 2][Q & 3] = __builtin_amdgcn_mfma_f32_16x16x32_bf16(__builtin_shufflevector(f.wl[Q & 3], f.wh[Q & 3], 0, 1, 2, 3, 4, 5, 6, 7), f.x[Q >> 2], acc[Q >> 2][Q & 3], 0, 0, 0);
}
template <int Q, int KN, int NI> DI void gap(f32x4 (&acc)[NI][4], const Frag<NI>& fc, Frag<NI>& fn, Addr& a, int& sl, int& d) {
  mm<Q>(acc, fc);
  SB0();
  if constexpr (Q % 3 == 0) rd<KN, 2 * (Q / 3)>(fn, a);
  if constexpr (Q % 3 == 2) rd<KN, 2 * (Q / 3) + 1>(fn, a);
  if constexpr (KN == 1) {
    if constexpr (Q == 13) { a.w[0] += d; asm volatile("" : "+v"(a.w[0])); }
    if constexpr (Q == 16) { a.w[1] += d; asm volatile("" : "+v"(a.w[1])); }
    if constexpr (Q == 19) { a.w[2] += d; asm volatile("" : "+v"(a.w[2])); }
    if constexpr (Q == 22) { a.w[3] += d; asm volatile("" : "+v"(a.w[3])); }
    if constexpr (Q == 25) { a.x0 += d; asm volatile("" : "+v"(a.x0)); }
    if constexpr (Q == 26) { a.x1 += d; asm volatile("" : "+v"(a.x1)); }
  } else {
    if constexpr (Q == 25) { sl = sl == 2 ? 0 : sl + 1; asm volatile("" : "+s"(sl)); }
    if constexpr (Q == 27) { d = sl == 2 ? -2 * SLOT : SLOT; asm volatile("" : "+s"(d)); }
  }
  if constexpr (Q == 4 * NI - 2) __builtin_amdgcn_s_waitcnt(0xC07F);
  SB0();
}
template <int KN, int NI, int... Qs> DI void kstep(f32x4 (&acc)[NI][4], const Frag<NI>& fc, Frag<NI>& fn, Addr& a, int& sl, int& d, std::integer_sequence<int, Qs...>) {
  (gap<Qs, KN>(acc, fc, fn, a, sl, d), ...);
}
template <int NI, int... Rs> DI void rd_all(Frag<NI>& f, const Addr& a, std::integer_sequence<int, Rs...>) { (rd<0, Rs>(f, a), ...); }

DI unsigned wave_umax(unsigned v) {
#pragma unroll
  for (int o = 32; o >= 1; o >>= 1) { unsigned t = __shfl_xor(v, o); v = t > v ? t : v; }
  return v;
}
DI bool timed_out(unsigned* tmo, unsigned call, u64 t0) {
  if (__hip_atomic_load(tmo, __ATOMIC_RELAXED, __HIP_MEMORY_SCOPE_AGENT) == call || __builtin_amdgcn_s_memrealtime() - t0 > DEADLINE) {
    __hip_atomic_store(tmo, call, __ATOMIC_RELAXED, __HIP_MEMORY_SCOPE_AGENT);
    return true;
  }
  return false;
}
DI int gperm(int u, int rt) {
  if (GPERM == 2) return (u & ~7) | ((u + rt) & 7);
  if (GPERM == 4) return (u & ~3) | ((u + rt) & 3);
  if (GPERM == 5) return (u & ~3) | ((u + 2 * rt) & 3);
  if (GPERM == 3) return ((u ^ rt) & 3) == 0 ? (u + 4) & 63 : u;
  return GPERM && ((u ^ rt) & 7) == 0 ? (u + 8) & 63 : u;
}

DI void help_slot(int e, int par, int& j, int& ho) {
  if (par == 0) { const int g = e >> 2, i = e & 3; if (g == 2) { j = 1 + 2 * (i % 3); ho = 4 + i / 3; } else { j = 2 * i; ho = g == 3 ? 6 : 3 * g; } }
  else if (e >= 6 && e < 10) { j = 2 * (e - 6); ho = 4; }
  else { const int f = e < 6 ? e : e - 4, g = f / 3; j = 1 + 2 * (f % 3); ho = 2 * g + (g >> 1); }
}

template <int NI> DI void mfma_role(LAS char* lds, int wid, int lane, int b, char* myws, char* obase, int rank, int u0) {
  volatile LAS unsigned* misc = (volatile LAS unsigned*)(lds + MISC);
  const int fr = lane & 15, g = lane >> 4, q = (lane & 15) >> 2, pp = lane & 3;
  const int xo0 = fr * 128 + ((g ^ (fr >> 1)) << 4);
  const int fk = q | ((g & 1) << 2);
  const int wo0 = XB + WROW * (8 * g + q) + ((wid >> 1) * 256) + ((((wid & 1) * 4) ^ fk) << 5) + 8 * pp;
  f32x4 acc[NI][4];
#pragma unroll
  for (int i = 0; i < NI; ++i)
#pragma unroll
    for (int j = 0; j < 4; ++j) acc[i][j] = (f32x4){0.f, 0.f, 0.f, 0.f};
  Frag<NI> f0, f1;
  Addr a = {lds + xo0, lds + (xo0 ^ 64), {lds + wo0, lds + (wo0 ^ 32), lds + (wo0 ^ 64), lds + (wo0 ^ 96)}};
  SB0();
  BAR();
  rd_all(f0, a, std::make_integer_sequence<int, 8 + NI>{});
  __builtin_amdgcn_s_waitcnt(0xC07F);
  int sl = 0, d = SLOT;
  for (int s = 0; s < NT; ++s) {
    SB0();
    kstep<1>(acc, f0, f1, a, sl, d, std::make_integer_sequence<int, 4 * NI>{});
    BAR();
    SB0();
    kstep<0>(acc, f1, f0, a, sl, d, std::make_integer_sequence<int, 4 * NI>{});
    if (SLOW) __builtin_amdgcn_s_sleep(SLOW);
  }
  SB0();
  float m = 0.f;
#pragma unroll
  for (int i = 0; i < NI; ++i)
#pragma unroll
    for (int j = 0; j < 4; ++j) {
      m = fmaxf(fmaxf(m, __builtin_fabsf(acc[i][j][0])), __builtin_fabsf(acc[i][j][1]));
      m = fmaxf(fmaxf(m, __builtin_fabsf(acc[i][j][2])), __builtin_fabsf(acc[i][j][3]));
    }
  const unsigned mu = wave_umax(__float_as_uint(m));
  if (lane == 0) misc[L_MAX + wid] = mu;
  LBAR();
  LBAR();
  if (wid == 0) STAMP(3);
  const unsigned gm = misc[L_GMAX];
  const bool bad = gm == 0xffffffffu;
  const float amax = __uint_as_float(gm);
  const float inv = amax > 0.f ? 127.f / amax : 0.f;
  char* orow = obase + (size_t)fr * NS + wid * 64 + 16 * g;
#pragma unroll
  for (int i = 0; i < NI; ++i) {
    unsigned d4[4];
#pragma unroll
    for (int j = 0; j < 4; ++j) {
      unsigned a0 = __float_as_uint(__builtin_fmaf(acc[i][j][0], inv, 12582912.f)), a1 = __float_as_uint(__builtin_fmaf(acc[i][j][1], inv, 12582912.f));
      unsigned a2 = __float_as_uint(__builtin_fmaf(acc[i][j][2], inv, 12582912.f)), a3 = __float_as_uint(__builtin_fmaf(acc[i][j][3], inv, 12582912.f));
      d4[j] = __builtin_amdgcn_perm(a1, a0, 0x0c0c0400) | __builtin_amdgcn_perm(a3, a2, 0x04000c0c);
    }
    auto r0 = __builtin_amdgcn_permlane32_swap(d4[0], d4[2], false, false); d4[0] = r0[0]; d4[2] = r0[1];
    auto r1 = __builtin_amdgcn_permlane32_swap(d4[1], d4[3], false, false); d4[1] = r1[0]; d4[3] = r1[1];
    auto r2 = __builtin_amdgcn_permlane16_swap(d4[0], d4[1], false, false); d4[0] = r2[0]; d4[1] = r2[1];
    auto r3 = __builtin_amdgcn_permlane16_swap(d4[2], d4[3], false, false); d4[2] = r3[0]; d4[3] = r3[1];
    u32x4 v = {d4[0], d4[1], d4[2], d4[3]};
    if (bad) v = (u32x4){0x80808080u, 0x80808080u, 0x80808080u, 0x80808080u};
    const int U = u0 + i;
    const int r16 = NI == 8 ? i * 16 : U < 32 ? rank * M + U * 16 : ((rank + (U >> 4) - 1) & 7) * M + (U & 15) * 16;
    asm volatile("global_store_dwordx4 %0, %1, off sc0 sc1\n\ts_nop 1" ::"v"(orow + (size_t)r16 * NS), "v"(v) : "memory");
  }
  if (wid == 0) STAMP(4);
  WAITV(0);
  LBAR();
  if (wid == 0) STAMP(5);
}

__global__ __launch_bounds__(512, 2) void gemm_a2a(const Params p) {
  extern __shared__ __attribute__((aligned(16))) char smem[];
  LAS char* lds = (LAS char*)smem;
  volatile LAS unsigned* misc = (volatile LAS unsigned*)(lds + MISC);
  const int tid = threadIdx.x, lane = tid & 63;
  const int wid = __builtin_amdgcn_readfirstlane(tid >> 6);
  const int b = blockIdx.x, xcd = b & 7, idx = b >> 3;
  const int rank = p.rank & 7;
  char* myws = p.ws[rank];
  unsigned* cnt = (unsigned*)(myws + WS_CNT + b * 128);
  unsigned* tmo = (unsigned*)(myws + WS_TMO);
  if (wid == 0) STAMP(0);
  auto rx = __builtin_amdgcn_make_buffer_rsrc((void*)p.x, 0, M * K * 2, 0x00020000);
  const unsigned xv0 = (unsigned)((lane >> 3) * K * 2 + (((lane & 7) ^ (lane >> 4)) << 4)), xv1 = (xv0 ^ 64) + 8 * K * 2;

  const int x4 = xcd >> 1;
  const bool isC = XSPEC ? (!(xcd & 1) && idx >= 28) : idx >= 30;
  if (isC) {
    const int un = XSPEC ? x4 * 4 + (idx & 3) : xcd * 2 + (idx & 1);
    auto stage = [&](int kt) { DMA(rx, lds + kt * 2048, xv0, un * UB + kt * 128, 0); DMA(rx, lds + kt * 2048 + 1024, xv1, un * UB + kt * 128, 0); };
    constexpr int AH = (GPERM == 1 || GPERM == 3) ? 2 : 1;
    stage(wid); if (AH == 2) stage(8 + wid);
    unsigned c0;
    asm volatile("global_load_dword %0, %1, off sc1\n\ts_waitcnt vmcnt(0)" : "=&v"(c0) : "v"(cnt) : "memory");
    const unsigned call = __builtin_amdgcn_readfirstlane(c0) + 1u;
    const int R = (wid - 1) * NU + un;
    const int grt = (32 + R) / 9;
    char* pw = p.ws[(rank - wid) & 7];
    char* dp = (NOCOMM ? myws + (32u << 20) : pw) + WS_XST + (size_t)R * UB + lane * 16;
    char* fa = pw + WS_XFLG + (size_t)R * 128;
    unsigned fv = call * 64u - (PDEPTH - 1);
    LAS char* la = lds + lane * 16;
    auto rw = __builtin_amdgcn_make_buffer_rsrc((void*)p.w, 0, K * N * 2, 0x00020000);
    const unsigned tv = (unsigned)((lane >> 2) * N * 2 + (lane & 3) * 128);
    const int par = (int)(((size_t)p.w >> 9) + xcd) & 1;
    unsigned t0_ = 0;
    auto help = [&](int n) {
      const int blk = n >> 3, e = (n & 7) * 2 + (idx & 1);
      int j, ho;
      help_slot(e, par, j, ho);
      const int kt = blk * 8 + CH0 + (ho ^ (blk & 1));
      if (kt >= NT) return;
      const unsigned o = (unsigned)(kt * 64 * N * 2 + (((rank + 1 + j) & 7) * 8 + ((xcd + j) & 7)) * TN * 2);
      asm volatile("buffer_load_dword %0, %1, %2, %3 offen\n\ts_waitcnt vmcnt(%7)\n\tbuffer_load_dword %0, %1, %2, %4 offen\n\ts_waitcnt vmcnt(%7)\n\t"
                   "buffer_load_dword %0, %1, %2, %5 offen\n\ts_waitcnt vmcnt(%7)\n\tbuffer_load_dword %0, %1, %2, %6 offen\n\ts_waitcnt vmcnt(%7)"
                   : "+v"(t0_) : "v"(tv), "s"(rw), "s"(o), "s"(o + 16 * N * 2), "s"(o + 32 * N * 2), "s"(o + 48 * N * 2), "n"(CWIN) : "memory");
    };
    u64 tn = PACE ? __builtin_amdgcn_s_memrealtime() - 3 * PACE : 0;
    for (int r = 0; r < 8; ++r) {
      BAR();
      if (wid == 0) {
        if (r + AH < 8) stage(8 * (r + AH));
        if (CHELP) for (int e = 0; e < CHELP; ++e) help(r * CHELP + e);
        WAITV(0);
        continue;
      }
      u32x4 a0 = *(LAS u32x4*)(la + gperm(r * 8, grt) * 2048), a1 = *(LAS u32x4*)(la + gperm(r * 8, grt) * 2048 + 1024);
#pragma unroll
      for (int j = 0; j < 8; ++j) {
        u32x4 n0 = a0, n1 = a1;
        if (j < 7) { n0 = *(LAS u32x4*)(la + gperm(r * 8 + j + 1, grt) * 2048); n1 = *(LAS u32x4*)(la + gperm(r * 8 + j + 1, grt) * 2048 + 1024); }
        if (PACE) { while ((long long)(__builtin_amdgcn_s_memrealtime() - tn) < 0) __builtin_amdgcn_s_sleep(1); tn += PACE; }
        asm volatile("global_store_dwordx4 %0, %1, off sc0 sc1\n\t"
                     "global_store_dwordx4 %0, %2, off offset:1024 sc0 sc1\n\t"
                     "s_waitcnt vmcnt(%5)\n\t"
                     "s_mov_b64 exec, 1\n\tglobal_atomic_umax %3, %4, off sc1\n\ts_mov_b64 exec, -1\n\ts_nop 0"
                     ::"v"(dp), "v"(a0), "v"(a1), "v"(fa), "v"(fv), "n"(3 * (PDEPTH - 1)) : "memory");
        fv += 1; dp += 2048;
        if (j == 2 && r + AH < 8) stage(8 * (r + AH) + wid);
        a0 = n0; a1 = n1;
      }
    }
    asm volatile("s_waitcnt vmcnt(0)" : "+v"(t0_)::"memory");
    if (lane == 0 && wid) __hip_atomic_fetch_max((unsigned*)fa, call * 64u + 63u, __ATOMIC_RELAXED, __HIP_MEMORY_SCOPE_SYSTEM);
    if (wid == 1) STAMP(7);
    if (un == 0 && wid == 0) {
      const char* gp = myws + WS_GRAN + lane * 16;
      const bool skip = lane >= 56;
      unsigned gm = 0xffffffffu;
      const u64 t0 = __builtin_amdgcn_s_memrealtime();
      for (unsigned np = 1;; ++np) {
        u32x4 a, c;
        asm volatile("global_load_dwordx4 %0, %2, off sc1\n\tglobal_load_dwordx4 %1, %2, off offset:1024 sc1\n\ts_waitcnt vmcnt(0)"
                     : "=&v"(a), "=&v"(c) : "v"(gp) : "memory");
        if (__all(a[1] == call && a[3] == call && (skip || (c[1] == call && c[3] == call)))) {
          gm = wave_umax(skip ? max(a[0], a[2]) : max(max(a[0], a[2]), max(c[0], c[2])));
          break;
        }
        if ((np & 63u) == 0 && __any(timed_out(tmo, call, t0))) break;
      }
      __hip_atomic_store((u64*)(p.ws[lane & 7] + WS_GMAX + ((call & 1) * 8 + (lane >> 3)) * 128 + (SOLO ? lane & 7 : rank) * 8), ((u64)call << 32) | gm, __ATOMIC_RELAXED, __HIP_MEMORY_SCOPE_SYSTEM);
      STAMP(8);
    }
    const int fi = un * 8 + wid;
    if (!SOLO && lane == 0 && (fi >> 4) != rank) {
      unsigned* fl = (unsigned*)(myws + WS_DONE + (size_t)fi * 128);
      const u64 t1 = __builtin_amdgcn_s_memrealtime();
      unsigned np = 0;
      while (__hip_atomic_load(fl, __ATOMIC_RELAXED, __HIP_MEMORY_SCOPE_SYSTEM) < call) {
        if ((++np & 63u) == 0 && timed_out(tmo, call, t1)) { p.out[rank][(size_t)rank * M * NS + fi] = (char)-128; break; }
        __builtin_amdgcn_s_sleep(DSLEEP);
      }
    }
    if (tid == 0) __hip_atomic_store(cnt, call, __ATOMIC_RELAXED, __HIP_MEMORY_SCOPE_AGENT);
    if (wid == 0) STAMP(6);
    return;
  }

  const bool isG = XSPEC ? (xcd & 1) : idx < 16;
  const int il = XSPEC ? idx % 14 : idx - 16;
  int rt, cl;
  if (!isG) { rt = il & 1; cl = XSPEC ? x4 * 2 + idx / 14 : (xcd + CMIX * (il >> 1)) & 7; }
  else if (XSPEC) { rt = (x4 & 1) * 8 + (idx & 7); cl = (x4 >> 1) * 4 + (idx >> 3); }
  else if (G44) { rt = (xcd & 3) * 4 + (idx & 3); cl = (xcd >> 2) * 4 + (idx >> 2); }
  else { rt = (xcd & 1) * 8 + (idx & 7); cl = (xcd >> 1) * 2 + (idx >> 3); }
  const int dst = isG ? rank : (rank + 1 + (il >> 1)) & 7;
  const int gslot = isG ? rt * 8 + cl : 128 + cl * 14 + il;
  const int ct = dst * 8 + cl;
  if (wid < 4) {
    __builtin_amdgcn_s_setprio(MPRIO);
    if (isG) mfma_role<9>(lds, wid, lane, b, myws, p.out[rank] + cl * TN, rank, rt * 9);
    else mfma_role<8>(lds, wid, lane, b, myws, p.out[dst] + (size_t)(rank * M + GR + rt * 128) * NS + cl * TN, rank, 0);
    return;
  }
  const int w = wid - 4;
  bool bad = false;
  unsigned call = 0;
  auto ktile = [&](int u) {
    const int uu = (u + ((il >> 1) & 1)) & 63;
    int kl = LPERM ? ((uu & 1) ? ((uu - LDIST) & 63) + 1 - rt : uu + rt) : u;
    if (LPERM == 2) { const int u2 = (u + (il & 2)) & 63; kl = (u2 & ~3) | ((u2 + 2 * rt) & 3); }
    return isG ? gperm(u, rt) : kl;
  };
  __builtin_amdgcn_s_setprio(LPRIO);
  if (w == 0 && isG) {
    const int u0 = rt * 9;
    auto rg = __builtin_amdgcn_make_buffer_rsrc((void*)(myws + WS_XST), 0, (NR - 1) * NU * UB, 0x00020000);
    const unsigned gv = lane * 16, gv1 = gv + 1024;
    const int lu = u0 + min(lane, 8);
    const unsigned* fp = (const unsigned*)(myws + WS_XFLG + (size_t)max(lu - 32, 0) * 128);
    unsigned rdy = u0 + 8 < 32 ? NT : 0, np = 0;
    if (rdy == 0) call = __builtin_amdgcn_readfirstlane(__hip_atomic_load(cnt, __ATOMIC_RELAXED, __HIP_MEMORY_SCOPE_AGENT)) + 1u;
    const u64 t0 = __builtin_amdgcn_s_memrealtime();
    auto seen = [&](unsigned f) {
      int n = lu < 32 ? NT : (int)(f - call * 64u) + 1;
      int m = __builtin_amdgcn_readlane(n, 0);
#pragma unroll
      for (int i = 1; i < 9; ++i) m = min(m, __builtin_amdgcn_readlane(n, i));
      rdy = max((int)rdy, m);
    };
    int so = 0, pend = 0;
    for (int t = 0; t < NT + 2; ++t) {
      if (t < NT) {
        while (rdy <= (unsigned)t) {
          seen(__hip_atomic_load(fp, __ATOMIC_RELAXED, __HIP_MEMORY_SCOPE_SYSTEM));
          if (rdy <= (unsigned)t && (++np & 63u) == 0 && __any(timed_out(tmo, call, t0))) { bad = true; rdy = NT; }
        }
        const bool ahead = rdy < NT && rdy < (unsigned)t + 5 && pend == 0;
        if (ahead) { __builtin_amdgcn_global_load_lds(fp, (LAS unsigned*)(lds + MISC + 256), 4, 0, 17); pend = rdy > (unsigned)t + 1 ? 2 : 1; }
        const int kt = gperm(t, rt);
        const unsigned ox = u0 * UB + kt * 128, og = (u0 - 32) * UB + (CUTG ? (t & 1) : t) * 2048;
#define XUNITS(NOWN) _Pragma("unroll") for (int i = 0; i < 9; ++i) { \
          if (i < NOWN) { DMA(rx, lds + so + i * 2048, xv0, ox + i * UB, 0); DMA(rx, lds + so + i * 2048 + 1024, xv1, ox + i * UB, 0); } \
          else { DMA(rg, lds + so + i * 2048, gv, og + i * UB, XAUX); DMA(rg, lds + so + i * 2048 + 1024, gv1, og + i * UB, XAUX); } }
        if (rt > 3) { XUNITS(0) } else if (rt == 3) { XUNITS(5) } else { XUNITS(9) }
        so = so == 2 * SLOT ? 0 : so + SLOT;
        if (pend == 2) WAITV(19); else WAITV(18);
        if (pend && --pend == 0) seen(*(volatile LAS unsigned*)(lds + MISC + 256 + lane * 4));
      } else WAITV(0);
      TSTAMP();
      if (t >= 1) BAR();
    }
    TSAVE();
  } else if (w == 0) {
    int so = 0;
    for (int t = 0; t < NT + 2; ++t) {
      if (t < NT) {
        const unsigned go = (unsigned)(NU + rt * 8) * UB + ktile(t) * 128;
#pragma unroll
        for (int i = 0; i < 8; ++i) { DMA(rx, lds + so + i * 2048, xv0, go + i * UB, 0); DMA(rx, lds + so + i * 2048 + 1024, xv1, go + i * UB, 0); }
        so = so == 2 * SLOT ? 0 : so + SLOT;
        WAITV(16);
      } else WAITV(0);
      TSTAMP();
      if (t >= 1) BAR();
    }
    TSAVE();
  } else if (w < 3) {
    auto rs = __builtin_amdgcn_make_buffer_rsrc((void*)p.w, 0, K * N * 2, 0x00020000);
    const unsigned v0 = (unsigned)(((lane >> 5) + (w - 1) * 32) * N * 2 + ct * TN * 2 + (((lane & 31) ^ ((lane >> 5) << 1)) << 4));
    const unsigned v1 = (v0 ^ 64) + 2 * N * 2, v2 = (v0 ^ 128) + 8 * N * 2, v3 = (v0 ^ 192) + 10 * N * 2;
    int so = XB + (w - 1) * 16384;
    for (int t = 0; t < NT + 2; ++t) {
      if (t < NT) {
        const int kt = (CUTG && isG) ? (t & 1) : ktile(t);
#define WPIECES(AUX) _Pragma("unroll") for (int j = 0; j < 16; ++j)   \
          DMA(rs, lds + so + j * 1024, (j & 4) ? ((j & 1) ? v3 : v2) : ((j & 1) ? v1 : v0), (kt * 64 + ((j & 2) << 1) + ((j & 8) << 1)) * N * 2, AUX);
        const bool hit = ((LPERM == 2 ? kt >> 1 : kt) & 1) != rt;
        if (!isG && ((LNT == 1 && hit) || LNT == 2 || (LNT == 3 && !hit) || (LNT == 4 && (kt & 2)) || (LNT == 5 && (kt & 3))) || (isG && GNT)) { WPIECES(NTAUX) } else { WPIECES(0) }
        so = so >= 2 * SLOT ? so - 2 * SLOT : so + SLOT;
        WAITV(16);
      } else WAITV(0);
      TSTAMP();
      if (t >= 1) BAR();
    }
    TSAVE();
  } else {
    auto rs = __builtin_amdgcn_make_buffer_rsrc((void*)p.w, 0, K * N * 2, 0x00020000);
    unsigned t0_ = 0, t1_ = 0, t2_ = 0, t3_ = 0, t4_ = 0;
    const unsigned tv = (unsigned)((lane >> 2) * N * 2 + (lane & 3) * 128);
    auto touch = [&](int kt, int c) {
      const unsigned o = (unsigned)(kt * 64 * N * 2 + c * TN * 2);
      asm volatile("buffer_load_dword %0, %4, %5, %6 offen\n\tbuffer_load_dword %1, %4, %5, %7 offen\n\t"
                   "buffer_load_dword %2, %4, %5, %8 offen\n\tbuffer_load_dword %3, %4, %5, %9 offen"
                   : "+v"(t0_), "+v"(t1_), "+v"(t2_), "+v"(t3_) : "v"(tv), "s"(rs), "s"(o), "s"(o + 16 * N * 2), "s"(o + 32 * N * 2), "s"(o + 48 * N * 2) : "memory");
    };
    int hc = 0, ho = 0, hp = -1;
    if (isG) {
      if (GPERM == 1 && (rt & 7) >= 2) touch(rt & 7, ct);
      if (GPERM == 3 && (rt & 7) >= 2 && (rt & 7) < 4) touch(rt & 3, ct);
      if (rt >= 3) {
        const unsigned pv = (unsigned)(((rt - 3) * 8 + cl) * 5 * K * 2 + lane * 128);
        asm volatile("buffer_load_dword %0, %5, %6, 0 offen\n\tbuffer_load_dword %1, %5, %6, %7 offen\n\t"
                     "buffer_load_dword %2, %5, %6, %8 offen\n\tbuffer_load_dword %3, %5, %6, %9 offen\n\t"
                     "buffer_load_dword %4, %5, %6, %10 offen"
                     : "+v"(t0_), "+v"(t1_), "+v"(t2_), "+v"(t3_), "+v"(t4_) : "v"(pv), "s"(rx), "s"(K * 2), "s"(K * 4), "s"(K * 6), "s"(K * 8) : "memory");
      }
      if (HELP) {
        const int par = (int)(((size_t)p.w >> 9) + xcd) & 1;
        int j;
        help_slot(idx, par, j, ho);
        hc = ((rank + 1 + j) & 7) * 8 + ((xcd + j) & 7);
        hp = (rt + 4) & 7;
      }
    }
    for (int t = 0; t <= NT; ++t) {
      BAR();
      if (HELP && (t & 7) == hp) {
        const int kt = (t & ~7) + HELP + (ho ^ ((t >> 3) & 1));
        if (kt < NT) touch(kt, hc);
      }
    }
    asm volatile("s_waitcnt vmcnt(0)" : "+v"(t0_), "+v"(t1_), "+v"(t2_), "+v"(t3_), "+v"(t4_)::"memory");
  }
  if (w == 0) call = __builtin_amdgcn_readfirstlane(__hip_atomic_load(cnt, __ATOMIC_RELAXED, __HIP_MEMORY_SCOPE_AGENT)) + 1u;
  LBAR();
  if (w == 0) {
    const u32x4 mv = *(volatile LAS u32x4*)(misc + L_MAX);
    const unsigned m = bad ? 0xffffffffu : max(max(mv[0], mv[1]), max(mv[2], mv[3]));
    if (lane == 0) __hip_atomic_store((u64*)(myws + WS_GRAN + gslot * 8), ((u64)call << 32) | m, __ATOMIC_RELAXED, __HIP_MEMORY_SCOPE_AGENT);
    const char* gp = myws + WS_GMAX + ((call & 1) * 8 + xcd) * 128 + (lane & 7) * 8;
    unsigned gm = 0xffffffffu;
    const u64 t0 = __builtin_amdgcn_s_memrealtime();
    for (;;) {
      u64 gq; unsigned tm;
      asm volatile("global_load_dwordx2 %0, %2, off sc0 sc1\n\tglobal_load_dword %1, %3, off sc1\n\ts_waitcnt vmcnt(0)"
                   : "=&v"(gq), "=&v"(tm) : "v"(gp), "v"(tmo) : "memory");
      if (__any(tm == call)) break;
      if (__all((unsigned)(gq >> 32) == call)) {
        gm = (unsigned)gq;
        gm = max(gm, (unsigned)__shfl_xor(gm, 1)); gm = max(gm, (unsigned)__shfl_xor(gm, 2)); gm = max(gm, (unsigned)__shfl_xor(gm, 4));
        break;
      }
      if (__builtin_amdgcn_s_memrealtime() - t0 > DEADLINE) { __hip_atomic_store(tmo, call, __ATOMIC_RELAXED, __HIP_MEMORY_SCOPE_AGENT); break; }
      __builtin_amdgcn_s_sleep(ASLEEP);
    }
    if (lane == 0) misc[L_GMAX] = gm;
  }
  LBAR();
  LBAR();
  if (w == 0 && lane == 0) {
    if (!isG)
      __hip_atomic_store((unsigned*)(p.ws[dst] + WS_DONE + (size_t)(rank * 16 + cl * 2 + rt) * 128), call, __ATOMIC_RELAXED, __HIP_MEMORY_SCOPE_SYSTEM);
    __hip_atomic_store(cnt, call, __ATOMIC_RELAXED, __HIP_MEMORY_SCOPE_AGENT);
    STAMP(6);
  }
}

extern "C" void kernel_launch(int rank, int nranks, ncclComm_t comm, hipStream_t stream,
                              void* const* d_in, const int* in_sizes, int n_in,
                              void* d_out, int out_size, const int* in_dtypes, int out_dtype,
                              void* const* d_out_all, void* const* d_ws_all, size_t ws_bytes) {
  if (nranks != NR || (unsigned)rank >= (unsigned)NR || n_in < 2 || ws_bytes < WS_END || in_sizes[0] != M * K || in_sizes[1] != K * N || out_size != NR * M * NS) {
    fprintf(stderr, "unexpected problem shape\n");
    return;
  }
  static bool init = false;
  if (!init) {
    HIP_CHECK(hipFuncSetAttribute((const void*)gemm_a2a, hipFuncAttributeMaxDynamicSharedMemorySize, LDS_BYTES));
    init = true;
  }
  Params p;
  p.x = d_in[0]; p.w = d_in[1];
  for (int i = 0; i < NR; ++i) { p.out[i] = (char*)d_out_all[i]; p.ws[i] = (char*)d_ws_all[i]; }
  p.rank = rank; p.pad = 0;
  gemm_a2a<<<NWG, 512, LDS_BYTES, stream>>>(p);
  HIP_CHECK(hipGetLastError());
}
```

```cpp
#include <hip/hip_runtime.h>
#include <rccl/rccl.h>
#include <cstdio>
#include <utility>

#define HIP_CHECK(expr) do { hipError_t _e = (expr); if (_e != hipSuccess) { \
    fprintf(stderr, "HIP error %s at %s:%d\n", hipGetErrorString(_e), __FILE__, __LINE__); \
    return; } } while (0)

typedef __bf16 bf16x4 __attribute__((ext_vector_type(4)));
typedef __bf16 bf16x8 __attribute__((ext_vector_type(8)));
typedef float f32x4 __attribute__((ext_vector_type(4)));
typedef unsigned u32x4 __attribute__((ext_vector_type(4)));
typedef unsigned long long u64;
#define LAS __attribute__((address_space(3)))
#define DI __device__ __forceinline__

#define GPERM 1
#define PDEPTH 2
#define ASLEEP 2
#define DSLEEP 4
#define SOLO 0
#define XSPEC 3
#define LAUX 2
#define LPACE 116
#define GPACE 0
#define PACE 72
#define NOMMA 0
#define NAGG 1
#define RWARM 0
#define SERP 1
#define XL2 4
#define LT1 10
#define LT2 30
#define LG1 0
#define LG2 0
#define LGT1 0
#define LGT2 0
#define WFIRST 0

constexpr int NR = 8;
constexpr int M = 512, N = 16384, K = 4096, NT = K / 64;
constexpr int NS = N / NR;
constexpr int NWG = 256;
constexpr int NU = 16, GR = NU * 16;
constexpr int UB = NT * 2048;
constexpr int LDS_BYTES = 163840;
template <bool G> struct Ring;
template <> struct Ring<true> { static constexpr int NI = 9, WROW = 512, XS = 144 * 128, WS = 64 * 512, NW = 3, W0 = 3 * XS; };
template <> struct Ring<false> { static constexpr int NI = 8, WROW = 256, XS = 256 * 128, WS = 64 * 256, NW = 4, W0 = 3 * XS; };
constexpr int GPOLL = Ring<true>::W0 + 3 * Ring<true>::WS;
static_assert(GPOLL + 512 <= LDS_BYTES && Ring<false>::W0 + 4 * Ring<false>::WS <= LDS_BYTES && UB <= LDS_BYTES, "LDS");
constexpr int L_MAX = 0, L_GMAX = 4;
constexpr size_t WS_CNT = 0;
constexpr size_t WS_TMO = WS_CNT + NWG * 128;
constexpr size_t WS_GRAN = WS_TMO + 1024;
constexpr size_t WS_GMAX = WS_GRAN + NWG * 8;
constexpr size_t WS_DONE = WS_GMAX + 2 * 8 * 128;
constexpr size_t WS_XFLG = WS_DONE + NR * 16 * 128;
constexpr size_t WS_XST = 2u << 20;
constexpr size_t WS_END = WS_XST + (size_t)(NR - 1) * NU * UB;
static_assert(WS_XFLG + (NR - 1) * NU * 128 <= WS_XST, "workspace map");
#define STAMP(i) do {} while (0)
#define TBAR(t) BAR()
#define TSAVE() do {} while (0)
constexpr u64 DEADLINE = 30000000ull;

struct Params {
  const void* x; const void* w;
  char* out[NR]; char* ws[NR];
  int rank; int pad;
};

#define WAITV(n) asm volatile("s_waitcnt vmcnt(%0)" ::"n"(n) : "memory")
#define BAR() asm volatile("s_barrier" ::: "memory")
#define LBAR() asm volatile("s_waitcnt lgkmcnt(0)\n\ts_barrier" ::: "memory")
#define SB0() __builtin_amdgcn_sched_barrier(0)
#define DMA(rs, dst, voff, soff, aux) __builtin_amdgcn_raw_ptr_buffer_load_lds(rs, (LAS void*)(dst), 16, voff, soff, 0, aux)

template <int NI> struct Frag { bf16x8 x[NI]; bf16x4 wl[4], wh[4]; };
struct Addr { LAS char* x0; LAS char* x1; LAS char* w[4]; int sx, sw, dx, dw; };

template <bool G, int KS, int R> DI void rd(Frag<Ring<G>::NI>& f, const Addr& a) {
  if constexpr (R < 8) {
    const bf16x4 v = __builtin_amdgcn_ds_read_tr16_b64_v4bf16((LAS bf16x4*)(a.w[R >> 1] + Ring<G>::WROW * (32 * KS + 4 * (R & 1))));
    if constexpr (R & 1) f.wh[R >> 1] = v; else f.wl[R >> 1] = v;
  } else if constexpr (R < 8 + Ring<G>::NI) f.x[R - 8] = *(LAS bf16x8*)((KS ? a.x1 : a.x0) + 2048 * (R - 8));
}
template <bool G, int Q, int KN> DI void gap(f32x4 (&acc)[Ring<G>::NI][4], const Frag<Ring<G>::NI>& fc, Frag<Ring<G>::NI>& fn, Addr& a) {
  using RG = Ring<G>;
  constexpr int I = Q >> 2, J = (SERP && (I & 1)) ? 3 - (Q & 3) : Q & 3;
  acc[I][J] = __builtin_amdgcn_mfma_f32_16x16x32_bf16(__builtin_shufflevector(fc.wl[J], fc.wh[J], 0, 1, 2, 3, 4, 5, 6, 7), fc.x[I], acc[I][J], 0, 0, 0);
  SB0();
  if constexpr (Q % 3 == 0) rd<G, KN, 2 * (Q / 3)>(fn, a);
  if constexpr (Q % 3 == 2) rd<G, KN, 2 * (Q / 3) + 1>(fn, a);
  if constexpr (KN == 1) {
    if constexpr (Q == 13) { a.w[0] += a.dw; asm volatile("" : "+v"(a.w[0])); }
    if constexpr (Q == 16) { a.w[1] += a.dw; asm volatile("" : "+v"(a.w[1])); }
    if constexpr (Q == 19) { a.w[2] += a.dw; asm volatile("" : "+v"(a.w[2])); }
    if constexpr (Q == 22) { a.w[3] += a.dw; asm volatile("" : "+v"(a.w[3])); }
    if constexpr (Q == 25) { a.x0 += a.dx; asm volatile("" : "+v"(a.x0)); }
    if constexpr (Q == 26) { a.x1 += a.dx; asm volatile("" : "+v"(a.x1)); }
  } else {
    if constexpr (Q == 25) { a.sx = a.sx == 2 ? 0 : a.sx + 1; asm volatile("" : "+s"(a.sx)); }
    if constexpr (Q == 26) { a.dx = a.sx == 2 ? -2 * RG::XS : RG::XS; asm volatile("" : "+s"(a.dx)); }
    if constexpr (Q == 27) { a.sw = a.sw == RG::NW - 1 ? 0 : a.sw + 1; asm volatile("" : "+s"(a.sw)); }
    if constexpr (Q == 28) { a.dw = a.sw == RG::NW - 1 ? -(RG::NW - 1) * RG::WS : RG::WS; asm volatile("" : "+s"(a.dw)); }
  }
  if constexpr (Q == 4 * RG::NI - 2) __builtin_amdgcn_s_waitcnt(0xC07F);
  SB0();
}
template <bool G, int KN, int... Qs> DI void kstep(f32x4 (&acc)[Ring<G>::NI][4], const Frag<Ring<G>::NI>& fc, Frag<Ring<G>::NI>& fn, Addr& a, std::integer_sequence<int, Qs...>) {
  (gap<G, Qs, KN>(acc, fc, fn, a), ...);
}
template <bool G, int... Rs> DI void rd_all(Frag<Ring<G>::NI>& f, const Addr& a, std::integer_sequence<int, Rs...>) { (rd<G, 0, Rs>(f, a), ...); }

DI unsigned wave_umax(unsigned v) {
  v = max(v, (unsigned)__builtin_amdgcn_update_dpp(0, (int)v, 0xB1, 0xf, 0xf, false));
  v = max(v, (unsigned)__builtin_amdgcn_update_dpp(0, (int)v, 0x4E, 0xf, 0xf, false));
  v = max(v, (unsigned)__builtin_amdgcn_update_dpp(0, (int)v, 0x141, 0xf, 0xf, false));
  v = max(v, (unsigned)__builtin_amdgcn_update_dpp(0, (int)v, 0x140, 0xf, 0xf, false));
  return max(max((unsigned)__builtin_amdgcn_readlane((int)v, 0), (unsigned)__builtin_amdgcn_readlane((int)v, 16)),
             max((unsigned)__builtin_amdgcn_readlane((int)v, 32), (unsigned)__builtin_amdgcn_readlane((int)v, 48)));
}
DI bool timed_out(unsigned* tmo, unsigned call, u64 t0) {
  if (__hip_atomic_load(tmo, __ATOMIC_RELAXED, __HIP_MEMORY_SCOPE_AGENT) == call || __builtin_amdgcn_s_memrealtime() - t0 > DEADLINE) {
    __hip_atomic_store(tmo, call, __ATOMIC_RELAXED, __HIP_MEMORY_SCOPE_AGENT);
    return true;
  }
  return false;
}
DI int gperm(int u, int rt) { return GPERM && rt >= (XSPEC == 2 ? 2 : 0) ? (u & ~7) | ((u + rt) & 7) : u; }
DI int unit_row(int U, int rank) { return U < 32 ? rank * M + (31 - U) * 16 : ((rank + (U >> 4) - 1) & 7) * M + (U & 15) * 16; }

template <bool G> DI void mfma_role(LAS char* lds, int wid, int lane, int b, char* myws, char* obase, int rank, int u0) {
  using RG = Ring<G>;
  constexpr int NI = RG::NI;
  volatile LAS unsigned* misc = (volatile LAS unsigned*)lds;
  const int wm = G ? 0 : wid >> 1, wn = G ? wid : wid & 1;
  const int fr = lane & 15, g = lane >> 4, q = (lane & 15) >> 2, pp = lane & 3;
  const int xo0 = (wm * 128 + fr) * 128 + ((g ^ (fr >> 1)) << 4);
  const int fk = q | ((g & 1) << 2);
  const int wo0 = RG::W0 + RG::WROW * (8 * g + q) + ((wn >> 1) * 256) + ((((wn & 1) * 4) ^ fk) << 5) + 8 * pp;
  f32x4 acc[NI][4];
#pragma unroll
  for (int i = 0; i < NI; ++i)
#pragma unroll
    for (int j = 0; j < 4; ++j) acc[i][j] = (f32x4){0.f, 0.f, 0.f, 0.f};
  Frag<NI> f0, f1;
  Addr a = {lds + xo0, lds + (xo0 ^ 64), {lds + wo0, lds + (wo0 ^ 32), lds + (wo0 ^ 64), lds + (wo0 ^ 96)}, 0, 0, RG::XS, RG::WS};
  SB0();
  BAR();
  rd_all<G>(f0, a, std::make_integer_sequence<int, 8 + NI>{});
  __builtin_amdgcn_s_waitcnt(0xC07F);
  if (NOMMA) for (int s = 0; s < NT; ++s) BAR();
  else for (int s = 0; s < NT; ++s) {
    SB0();
    kstep<G, 1>(acc, f0, f1, a, std::make_integer_sequence<int, 4 * NI>{});
    BAR();
    SB0();
    kstep<G, 0>(acc, f1, f0, a, std::make_integer_sequence<int, 4 * NI>{});
  }
  SB0();
  float m = 0.f, m2 = 0.f;
#pragma unroll
  for (int i = 0; i < NI; ++i)
#pragma unroll
    for (int j = 0; j < 4; ++j) {
      m = fmaxf(fmaxf(m, __builtin_fabsf(acc[i][j][0])), __builtin_fabsf(acc[i][j][1]));
      m2 = fmaxf(fmaxf(m2, __builtin_fabsf(acc[i][j][2])), __builtin_fabsf(acc[i][j][3]));
    }
  const unsigned mu = wave_umax(__float_as_uint(fmaxf(m, m2)));
  if (lane == 0) misc[L_MAX + wid] = mu;
  LBAR();
  LBAR();
  if (wid == 0) STAMP(3);
  const unsigned gm = misc[L_GMAX];
  const bool bad = gm == 0xffffffffu;
  const float amax = __uint_as_float(gm);
  const float inv = amax > 0.f ? 127.f / amax : 0.f;
  char* orow = obase + (size_t)(wm * 128 + fr) * NS + wn * 64 + 16 * g;
#pragma unroll
  for (int i = 0; i < NI; ++i) {
    unsigned d4[4];
#pragma unroll
    for (int j = 0; j < 4; ++j) {
      unsigned a0 = __float_as_uint(__builtin_fmaf(acc[i][j][0], inv, 12582912.f)), a1 = __float_as_uint(__builtin_fmaf(acc[i][j][1], inv, 12582912.f));
      unsigned a2 = __float_as_uint(__builtin_fmaf(acc[i][j][2], inv, 12582912.f)), a3 = __float_as_uint(__builtin_fmaf(acc[i][j][3], inv, 12582912.f));
      d4[j] = __builtin_amdgcn_perm(a1, a0, 0x0c0c0400) | __builtin_amdgcn_perm(a3, a2, 0x04000c0c);
    }
    auto r0 = __builtin_amdgcn_permlane32_swap(d4[0], d4[2], false, false); d4[0] = r0[0]; d4[2] = r0[1];
    auto r1 = __builtin_amdgcn_permlane32_swap(d4[1], d4[3], false, false); d4[1] = r1[0]; d4[3] = r1[1];
    auto r2 = __builtin_amdgcn_permlane16_swap(d4[0], d4[1], false, false); d4[0] = r2[0]; d4[1] = r2[1];
    auto r3 = __builtin_amdgcn_permlane16_swap(d4[2], d4[3], false, false); d4[2] = r3[0]; d4[3] = r3[1];
    u32x4 v = {d4[0], d4[1], d4[2], d4[3]};
    if (bad) v = (u32x4){0x80808080u, 0x80808080u, 0x80808080u, 0x80808080u};
    const int r16 = G ? unit_row(u0 + i, rank) : i * 16;
    asm volatile("global_store_dwordx4 %0, %1, off sc0 sc1\n\ts_nop 1" ::"v"(orow + (size_t)r16 * NS), "v"(v) : "memory");
  }
  if (wid == 0) STAMP(4);
  WAITV(0);
  LBAR();
  if (wid == 0) STAMP(5);
}

__global__ __launch_bounds__(512, 2) void gemm_a2a(const Params p) {
  extern __shared__ __attribute__((aligned(16))) char smem[];
  LAS char* lds = (LAS char*)smem;
  volatile LAS unsigned* misc = (volatile LAS unsigned*)lds;
  const int tid = threadIdx.x, lane = tid & 63;
  const int wid = __builtin_amdgcn_readfirstlane(tid >> 6);
  const int b = blockIdx.x, xcd = b & 7, idx = b >> 3;
  const int rank = p.rank & 7;
  char* myws = p.ws[rank];
  unsigned* cnt = (unsigned*)(myws + WS_CNT + b * 128);
  unsigned* tmo = (unsigned*)(myws + WS_TMO);
  if (wid == 0) STAMP(0);
  auto rx = __builtin_amdgcn_make_buffer_rsrc((void*)p.x, 0, M * K * 2, 0x00020000);
  const unsigned xv0 = (unsigned)((lane >> 3) * K * 2 + (((lane & 7) ^ (lane >> 4)) << 4)), xv1 = (xv0 ^ 64) + 8 * K * 2;

  const int x4 = xcd >> 1;
  const bool odd = xcd & 1;
  if (XSPEC == 3 ? idx >= 30 : (idx >= 28 && odd == (XSPEC == 2))) {
    const int un = XSPEC == 3 ? xcd * 2 + (idx & 1) : x4 * 4 + (idx & 3);
    auto stage = [&](int kt) { DMA(rx, lds + kt * 2048, xv0, un * UB + kt * 128, 0); DMA(rx, lds + kt * 2048 + 1024, xv1, un * UB + kt * 128, 0); };
    stage(wid);
    const int R = max(wid - 1, 0) * NU + un;
    const int grt = (32 + R) / 9;
    char* pw = p.ws[(rank - wid) & 7];
    char* dp = pw + WS_XST + (size_t)R * UB + lane * 16;
    char* fa = pw + WS_XFLG + (size_t)R * 128;
    unsigned c0, c1 = 0;
    if (RWARM) asm volatile("global_load_dword %0, %2, off sc1\n\tglobal_load_dword %1, %3, off sc0 sc1\n\ts_waitcnt vmcnt(1)"
                            : "=&v"(c0), "+v"(c1) : "v"(cnt), "v"(lane & 1 ? dp - lane * 16 : fa) : "memory");
    else asm volatile("global_load_dword %0, %1, off sc1\n\ts_waitcnt vmcnt(0)" : "=&v"(c0) : "v"(cnt) : "memory");
    const unsigned call = __builtin_amdgcn_readfirstlane(c0) + 1u;
    unsigned fv = call * 64u - (PDEPTH - 1);
    LAS char* la = lds + lane * 16;
    u64 tn = PACE ? __builtin_amdgcn_s_memrealtime() - 3 * PACE : 0;
    for (int r = 0; r < 8; ++r) {
      BAR();
      if (wid == 0) {
        if (r < 7) { stage(8 * (r + 1)); WAITV(0); }
        continue;
      }
      u32x4 a0 = *(LAS u32x4*)(la + gperm(r * 8, grt) * 2048), a1 = *(LAS u32x4*)(la + gperm(r * 8, grt) * 2048 + 1024);
#pragma unroll
      for (int j = 0; j < 8; ++j) {
        u32x4 n0 = a0, n1 = a1;
        if (j < 7) { n0 = *(LAS u32x4*)(la + gperm(r * 8 + j + 1, grt) * 2048); n1 = *(LAS u32x4*)(la + gperm(r * 8 + j + 1, grt) * 2048 + 1024); }
        if (PACE) { while ((long long)(__builtin_amdgcn_s_memrealtime() - tn) < 0) __builtin_amdgcn_s_sleep(1); tn += PACE; }
        asm volatile("global_store_dwordx4 %0, %1, off sc0 sc1\n\t"
                     "global_store_dwordx4 %0, %2, off offset:1024 sc0 sc1\n\t"
                     "s_waitcnt vmcnt(%5)\n\t"
                     "s_mov_b64 exec, 1\n\tglobal_store_dword %3, %4, off offset:%6 sc0 sc1\n\ts_mov_b64 exec, -1\n\ts_nop 0"
                     ::"v"(dp), "v"(a0), "v"(a1), "v"(fa), "v"(fv), "n"(3 * (PDEPTH - 1)), "n"((j & 3) * 4) : "memory");
        fv += 1; dp += 2048;
        if (j == 2 && r < 7) stage(8 * (r + 1) + wid);
        a0 = n0; a1 = n1;
      }
    }
    asm volatile("s_waitcnt vmcnt(0)" : "+v"(c1)::"memory");
    if (lane == 0 && wid) __hip_atomic_store((unsigned*)fa, call * 64u + 63u, __ATOMIC_RELAXED, __HIP_MEMORY_SCOPE_SYSTEM);
    if (wid == 1) STAMP(7);
    if (un < NAGG && wid == 0) {
      const char* gp = myws + WS_GRAN + lane * 16;
      const bool skip = lane >= 56;
      unsigned gm = 0xffffffffu;
      const u64 t0 = __builtin_amdgcn_s_memrealtime();
      for (unsigned np = 1;; ++np) {
        u32x4 a, c;
        asm volatile("global_load_dwordx4 %0, %2, off sc1\n\tglobal_load_dwordx4 %1, %2, off offset:1024 sc1\n\ts_waitcnt vmcnt(0)"
                     : "=&v"(a), "=&v"(c) : "v"(gp) : "memory");
        if (__all(a[1] == call && a[3] == call && (skip || (c[1] == call && c[3] == call)))) {
          gm = wave_umax(skip ? max(a[0], a[2]) : max(max(a[0], a[2]), max(c[0], c[2])));
          break;
        }
        if ((np & 63u) == 0 && __any(timed_out(tmo, call, t0))) break;
      }
      __hip_atomic_store((u64*)(p.ws[lane & 7] + WS_GMAX + ((call & 1) * 8 + (lane >> 3)) * 128 + (SOLO ? lane & 7 : rank) * 8), ((u64)call << 32) | gm, __ATOMIC_RELAXED, __HIP_MEMORY_SCOPE_SYSTEM);
      STAMP(8);
    }
    const int fi = un * 8 + wid;
    if (!SOLO && lane == 0 && (fi >> 4) != rank) {
      unsigned* fl = (unsigned*)(myws + WS_DONE + (size_t)fi * 128);
      const u64 t1 = __builtin_amdgcn_s_memrealtime();
      unsigned np = 0;
      while (__hip_atomic_load(fl, __ATOMIC_RELAXED, __HIP_MEMORY_SCOPE_SYSTEM) < call) {
        if ((++np & 63u) == 0 && timed_out(tmo, call, t1)) { p.out[rank][(size_t)rank * M * NS + fi] = (char)-128; break; }
        __builtin_amdgcn_s_sleep(DSLEEP);
      }
    }
    if (tid == 0) __hip_atomic_store(cnt, call, __ATOMIC_RELAXED, __HIP_MEMORY_SCOPE_AGENT);
    if (wid == 0) STAMP(6);
    return;
  }

  const bool isG = odd || idx >= 28;
  int rt = 0, cl, dst = rank;
  if (!isG) { cl = x4 * 4 + (idx & 3); dst = (rank + 1 + (idx >> 2)) & 7; }
  else if (XSPEC == 1) { rt = (x4 & 1) * 8 + (idx & 7); cl = (x4 >> 1) * 4 + (idx >> 3); }
  else if (XSPEC == 3) {
    if (!odd) { rt = 0; cl = x4 * 2 + (idx & 1); }
    else if (idx < 28) { rt = 1 + (x4 & 1) * 8 + idx % 7; cl = (x4 >> 1) * 4 + idx / 7; }
    else { rt = 8; cl = x4 * 2 + (idx & 1); }
  }
  else if (odd) { rt = 2 + (x4 & 1) * 7 + idx % 7; cl = (x4 >> 1) * 4 + idx / 7; }
  else { rt = idx & 1; cl = x4 * 2 + ((idx >> 1) & 1); }
  const int gslot = isG ? rt * 8 + cl : 128 + (idx >> 2) * 16 + cl;
  if (wid < 4) {
    __builtin_amdgcn_s_setprio(2);
    if (isG) mfma_role<true>(lds, wid, lane, b, myws, p.out[rank] + cl * 256, rank, rt * 9);
    else mfma_role<false>(lds, wid, lane, b, myws, p.out[dst] + (size_t)(rank * M + GR) * NS + cl * 128, rank, 0);
    return;
  }
  const int w = wid - 4;
  unsigned call = 0;
  auto rw = __builtin_amdgcn_make_buffer_rsrc((void*)p.w, 0, K * N * 2, 0x00020000);
  if (!isG) {
    using RG = Ring<false>;
    const unsigned v0 = (unsigned)(((lane >> 4) + w * 16) * N * 2 + (dst * 16 + cl) * 256 + (((lane & 15) ^ ((lane >> 4) << 1)) << 4)), v1 = (v0 ^ 128) + 8 * N * 2;
    auto ldx = [&](int t, int sx) {
#pragma unroll
      for (int i = 0; i < 4; ++i) {
        DMA(rx, lds + sx * RG::XS + (w * 4 + i) * 2048, xv0, (NU + w * 4 + i) * UB + t * 128, 0);
        DMA(rx, lds + sx * RG::XS + (w * 4 + i) * 2048 + 1024, xv1, (NU + w * 4 + i) * UB + t * 128, 0);
      }
    };
    auto ldw = [&](int t) {
#pragma unroll
      for (int j = 0; j < 4; ++j)
        DMA(rw, lds + RG::W0 + (t & 3) * RG::WS + (w * 4 + j) * 1024, (j & 2) ? v1 : v0, (t * 64 + (j & 1) * 4) * N * 2, LAUX);
    };
    const unsigned pv = lane < 20 && idx * 10 + (lane >> 1) < 256 ? (unsigned)((GR + idx * 10 + (lane >> 1)) * K * 2 + (lane & 1) * 128) : 0xfffff000u;
    unsigned t0_ = 0;
    int sx = 0;
    const bool slow = ((((size_t)p.w >> 8) + cl) & 3) == 1;
    const u64 tb = __builtin_amdgcn_s_memrealtime();
    u64 tp = tb + 200 + LT1 * 69;
    for (int t = 0; t < NT; t += 2) {
      if (WFIRST) { ldw(t); ldw(t + 1); ldx(t, sx); } else { ldx(t, sx); ldw(t); ldw(t + 1); }
      sx = sx == 2 ? 0 : sx + 1;
      if (XL2 && w == 3) {
        asm volatile("buffer_load_dword %0, %1, %2, %3 offen" : "+v"(t0_) : "v"(pv), "s"(rx), "s"(((t + XL2) & 63) * 128) : "memory");
        WAITV(17);
      } else WAITV(16);
      if (t) TBAR(t - 1);
      ldx(t + 1, sx);
      if (LG1 && w == 0 && !slow && (t == LGT1 || t == LGT2)) { while ((long long)(__builtin_amdgcn_s_memrealtime() - tb - (t == LGT1 ? LG1 : LG2)) < 0) __builtin_amdgcn_s_sleep(1); }
      if (LPACE && w == 0 && !slow && t >= LT1 && t < LT2) { while ((long long)(__builtin_amdgcn_s_memrealtime() - tp) < 0) __builtin_amdgcn_s_sleep(1); tp += 2 * LPACE; }
      sx = sx == 2 ? 0 : sx + 1;
      if (XL2 && w == 3) WAITV(13); else if (WFIRST) WAITV(8); else WAITV(12);
      TBAR(t);
    }
    asm volatile("s_waitcnt vmcnt(0)" : "+v"(t0_)::"memory");
    TBAR(NT - 1);
    BAR();
  } else if (w == 0 || w == 3) {
    using RG = Ring<true>;
    const int i0 = w ? 5 : 0, u0 = rt * 9 + i0;
    const bool own = u0 < 32;
    if (w == 3 && rt >= 3) {
      unsigned t0_, t1_, t2_, t3_, t4_;
      const unsigned pv = (unsigned)(((rt - 3) * 8 + cl) * 5 * K * 2 + lane * 128);
      asm volatile("buffer_load_dword %0, %5, %6, 0 offen\n\tbuffer_load_dword %1, %5, %6, %7 offen\n\t"
                   "buffer_load_dword %2, %5, %6, %8 offen\n\tbuffer_load_dword %3, %5, %6, %9 offen\n\t"
                   "buffer_load_dword %4, %5, %6, %10 offen\n\ts_waitcnt vmcnt(0)"
                   : "=&v"(t0_), "=&v"(t1_), "=&v"(t2_), "=&v"(t3_), "=&v"(t4_) : "v"(pv), "s"(rx), "s"(K * 2), "s"(K * 4), "s"(K * 6), "s"(K * 8) : "memory");
    }
    auto rg = __builtin_amdgcn_make_buffer_rsrc((void*)(myws + WS_XST), 0, (NR - 1) * NU * UB, 0x00020000);
    const unsigned gv = lane * 16, gv1 = gv + 1024;
    const unsigned* fp = (const unsigned*)(myws + WS_XFLG + (size_t)(own ? 0 : u0 - 32 + min(lane >> 2, w ? 3 : 4)) * 128) + (lane & 3);
    LAS unsigned* pl = (LAS unsigned*)(lds + GPOLL + (w ? 256 : 0));
    unsigned rdy = own ? NT : 0, np = 0;
    if (!own) call = __builtin_amdgcn_readfirstlane(__hip_atomic_load(cnt, __ATOMIC_RELAXED, __HIP_MEMORY_SCOPE_AGENT)) + 1u;
    const u64 t0 = __builtin_amdgcn_s_memrealtime();
    auto seen = [&](unsigned f) {
      int n = (int)(f - call * 64u) + 1;
      n = max(n, __builtin_amdgcn_update_dpp(0, n, 0xB1, 0xf, 0xf, false));
      n = max(n, __builtin_amdgcn_update_dpp(0, n, 0x4E, 0xf, 0xf, false));
      int m = __builtin_amdgcn_readlane(n, 0);
#pragma unroll
      for (int i = 1; i < 5; ++i) m = min(m, __builtin_amdgcn_readlane(n, 4 * i));
      rdy = max((int)rdy, m);
    };
    int so = i0 * 2048, pend = 0;
    for (int t = 0; t < NT + 2; ++t) {
      if (t < NT) {
        if (own) {
          const unsigned ox = (31 - u0) * UB + gperm(t, rt) * 128;
#pragma unroll
          for (int i = 0; i < 5; ++i)
            if (i < 4 || w == 0) { DMA(rx, lds + so + i * 2048, xv0, ox - i * UB, 0); DMA(rx, lds + so + i * 2048 + 1024, xv1, ox - i * UB, 0); }
        } else {
          while (rdy <= (unsigned)t) {
            seen(__hip_atomic_load(fp, __ATOMIC_RELAXED, __HIP_MEMORY_SCOPE_SYSTEM));
            if (rdy <= (unsigned)t && (++np & 63u) == 0 && __any(timed_out(tmo, call, t0))) rdy = NT;
          }
          if (rdy < NT && rdy < (unsigned)t + 5 && pend == 0) {
            __builtin_amdgcn_global_load_lds(fp, pl, 4, 0, 17);
            pend = rdy > (unsigned)t + 1 ? 2 : 1;
          }
          const unsigned og = (u0 - 32) * UB + t * 2048;
#pragma unroll
          for (int i = 0; i < 5; ++i)
            if (i < 4 || w == 0) { DMA(rg, lds + so + i * 2048, gv, og + i * UB, 17); DMA(rg, lds + so + i * 2048 + 1024, gv1, og + i * UB, 17); }
        }
        so = so >= 2 * RG::XS ? so - 2 * RG::XS : so + RG::XS;
        if (w == 0) { if (pend == 2) WAITV(11); else WAITV(10); } else { if (pend == 2) WAITV(9); else WAITV(8); }
        if (pend && --pend == 0) seen(*(volatile LAS unsigned*)(pl + lane));
      } else WAITV(0);
      if (t >= 1) TBAR(t - 1);
    }
  } else {
    using RG = Ring<true>;
    const unsigned v0 = (unsigned)(((lane >> 5) + (w - 1) * 32) * N * 2 + (rank * 8 + cl) * 512 + (((lane & 31) ^ ((lane >> 5) << 1)) << 4));
    const unsigned v1 = (v0 ^ 64) + 2 * N * 2, v2 = (v0 ^ 128) + 8 * N * 2, v3 = (v0 ^ 192) + 10 * N * 2;
    int so = 0;
    u64 tp = GPACE ? __builtin_amdgcn_s_memrealtime() + 300 : 0;
    for (int t = 0; t < NT + 2; ++t) {
      if (t < NT) {
        const int kt = gperm(t, rt);
#pragma unroll
        for (int j = 0; j < 16; ++j)
          DMA(rw, lds + RG::W0 + so + (w - 1) * 16384 + j * 1024, (j & 4) ? ((j & 1) ? v3 : v2) : ((j & 1) ? v1 : v0), (kt * 64 + ((j & 2) << 1) + ((j & 8) << 1)) * N * 2, 0);
        so = so == 2 * RG::WS ? 0 : so + RG::WS;
        WAITV(16);
        if (GPACE && w == 1) { while ((long long)(__builtin_amdgcn_s_memrealtime() - tp) < 0) __builtin_amdgcn_s_sleep(1); tp += GPACE; }
      } else WAITV(0);
      if (t >= 1) TBAR(t - 1);
    }
  }
  TSAVE();
  if (w == 0 && call == 0) call = __builtin_amdgcn_readfirstlane(__hip_atomic_load(cnt, __ATOMIC_RELAXED, __HIP_MEMORY_SCOPE_AGENT)) + 1u;
  LBAR();
  if (w == 0) {
    const u32x4 mv = *(volatile LAS u32x4*)(misc + L_MAX);
    const unsigned m = max(max(mv[0], mv[1]), max(mv[2], mv[3]));
    if (lane == 0) __hip_atomic_store((u64*)(myws + WS_GRAN + gslot * 8), ((u64)call << 32) | m, __ATOMIC_RELAXED, __HIP_MEMORY_SCOPE_AGENT);
    const char* gp = myws + WS_GMAX + ((call & 1) * 8 + xcd) * 128 + (lane & 7) * 8;
    unsigned gm = 0xffffffffu;
    const u64 t0 = __builtin_amdgcn_s_memrealtime();
    for (;;) {
      u64 gq; unsigned tm;
      asm volatile("global_load_dwordx2 %0, %2, off sc0 sc1\n\tglobal_load_dword %1, %3, off sc1\n\ts_waitcnt vmcnt(0)"
                   : "=&v"(gq), "=&v"(tm) : "v"(gp), "v"(tmo) : "memory");
      if (__any(tm == call)) break;
      if (__all((unsigned)(gq >> 32) == call)) {
        gm = wave_umax((unsigned)gq);
        break;
      }
      if (__builtin_amdgcn_s_memrealtime() - t0 > DEADLINE) { __hip_atomic_store(tmo, call, __ATOMIC_RELAXED, __HIP_MEMORY_SCOPE_AGENT); break; }
      __builtin_amdgcn_s_sleep(ASLEEP);
    }
    if (lane == 0) misc[L_GMAX] = gm;
  }
  LBAR();
  LBAR();
  if (w == 0 && lane == 0) {
    if (!isG)
      __hip_atomic_store((unsigned*)(p.ws[dst] + WS_DONE + (size_t)(rank * 16 + cl) * 128), call, __ATOMIC_RELAXED, __HIP_MEMORY_SCOPE_SYSTEM);
    __hip_atomic_store(cnt, call, __ATOMIC_RELAXED, __HIP_MEMORY_SCOPE_AGENT);
    STAMP(6);
  }
}

extern "C" void kernel_launch(int rank, int nranks, ncclComm_t comm, hipStream_t stream,
                              void* const* d_in, const int* in_sizes, int n_in,
                              void* d_out, int out_size, const int* in_dtypes, int out_dtype,
                              void* const* d_out_all, void* const* d_ws_all, size_t ws_bytes) {
  if (nranks != NR || (unsigned)rank >= (unsigned)NR || n_in < 2 || ws_bytes < WS_END || in_sizes[0] != M * K || in_sizes[1] != K * N || out_size != NR * M * NS) {
    fprintf(stderr, "unexpected problem shape\n");
    return;
  }
  static bool init = false;
  if (!init) {
    HIP_CHECK(hipFuncSetAttribute((const void*)gemm_a2a, hipFuncAttributeMaxDynamicSharedMemorySize, LDS_BYTES));
    init = true;
  }
  Params p;
  p.x = d_in[0]; p.w = d_in[1];
  for (int i = 0; i < NR; ++i) { p.out[i] = (char*)d_out_all[i]; p.ws[i] = (char*)d_ws_all[i]; }
  p.rank = rank; p.pad = 0;
  gemm_a2a<<<NWG, 512, LDS_BYTES, stream>>>(p);
  HIP_CHECK(hipGetLastError());
}
```

```cpp
#include <hip/hip_runtime.h>
#include <rccl/rccl.h>
#include <cstdio>
#include <utility>

#define HIP_CHECK(expr) do { hipError_t _e = (expr); if (_e != hipSuccess) { \
    fprintf(stderr, "HIP error %s at %s:%d\n", hipGetErrorString(_e), __FILE__, __LINE__); \
    return; } } while (0)

typedef __bf16 bf16x4 __attribute__((ext_vector_type(4)));
typedef __bf16 bf16x8 __attribute__((ext_vector_type(8)));
typedef float f32x4 __attribute__((ext_vector_type(4)));
typedef unsigned u32x4 __attribute__((ext_vector_type(4)));
typedef unsigned long long u64;
#define LAS __attribute__((address_space(3)))
#define DI __device__ __forceinline__

#define GPERM 1
#define PDEPTH 2
#define ASLEEP 2
#define DSLEEP 4
#define SOLO 0
#define XSPEC 3
#define LAUX 2
#define LPACE 116
#define GPACE 0
#define PACE 72
#define NOMMA 0
#define NAGG 1
#define RWARM 0
#define SERP 1
#define XL2 3
#define LT1 6
#define LT2 30
#define LG1 0
#define LG2 0
#define LGT1 0
#define LGT2 0
#define CMIX 0
#define GST 0
#define LINES 0
#define OWIN -1
#define YMODE 0
#define YWIN 1
#define GWPF 0
#define PLEAD 3
#define YLIMIT 5600
#define GT2 64
#define G0 350
#define EPACE 0
#define LT2E 0
#define WFIRST 0

constexpr int NR = 8;
constexpr int M = 512, N = 16384, K = 4096, NT = K / 64;
constexpr int NS = N / NR;
constexpr int NWG = 256;
constexpr int NU = 16, GR = NU * 16;
constexpr int UB = NT * 2048;
constexpr int LDS_BYTES = 163840;
template <bool G> struct Ring;
template <> struct Ring<true> { static constexpr int NI = 9, WROW = 512, XS = 144 * 128, WS = 64 * 512, NW = 3, W0 = 3 * XS; };
template <> struct Ring<false> { static constexpr int NI = 8, WROW = 256, XS = 256 * 128, WS = 64 * 256, NW = 4, W0 = 3 * XS; };
constexpr int GPOLL = Ring<true>::W0 + 3 * Ring<true>::WS;
static_assert(GPOLL + 512 <= LDS_BYTES && Ring<false>::W0 + 4 * Ring<false>::WS <= LDS_BYTES && UB <= LDS_BYTES, "LDS");
constexpr int L_MAX = 0, L_GMAX = 4, L_SEND = 5;
constexpr size_t WS_CNT = 0;
constexpr size_t WS_TMO = WS_CNT + NWG * 128;
constexpr size_t WS_GRAN = WS_TMO + 1024;
constexpr size_t WS_GMAX = WS_GRAN + NWG * 8;
constexpr size_t WS_DONE = WS_GMAX + 2 * 8 * 128;
constexpr size_t WS_XFLG = WS_DONE + NR * 16 * 128;
constexpr size_t WS_XST = 2u << 20;
constexpr size_t WS_YFLG = 1u << 20;
constexpr size_t WS_YST = 16u << 20;
constexpr size_t WS_END = WS_YST + (size_t)2 * 7 * 16 * 65536;
static_assert(WS_XST + (size_t)(NR - 1) * NU * UB <= WS_YST && WS_YFLG + 2 * 7 * 16 * 4 * 128 <= WS_XST, "workspace map");
static_assert(WS_XFLG + (NR - 1) * NU * 128 <= WS_YFLG, "workspace map");
#define STAMP(i) do {} while (0)
#define TBAR(t) BAR()
#define TSAVE() do {} while (0)
constexpr u64 DEADLINE = 30000000ull;

struct Params {
  const void* x; const void* w;
  char* out[NR]; char* ws[NR];
  int rank; int pad;
};

#define WAITV(n) asm volatile("s_waitcnt vmcnt(%0)" ::"n"(n) : "memory")
#define BAR() asm volatile("s_barrier" ::: "memory")
#define LBAR() asm volatile("s_waitcnt lgkmcnt(0)\n\ts_barrier" ::: "memory")
#define SB0() __builtin_amdgcn_sched_barrier(0)
#define DMA(rs, dst, voff, soff, aux) __builtin_amdgcn_raw_ptr_buffer_load_lds(rs, (LAS void*)(dst), 16, voff, soff, 0, aux)

template <int NI> struct Frag { bf16x8 x[NI]; bf16x4 wl[4], wh[4]; };
struct Addr { LAS char* x0; LAS char* x1; LAS char* w[4]; int sx, sw, dx, dw; };

template <bool G, int KS, int R> DI void rd(Frag<Ring<G>::NI>& f, const Addr& a) {
  if constexpr (R < 8) {
    const bf16x4 v = __builtin_amdgcn_ds_read_tr16_b64_v4bf16((LAS bf16x4*)(a.w[R >> 1] + Ring<G>::WROW * (32 * KS + 4 * (R & 1))));
    if constexpr (R & 1) f.wh[R >> 1] = v; else f.wl[R >> 1] = v;
  } else if constexpr (R < 8 + Ring<G>::NI) f.x[R - 8] = *(LAS bf16x8*)((KS ? a.x1 : a.x0) + 2048 * (R - 8));
}
template <bool G, int Q, int KN> DI void gap(f32x4 (&acc)[Ring<G>::NI][4], const Frag<Ring<G>::NI>& fc, Frag<Ring<G>::NI>& fn, Addr& a) {
  using RG = Ring<G>;
  constexpr int I = Q >> 2, J = (SERP && (I & 1)) ? 3 - (Q & 3) : Q & 3;
  acc[I][J] = __builtin_amdgcn_mfma_f32_16x16x32_bf16(__builtin_shufflevector(fc.wl[J], fc.wh[J], 0, 1, 2, 3, 4, 5, 6, 7), fc.x[I], acc[I][J], 0, 0, 0);
  SB0();
  if constexpr (Q % 3 == 0) rd<G, KN, 2 * (Q / 3)>(fn, a);
  if constexpr (Q % 3 == 2) rd<G, KN, 2 * (Q / 3) + 1>(fn, a);
  if constexpr (KN == 1) {
    if constexpr (Q == 13) { a.w[0] += a.dw; asm volatile("" : "+v"(a.w[0])); }
    if constexpr (Q == 16) { a.w[1] += a.dw; asm volatile("" : "+v"(a.w[1])); }
    if constexpr (Q == 19) { a.w[2] += a.dw; asm volatile("" : "+v"(a.w[2])); }
    if constexpr (Q == 22) { a.w[3] += a.dw; asm volatile("" : "+v"(a.w[3])); }
    if constexpr (Q == 25) { a.x0 += a.dx; asm volatile("" : "+v"(a.x0)); }
    if constexpr (Q == 26) { a.x1 += a.dx; asm volatile("" : "+v"(a.x1)); }
  } else {
    if constexpr (Q == 25) { a.sx = a.sx == 2 ? 0 : a.sx + 1; asm volatile("" : "+s"(a.sx)); }
    if constexpr (Q == 26) { a.dx = a.sx == 2 ? -2 * RG::XS : RG::XS; asm volatile("" : "+s"(a.dx)); }
    if constexpr (Q == 27) { a.sw = a.sw == RG::NW - 1 ? 0 : a.sw + 1; asm volatile("" : "+s"(a.sw)); }
    if constexpr (Q == 28) { a.dw = a.sw == RG::NW - 1 ? -(RG::NW - 1) * RG::WS : RG::WS; asm volatile("" : "+s"(a.dw)); }
  }
  if constexpr (Q == 4 * RG::NI - 2) __builtin_amdgcn_s_waitcnt(0xC07F);
  SB0();
}
template <bool G, int KN, int... Qs> DI void kstep(f32x4 (&acc)[Ring<G>::NI][4], const Frag<Ring<G>::NI>& fc, Frag<Ring<G>::NI>& fn, Addr& a, std::integer_sequence<int, Qs...>) {
  (gap<G, Qs, KN>(acc, fc, fn, a), ...);
}
template <bool G, int... Rs> DI void rd_all(Frag<Ring<G>::NI>& f, const Addr& a, std::integer_sequence<int, Rs...>) { (rd<G, 0, Rs>(f, a), ...); }

DI unsigned wave_umax(unsigned v) {
  v = max(v, (unsigned)__builtin_amdgcn_update_dpp(0, (int)v, 0xB1, 0xf, 0xf, false));
  v = max(v, (unsigned)__builtin_amdgcn_update_dpp(0, (int)v, 0x4E, 0xf, 0xf, false));
  v = max(v, (unsigned)__builtin_amdgcn_update_dpp(0, (int)v, 0x141, 0xf, 0xf, false));
  v = max(v, (unsigned)__builtin_amdgcn_update_dpp(0, (int)v, 0x140, 0xf, 0xf, false));
  return max(max((unsigned)__builtin_amdgcn_readlane((int)v, 0), (unsigned)__builtin_amdgcn_readlane((int)v, 16)),
             max((unsigned)__builtin_amdgcn_readlane((int)v, 32), (unsigned)__builtin_amdgcn_readlane((int)v, 48)));
}
DI bool timed_out(unsigned* tmo, unsigned call, u64 t0) {
  if (__hip_atomic_load(tmo, __ATOMIC_RELAXED, __HIP_MEMORY_SCOPE_AGENT) == call || __builtin_amdgcn_s_memrealtime() - t0 > DEADLINE) {
    __hip_atomic_store(tmo, call, __ATOMIC_RELAXED, __HIP_MEMORY_SCOPE_AGENT);
    return true;
  }
  return false;
}
DI int gperm(int u, int rt) { return GPERM && rt >= (XSPEC == 2 ? 2 : 0) ? (u & ~7) | ((u + rt) & 7) : u; }
DI bool early_tile(int cl) { return YMODE == 6 ? (cl & 3) == 1 : YMODE == 7 ? ((cl & 3) == 1 || (cl & 7) == 0) : YMODE == 1 ? (cl & 3) == 0 : YMODE == 2 ? (cl & 1) == 0 : YMODE == 3 ? (cl & 3) != 1 : YMODE == 4 ? ((cl & 1) == 0 || (cl & 7) == 3) : YMODE == 5 ? ((cl & 3) == 0 || (cl & 7) == 2) : false; }
DI int unit_row(int U, int rank) { return U < 32 ? rank * M + (31 - U) * 16 : ((rank + (U >> 4) - 1) & 7) * M + (U & 15) * 16; }

template <bool G> DI void mfma_role(LAS char* lds, int wid, int lane, int b, char* myws, char* obase, int rank, int u0, char* yws, char* ybase, u64 tstart) {
  using RG = Ring<G>;
  constexpr int NI = RG::NI;
  volatile LAS unsigned* misc = (volatile LAS unsigned*)lds;
  const int wm = G ? 0 : wid >> 1, wn = G ? wid : wid & 1;
  const int fr = lane & 15, g = lane >> 4, q = (lane & 15) >> 2, pp = lane & 3;
  const int xo0 = (wm * 128 + fr) * 128 + ((g ^ (fr >> 1)) << 4);
  const int fk = q | ((g & 1) << 2);
  const int wo0 = RG::W0 + RG::WROW * (8 * g + q) + ((wn >> 1) * 256) + ((((wn & 1) * 4) ^ fk) << 5) + 8 * pp;
  f32x4 acc[NI][4];
#pragma unroll
  for (int i = 0; i < NI; ++i)
#pragma unroll
    for (int j = 0; j < 4; ++j) acc[i][j] = (f32x4){0.f, 0.f, 0.f, 0.f};
  Frag<NI> f0, f1;
  Addr a = {lds + xo0, lds + (xo0 ^ 64), {lds + wo0, lds + (wo0 ^ 32), lds + (wo0 ^ 64), lds + (wo0 ^ 96)}, 0, 0, RG::XS, RG::WS};
  SB0();
  BAR();
  rd_all<G>(f0, a, std::make_integer_sequence<int, 8 + NI>{});
  __builtin_amdgcn_s_waitcnt(0xC07F);
  if (NOMMA) for (int s = 0; s < NT; ++s) BAR();
  else for (int s = 0; s < NT; ++s) {
    SB0();
    kstep<G, 1>(acc, f0, f1, a, std::make_integer_sequence<int, 4 * NI>{});
    BAR();
    SB0();
    kstep<G, 0>(acc, f1, f0, a, std::make_integer_sequence<int, 4 * NI>{});
  }
  SB0();
  float m = 0.f, m2 = 0.f;
#pragma unroll
  for (int i = 0; i < NI; ++i)
#pragma unroll
    for (int j = 0; j < 4; ++j) {
      m = fmaxf(fmaxf(m, __builtin_fabsf(acc[i][j][0])), __builtin_fabsf(acc[i][j][1]));
      m2 = fmaxf(fmaxf(m2, __builtin_fabsf(acc[i][j][2])), __builtin_fabsf(acc[i][j][3]));
    }
  const unsigned mu = wave_umax(__float_as_uint(fmaxf(m, m2)));
  if (lane == 0) misc[L_MAX + wid] = mu;
  if (!G && wid == 0 && lane == 0) misc[L_SEND] = __builtin_amdgcn_s_memrealtime() - tstart < YLIMIT;
  LBAR();
  bool bad = false;
  auto finish = [&](char* ob, float pre, auto val) {
    const unsigned gm = misc[L_GMAX];
    bad = bad || gm == 0xffffffffu;
    const float amax = __uint_as_float(gm);
    const float inv = amax > 0.f ? pre * (127.f / amax) : 0.f;
    char* orow = ob + (size_t)(wm * 128 + fr) * NS + wn * 64 + 16 * g;
#pragma unroll
    for (int i = 0; i < NI; ++i) {
      unsigned d4[4];
#pragma unroll
      for (int j = 0; j < 4; ++j) {
        unsigned a0 = __float_as_uint(__builtin_fmaf(val(i, j, 0), inv, 12582912.f)), a1 = __float_as_uint(__builtin_fmaf(val(i, j, 1), inv, 12582912.f));
        unsigned a2 = __float_as_uint(__builtin_fmaf(val(i, j, 2), inv, 12582912.f)), a3 = __float_as_uint(__builtin_fmaf(val(i, j, 3), inv, 12582912.f));
        d4[j] = __builtin_amdgcn_perm(a1, a0, 0x0c0c0400) | __builtin_amdgcn_perm(a3, a2, 0x04000c0c);
      }
      auto r0 = __builtin_amdgcn_permlane32_swap(d4[0], d4[2], false, false); d4[0] = r0[0]; d4[2] = r0[1];
      auto r1 = __builtin_amdgcn_permlane32_swap(d4[1], d4[3], false, false); d4[1] = r1[0]; d4[3] = r1[1];
      auto r2 = __builtin_amdgcn_permlane16_swap(d4[0], d4[1], false, false); d4[0] = r2[0]; d4[1] = r2[1];
      auto r3 = __builtin_amdgcn_permlane16_swap(d4[2], d4[3], false, false); d4[2] = r3[0]; d4[3] = r3[1];
      u32x4 v = {d4[0], d4[1], d4[2], d4[3]};
      if (bad) v = (u32x4){0x80808080u, 0x80808080u, 0x80808080u, 0x80808080u};
      const int r16 = G ? unit_row(u0 + i, rank) : i * 16;
      asm volatile("global_store_dwordx4 %0, %1, off sc0 sc1\n\ts_nop 1" ::"v"(orow + (size_t)r16 * NS), "v"(v) : "memory");
    }
  };
  auto own = [&](int i, int j, int e) { return acc[i][j][e]; };
  if (G || !yws) {
    LBAR();
    if (wid == 0) STAMP(3);
    finish(obase, 1.f, own);
    if (wid == 0) STAMP(4);
    WAITV(0);
    LBAR();
    if (wid == 0) STAMP(5);
    return;
  }
  const bool send = __builtin_amdgcn_readfirstlane(misc[L_SEND]);
  const unsigned call = __builtin_amdgcn_readfirstlane(__hip_atomic_load((unsigned*)(myws + WS_CNT + b * 128), __ATOMIC_RELAXED, __HIP_MEMORY_SCOPE_AGENT)) + 1u;
  const size_t slot = (size_t)(call & 1) * (7 * 16 * 4) + u0 * 4 + wid;
  if (send) {
    const float rs = mu ? 1.f / __uint_as_float(mu) : 0.f;
    char* yp = yws + WS_YST + slot * 16384 + lane * 16;
#pragma unroll
    for (int i = 0; i < NI; ++i)
#pragma unroll
      for (int h = 0; h < 2; ++h) {
        u32x4 v;
#pragma unroll
        for (int e = 0; e < 4; ++e) v[e] = __builtin_bit_cast(unsigned, __builtin_amdgcn_cvt_pknorm_i16(acc[i][2 * h + (e >> 1)][2 * (e & 1)] * rs, acc[i][2 * h + (e >> 1)][2 * (e & 1) + 1] * rs));
        asm volatile("s_waitcnt vmcnt(%2)\n\tglobal_store_dwordx4 %0, %1, off sc0 sc1\n\ts_nop 1" ::"v"(yp + (i * 2 + h) * 1024), "v"(v), "n"(YWIN) : "memory");
      }
    WAITV(0);
  }
  if (lane == 0) __hip_atomic_store((u64*)(yws + WS_YFLG + slot * 128), ((u64)call << 32) | (send ? mu : 0x80000000u), __ATOMIC_RELAXED, __HIP_MEMORY_SCOPE_SYSTEM);
  if (wid == 0) STAMP(7);
  if (!send) {
    LBAR();
    finish(obase, 1.f, own);
    WAITV(0);
    LBAR();
  }
  const u64 t0 = __builtin_amdgcn_s_memrealtime();
  u64 f;
  for (unsigned np = 1;; ++np) {
    f = __hip_atomic_load((u64*)(myws + WS_YFLG + slot * 128), __ATOMIC_RELAXED, __HIP_MEMORY_SCOPE_SYSTEM);
    if (__all((unsigned)(f >> 32) == call)) break;
    if ((np & 63u) == 0 && __any(timed_out((unsigned*)(myws + WS_TMO), call, t0))) { bad = true; f = 0; break; }
    __builtin_amdgcn_s_sleep(4);
  }
  const bool got = __builtin_amdgcn_readfirstlane((unsigned)f) != 0x80000000u;
  if (!got) {
    if (send) { LBAR(); LBAR(); }
    return;
  }
  u32x4 v[16];
  {
    const char* yq = myws + WS_YST + slot * 16384 + lane * 16;
#define YL(n, a, o) "global_load_dwordx4 %" #n ", %" #a ", off offset:" #o " sc0 sc1\n\t"
    asm volatile(YL(0, 16, 0) YL(1, 16, 1024) YL(2, 16, 2048) YL(3, 16, 3072) YL(4, 17, 0) YL(5, 17, 1024) YL(6, 17, 2048) YL(7, 17, 3072)
                 YL(8, 18, 0) YL(9, 18, 1024) YL(10, 18, 2048) YL(11, 18, 3072) YL(12, 19, 0) YL(13, 19, 1024) YL(14, 19, 2048) YL(15, 19, 3072) "s_waitcnt vmcnt(0)"
                 : "=&v"(v[0]), "=&v"(v[1]), "=&v"(v[2]), "=&v"(v[3]), "=&v"(v[4]), "=&v"(v[5]), "=&v"(v[6]), "=&v"(v[7]),
                   "=&v"(v[8]), "=&v"(v[9]), "=&v"(v[10]), "=&v"(v[11]), "=&v"(v[12]), "=&v"(v[13]), "=&v"(v[14]), "=&v"(v[15])
                 : "v"(yq), "v"(yq + 4096), "v"(yq + 8192), "v"(yq + 12288) : "memory");
  }
  if (wid == 0) STAMP(8);
  if (send) { LBAR(); LBAR(); }
  if (wid == 0) STAMP(3);
  finish(ybase, __uint_as_float((unsigned)f) * (1.f / 32767.f), [&](int i, int j, int e) {
    const unsigned q2 = v[(i * 2 + (j >> 1)) & 15][(j & 1) * 2 + (e >> 1)];
    return (e & 1) ? (float)((int)q2 >> 16) : (float)(short)(q2 & 0xffffu); });
  if (wid == 0) STAMP(5);
}

__global__ __launch_bounds__(512, 2) void gemm_a2a(const Params p) {
  extern __shared__ __attribute__((aligned(16))) char smem[];
  LAS char* lds = (LAS char*)smem;
  volatile LAS unsigned* misc = (volatile LAS unsigned*)lds;
  const int tid = threadIdx.x, lane = tid & 63;
  const int wid = __builtin_amdgcn_readfirstlane(tid >> 6);
  const int b = blockIdx.x, xcd = b & 7, idx = b >> 3;
  const int rank = p.rank & 7;
  char* myws = p.ws[rank];
  unsigned* cnt = (unsigned*)(myws + WS_CNT + b * 128);
  unsigned* tmo = (unsigned*)(myws + WS_TMO);
  if (wid == 0) STAMP(0);
  auto rx = __builtin_amdgcn_make_buffer_rsrc((void*)p.x, 0, M * K * 2, 0x00020000);
  const unsigned xv0 = (unsigned)((lane >> 3) * K * 2 + (((lane & 7) ^ (lane >> 4)) << 4)), xv1 = (xv0 ^ 64) + 8 * K * 2;

  const int x4 = xcd >> 1;
  const bool odd = xcd & 1;
  if (XSPEC == 5 ? idx >= (odd ? 29 : 31) : XSPEC >= 3 ? idx >= 30 : (idx >= 28 && odd == (XSPEC == 2))) {
    const int un = XSPEC == 5 ? x4 * 4 + (odd ? idx - 29 : 3) : XSPEC >= 3 ? xcd * 2 + (idx & 1) : x4 * 4 + (idx & 3);
    auto stage = [&](int kt) { DMA(rx, lds + kt * 2048, xv0, un * UB + kt * 128, 0); DMA(rx, lds + kt * 2048 + 1024, xv1, un * UB + kt * 128, 0); };
    stage(wid);
    const int R = max(wid - 1, 0) * NU + un;
    const int grt = (32 + R) / 9;
    char* pw = p.ws[(rank - wid) & 7];
    char* dp = pw + WS_XST + (size_t)R * UB + lane * 16;
    char* fa = pw + WS_XFLG + (size_t)R * 128;
    unsigned c0, c1 = 0;
    if (RWARM) asm volatile("global_load_dword %0, %2, off sc1\n\tglobal_load_dword %1, %3, off sc0 sc1\n\ts_waitcnt vmcnt(1)"
                            : "=&v"(c0), "+v"(c1) : "v"(cnt), "v"(lane & 1 ? dp - lane * 16 : fa) : "memory");
    else asm volatile("global_load_dword %0, %1, off sc1\n\ts_waitcnt vmcnt(0)" : "=&v"(c0) : "v"(cnt) : "memory");
    const unsigned call = __builtin_amdgcn_readfirstlane(c0) + 1u;
    unsigned fv = call * 64u - (PDEPTH - 1);
    LAS char* la = lds + lane * 16;
    u64 tn = PACE ? __builtin_amdgcn_s_memrealtime() - PLEAD * PACE : 0;
#pragma unroll 1
    for (int r = 0; r < 8; ++r) {
      BAR();
      if (wid == 0) {
        if (r < 7) { stage(8 * (r + 1)); WAITV(0); }
        continue;
      }
      u32x4 a0 = *(LAS u32x4*)(la + gperm(r * 8, grt) * 2048), a1 = *(LAS u32x4*)(la + gperm(r * 8, grt) * 2048 + 1024);
#pragma unroll
      for (int j = 0; j < 8; ++j) {
        u32x4 n0 = a0, n1 = a1;
        if (j < 7) { n0 = *(LAS u32x4*)(la + gperm(r * 8 + j + 1, grt) * 2048); n1 = *(LAS u32x4*)(la + gperm(r * 8 + j + 1, grt) * 2048 + 1024); }
        if (PACE) { while ((long long)(__builtin_amdgcn_s_memrealtime() - tn) < 0) __builtin_amdgcn_s_sleep(1); tn += PACE; }
        asm volatile("global_store_dwordx4 %0, %1, off sc0 sc1\n\t"
                     "global_store_dwordx4 %0, %2, off offset:1024 sc0 sc1\n\t"
                     "s_waitcnt vmcnt(%5)\n\t"
                     "s_mov_b64 exec, 1\n\tglobal_store_dword %3, %4, off offset:%6 sc0 sc1\n\ts_mov_b64 exec, -1\n\ts_nop 0"
                     ::"v"(dp), "v"(a0), "v"(a1), "v"(fa), "v"(fv), "n"(3 * (PDEPTH - 1)), "n"((j & 3) * 4) : "memory");
        fv += 1; dp += 2048;
        if (j == 2 && r < 7) stage(8 * (r + 1) + wid);
        a0 = n0; a1 = n1;
      }
    }
    asm volatile("s_waitcnt vmcnt(0)" : "+v"(c1)::"memory");
    if (lane == 0 && wid) __hip_atomic_store((unsigned*)fa, call * 64u + 63u, __ATOMIC_RELAXED, __HIP_MEMORY_SCOPE_SYSTEM);
    if (wid == 1) STAMP(7);
    if (un < NAGG && wid == 0) {
      const char* gp = myws + WS_GRAN + lane * 16;
      const bool skip = lane >= 56;
      unsigned gm = 0xffffffffu;
      const u64 t0 = __builtin_amdgcn_s_memrealtime();
      for (unsigned np = 1;; ++np) {
        u32x4 a, c;
        asm volatile("global_load_dwordx4 %0, %2, off sc1\n\tglobal_load_dwordx4 %1, %2, off offset:1024 sc1\n\ts_waitcnt vmcnt(0)"
                     : "=&v"(a), "=&v"(c) : "v"(gp) : "memory");
        if (__all(a[1] == call && a[3] == call && (skip || (c[1] == call && c[3] == call)))) {
          gm = wave_umax(skip ? max(a[0], a[2]) : max(max(a[0], a[2]), max(c[0], c[2])));
          break;
        }
        if ((np & 63u) == 0 && __any(timed_out(tmo, call, t0))) break;
      }
      __hip_atomic_store((u64*)(p.ws[lane & 7] + WS_GMAX + ((call & 1) * 8 + (lane >> 3)) * 128 + (SOLO ? lane & 7 : rank) * 8), ((u64)call << 32) | gm, __ATOMIC_RELAXED, __HIP_MEMORY_SCOPE_SYSTEM);
      STAMP(8);
    }
    const int fi = un * 8 + wid;
    if (!SOLO && lane == 0 && (fi >> 4) != rank) {
      unsigned* fl = (unsigned*)(myws + WS_DONE + (size_t)fi * 128);
      const u64 t1 = __builtin_amdgcn_s_memrealtime();
      unsigned np = 0;
      while (__hip_atomic_load(fl, __ATOMIC_RELAXED, __HIP_MEMORY_SCOPE_SYSTEM) < call) {
        if ((++np & 63u) == 0 && timed_out(tmo, call, t1)) { p.out[rank][(size_t)rank * M * NS + fi] = (char)-128; break; }
        __builtin_amdgcn_s_sleep(DSLEEP);
      }
    }
    if (tid == 0) __hip_atomic_store(cnt, call, __ATOMIC_RELAXED, __HIP_MEMORY_SCOPE_AGENT);
    if (wid == 0) STAMP(6);
    return;
  }

  const bool isG = XSPEC == 5 ? (odd ? idx < 28 : idx >= 27) : XSPEC == 4 ? (odd ? idx < 24 : idx >= 22) : odd || idx >= 28;
  int rt = 0, cl, dst = rank;
  int dj = idx >> 2;
  if (!isG && XSPEC == 4) { const int li = x4 * 28 + (odd ? idx - 2 : idx); dj = li % 7; cl = (li / 7 + CMIX * dj) & 15; dst = (rank + 1 + dj) & 7; }
  else if (!isG && XSPEC == 5) { const int li = odd ? 27 : idx; dj = li >> 2; cl = x4 * 4 + (li & 3); dst = (rank + 1 + dj) & 7; }
  else if (XSPEC == 5 && !odd) { rt = (idx - 27) & 1; cl = x4 * 2 + ((idx - 27) >> 1); }
  else if (!isG) { cl = (x4 * 4 + (idx & 3) + CMIX * dj) & 15; dst = (rank + 1 + dj) & 7; }
  else if (XSPEC == 4) {
    if (odd) { rt = 4 + (x4 & 1) * 6 + idx % 6; cl = (x4 >> 1) * 4 + idx / 6; }
    else { rt = (idx - 22) & 3; cl = x4 * 2 + ((idx - 22) >> 2); }
  }
  else if (XSPEC == 1) { rt = (x4 & 1) * 8 + (idx & 7); cl = (x4 >> 1) * 4 + (idx >> 3); }
  else if (XSPEC == 3) {
    if (!odd) { rt = 0; cl = x4 * 2 + (idx & 1); }
    else if (idx < 28) { rt = 1 + (x4 & 1) * 8 + idx % 7; cl = (x4 >> 1) * 4 + idx / 7; }
    else { rt = 8; cl = x4 * 2 + (idx & 1); }
  }
  else if (odd) { rt = 2 + (x4 & 1) * 7 + idx % 7; cl = (x4 >> 1) * 4 + idx / 7; }
  else { rt = idx & 1; cl = x4 * 2 + ((idx >> 1) & 1); }
  const bool early = !isG && early_tile(cl);
  const int gslot = isG ? rt * 8 + cl : 128 + dj * 16 + cl;
  if (wid < 4) {
    __builtin_amdgcn_s_setprio(2);
    if (isG) mfma_role<true>(lds, wid, lane, b, myws, p.out[rank] + cl * 256, rank, rt * 9, nullptr, nullptr, 0);
    else mfma_role<false>(lds, wid, lane, b, myws, p.out[dst] + (size_t)(rank * M + GR) * NS + cl * 128, rank, dj * 16 + cl, early ? p.ws[dst] : nullptr,
                          p.out[rank] + (size_t)(((rank - 1 - dj) & 7) * M + GR) * NS + cl * 128, __builtin_amdgcn_s_memrealtime());
    return;
  }
  const int w = wid - 4;
  unsigned call = 0;
  auto rw = __builtin_amdgcn_make_buffer_rsrc((void*)p.w, 0, K * N * 2, 0x00020000);
  if (!isG) {
    using RG = Ring<false>;
    const unsigned v0 = (unsigned)(((lane >> 4) + w * 16) * N * 2 + (dst * 16 + cl) * 256 + (((lane & 15) ^ ((lane >> 4) << 1)) << 4)), v1 = (v0 ^ 128) + 8 * N * 2;
    auto ldx = [&](int t, int sx) {
#pragma unroll
      for (int i = 0; i < 4; ++i) {
        DMA(rx, lds + sx * RG::XS + (w * 4 + i) * 2048, xv0, (NU + w * 4 + i) * UB + t * 128, 0);
        DMA(rx, lds + sx * RG::XS + (w * 4 + i) * 2048 + 1024, xv1, (NU + w * 4 + i) * UB + t * 128, 0);
      }
    };
    auto ldw = [&](int t) {
#pragma unroll
      for (int j = 0; j < 4; ++j)
        DMA(rw, lds + RG::W0 + (t & 3) * RG::WS + (w * 4 + j) * 1024, (j & 2) ? v1 : v0, (t * 64 + (j & 1) * 4) * N * 2, LAUX);
    };
    const bool xpf = XL2 && w == 3 && (idx & 3) != 1;
    unsigned t0_ = 0;
    int sx = 0;
    const int al = (int)((size_t)p.w >> 8) + dst * 16 + cl;
    const bool slow = (al & 3) == 1 || (al & 31) == 31;
    const u64 tb = __builtin_amdgcn_s_memrealtime();
    u64 tp = tb + 200 + LT1 * 69;
    for (int t = 0; t < NT; t += 2) {
      if (WFIRST) { ldw(t); ldw(t + 1); ldx(t, sx); } else { ldx(t, sx); ldw(t); ldw(t + 1); }
      sx = sx == 2 ? 0 : sx + 1;
      if (xpf) {
        const unsigned n = ((t >> 1) * 21 + (idx >> 2) * 3 + max((idx & 3) - 1, 0)) * 64 + lane, kt = (n >> 8) + XL2;
        asm volatile("buffer_load_dword %0, %1, %2, 0 offen" : "+v"(t0_) : "v"(kt < NT ? (GR + (n & 255)) * K * 2 + kt * 128 : 0xfffff000u), "s"(rx) : "memory");
        WAITV(17);
      } else WAITV(16);
      if (t) TBAR(t - 1);
      ldx(t + 1, sx);
      if (EPACE && w == 0 && early) { while ((long long)(__builtin_amdgcn_s_memrealtime() - tb - 200 - t * EPACE) < 0) __builtin_amdgcn_s_sleep(1); }
      if (LG1 && w == 0 && !slow && (t == LGT1 || t == LGT2)) { while ((long long)(__builtin_amdgcn_s_memrealtime() - tb - (t == LGT1 ? LG1 : LG2)) < 0) __builtin_amdgcn_s_sleep(1); }
      if (LPACE && w == 0 && !slow && t >= LT1 && t < (early ? LT2E : LT2)) { while ((long long)(__builtin_amdgcn_s_memrealtime() - tp) < 0) __builtin_amdgcn_s_sleep(1); tp += 2 * LPACE; }
      sx = sx == 2 ? 0 : sx + 1;
      if (xpf) WAITV(13); else if (WFIRST) WAITV(8); else WAITV(12);
      TBAR(t);
    }
    asm volatile("s_waitcnt vmcnt(0)" : "+v"(t0_)::"memory");
    TBAR(NT - 1);
    BAR();
  } else if (w == 0 || w == 3) {
    using RG = Ring<true>;
    const int i0 = w ? 5 : 0, u0 = rt * 9 + i0;
    const bool own = u0 < 32;
    if (w == 3 && rt >= 3) {
      unsigned t0_, t1_, t2_, t3_, t4_;
      const unsigned pv = (unsigned)(((rt - 3) * 8 + cl) * 5 * K * 2 + lane * 128);
      asm volatile("buffer_load_dword %0, %5, %6, 0 offen\n\tbuffer_load_dword %1, %5, %6, %7 offen\n\t"
                   "buffer_load_dword %2, %5, %6, %8 offen\n\tbuffer_load_dword %3, %5, %6, %9 offen\n\t"
                   "buffer_load_dword %4, %5, %6, %10 offen\n\ts_waitcnt vmcnt(0)"
                   : "=&v"(t0_), "=&v"(t1_), "=&v"(t2_), "=&v"(t3_), "=&v"(t4_) : "v"(pv), "s"(rx), "s"(K * 2), "s"(K * 4), "s"(K * 6), "s"(K * 8) : "memory");
    }
    auto rg = __builtin_amdgcn_make_buffer_rsrc((void*)(myws + WS_XST), 0, (NR - 1) * NU * UB, 0x00020000);
    const unsigned gv = lane * 16, gv1 = gv + 1024;
    const unsigned* fp = (const unsigned*)(myws + WS_XFLG + (size_t)(own ? 0 : u0 - 32 + min(lane >> 2, w ? 3 : 4)) * 128) + (lane & 3);
    LAS unsigned* pl = (LAS unsigned*)(lds + GPOLL + (w ? 256 : 0));
    unsigned rdy = own ? NT : 0, np = 0;
    if (!own) call = __builtin_amdgcn_readfirstlane(__hip_atomic_load(cnt, __ATOMIC_RELAXED, __HIP_MEMORY_SCOPE_AGENT)) + 1u;
    const u64 t0 = __builtin_amdgcn_s_memrealtime();
    auto seen = [&](unsigned f) {
      int n = (int)(f - call * 64u) + 1;
      n = max(n, __builtin_amdgcn_update_dpp(0, n, 0xB1, 0xf, 0xf, false));
      n = max(n, __builtin_amdgcn_update_dpp(0, n, 0x4E, 0xf, 0xf, false));
      int m = __builtin_amdgcn_readlane(n, 0);
#pragma unroll
      for (int i = 1; i < 5; ++i) m = min(m, __builtin_amdgcn_readlane(n, 4 * i));
      rdy = max((int)rdy, m);
    };
    int so = i0 * 2048, pend = 0;
    for (int t = 0; t < NT + 2; ++t) {
      if (t < NT) {
        if (own) {
          const unsigned ox = (31 - u0) * UB + gperm(t, rt) * 128;
#pragma unroll
          for (int i = 0; i < 5; ++i)
            if (i < 4 || w == 0) { DMA(rx, lds + so + i * 2048, xv0, ox - i * UB, 0); DMA(rx, lds + so + i * 2048 + 1024, xv1, ox - i * UB, 0); }
        } else {
          if (pend && rdy <= (unsigned)t) {
            if (w == 0) WAITV(10); else WAITV(8);
            seen(*(volatile LAS unsigned*)(pl + lane)); pend = 0;
          }
          while (rdy <= (unsigned)t) {
            seen(__hip_atomic_load(fp, __ATOMIC_RELAXED, __HIP_MEMORY_SCOPE_SYSTEM));
            if (rdy <= (unsigned)t && (++np & 63u) == 0 && __any(timed_out(tmo, call, t0))) rdy = NT;
          }
          if (rdy < NT && rdy < (unsigned)t + 5 && pend == 0) {
            __builtin_amdgcn_global_load_lds(fp, pl, 4, 0, 17);
            pend = 2;
          }
          const unsigned og = (u0 - 32) * UB + t * 2048;
#pragma unroll
          for (int i = 0; i < 5; ++i)
            if (i < 4 || w == 0) { DMA(rg, lds + so + i * 2048, gv, og + i * UB, 17); DMA(rg, lds + so + i * 2048 + 1024, gv1, og + i * UB, 17); }
        }
        so = so >= 2 * RG::XS ? so - 2 * RG::XS : so + RG::XS;
        if (w == 0) { if (pend == 2) WAITV(11); else WAITV(10); } else { if (pend == 2) WAITV(9); else WAITV(8); }
        if (pend && --pend == 0) seen(*(volatile LAS unsigned*)(pl + lane));
      } else WAITV(0);
      if (t >= 1) TBAR(t - 1);
    }
  } else {
    using RG = Ring<true>;
    const unsigned v0 = (unsigned)(((lane >> 5) + (w - 1) * 32) * N * 2 + (rank * 8 + cl) * 512 + (((lane & 31) ^ ((lane >> 5) << 1)) << 4));
    const unsigned v1 = (v0 ^ 64) + 2 * N * 2, v2 = (v0 ^ 128) + 8 * N * 2, v3 = (v0 ^ 192) + 10 * N * 2;
    int so = 0;
    u64 tp = GPACE ? __builtin_amdgcn_s_memrealtime() + G0 : 0;
    for (int t = 0; t < NT + 2; ++t) {
      if (t < NT) {
        const int kt = gperm(t, rt);
#pragma unroll
        for (int j = 0; j < 16; ++j)
          DMA(rw, lds + RG::W0 + so + (w - 1) * 16384 + j * 1024, (j & 4) ? ((j & 1) ? v3 : v2) : ((j & 1) ? v1 : v0), (kt * 64 + ((j & 2) << 1) + ((j & 8) << 1)) * N * 2, 0);
        so = so == 2 * RG::WS ? 0 : so + RG::WS;
        WAITV(16);
        if (GPACE && w == 1 && t < GT2) { while ((long long)(__builtin_amdgcn_s_memrealtime() - tp) < 0) __builtin_amdgcn_s_sleep(1); tp += GPACE; }
      } else WAITV(0);
      if (t >= 1) TBAR(t - 1);
    }
  }
  TSAVE();
  if (w == 0 && call == 0) call = __builtin_amdgcn_readfirstlane(__hip_atomic_load(cnt, __ATOMIC_RELAXED, __HIP_MEMORY_SCOPE_AGENT)) + 1u;
  LBAR();
  if (w == 0) {
    const u32x4 mv = *(volatile LAS u32x4*)(misc + L_MAX);
    const unsigned m = max(max(mv[0], mv[1]), max(mv[2], mv[3]));
    if (lane == 0) __hip_atomic_store((u64*)(myws + WS_GRAN + gslot * 8), ((u64)call << 32) | m, __ATOMIC_RELAXED, __HIP_MEMORY_SCOPE_AGENT);
    const char* gp = myws + WS_GMAX + ((call & 1) * 8 + xcd) * 128 + (lane & 7) * 8;
    unsigned gm = 0xffffffffu;
    const u64 t0 = __builtin_amdgcn_s_memrealtime();
    for (;;) {
      u64 gq; unsigned tm;
      asm volatile("global_load_dwordx2 %0, %2, off sc0 sc1\n\tglobal_load_dword %1, %3, off sc1\n\ts_waitcnt vmcnt(0)"
                   : "=&v"(gq), "=&v"(tm) : "v"(gp), "v"(tmo) : "memory");
      if (__any(tm == call)) break;
      if (__all((unsigned)(gq >> 32) == call)) {
        gm = wave_umax((unsigned)gq);
        break;
      }
      if (__builtin_amdgcn_s_memrealtime() - t0 > DEADLINE) { __hip_atomic_store(tmo, call, __ATOMIC_RELAXED, __HIP_MEMORY_SCOPE_AGENT); break; }
      __builtin_amdgcn_s_sleep(ASLEEP);
    }
    if (lane == 0) misc[L_GMAX] = gm;
  }
  LBAR();
  LBAR();
  if (w == 0 && lane == 0) {
    if (!isG)
      __hip_atomic_store((unsigned*)(p.ws[dst] + WS_DONE + (size_t)(rank * 16 + cl) * 128), call, __ATOMIC_RELAXED, __HIP_MEMORY_SCOPE_SYSTEM);
    __hip_atomic_store(cnt, call, __ATOMIC_RELAXED, __HIP_MEMORY_SCOPE_AGENT);
    STAMP(6);
  }
}

extern "C" void kernel_launch(int rank, int nranks, ncclComm_t comm, hipStream_t stream,
                              void* const* d_in, const int* in_sizes, int n_in,
                              void* d_out, int out_size, const int* in_dtypes, int out_dtype,
                              void* const* d_out_all, void* const* d_ws_all, size_t ws_bytes) {
  if (nranks != NR || (unsigned)rank >= (unsigned)NR || n_in < 2 || ws_bytes < WS_END || in_sizes[0] != M * K || in_sizes[1] != K * N || out_size != NR * M * NS) {
    fprintf(stderr, "unexpected problem shape\n");
    return;
  }
  static bool init = false;
  if (!init) {
    HIP_CHECK(hipFuncSetAttribute((const void*)gemm_a2a, hipFuncAttributeMaxDynamicSharedMemorySize, LDS_BYTES));
    init = true;
  }
  Params p;
  p.x = d_in[0]; p.w = d_in[1];
  for (int i = 0; i < NR; ++i) { p.out[i] = (char*)d_out_all[i]; p.ws[i] = (char*)d_ws_all[i]; }
  p.rank = rank; p.pad = 0;
  gemm_a2a<<<NWG, 512, LDS_BYTES, stream>>>(p);
  HIP_CHECK(hipGetLastError());
}
```

```cpp
#include <hip/hip_runtime.h>
#include <rccl/rccl.h>
#include <cstdio>
#include <utility>

#define HIP_CHECK(expr) do { hipError_t _e = (expr); if (_e != hipSuccess) { \
    fprintf(stderr, "HIP error %s at %s:%d\n", hipGetErrorString(_e), __FILE__, __LINE__); \
    return; } } while (0)

typedef __bf16 bf16x4 __attribute__((ext_vector_type(4)));
typedef __bf16 bf16x8 __attribute__((ext_vector_type(8)));
typedef float f32x4 __attribute__((ext_vector_type(4)));
typedef unsigned u32x4 __attribute__((ext_vector_type(4)));
typedef unsigned long long u64;
#define LAS __attribute__((address_space(3)))
#define DI __device__ __forceinline__

#define LPACE 116
#define LT1 4
#define LT2 32
#define LT2E 20
#define SOLO 0

constexpr int NR = 8;
constexpr int M = 512, N = 16384, K = 4096, NT = K / 64;
constexpr int NS = N / NR;
constexpr int NWG = 256;
constexpr int NU = 16, GR = NU * 16;
constexpr int UB = NT * 2048;
constexpr int LDS_BYTES = 163840;
template <bool G> struct Ring;
template <> struct Ring<true> { static constexpr int NI = 9, WROW = 512, XS = 144 * 128, WS = 64 * 512, NW = 3, W0 = 3 * XS; };
template <> struct Ring<false> { static constexpr int NI = 8, WROW = 256, XS = 256 * 128, WS = 64 * 256, NW = 4, W0 = 3 * XS; };
constexpr int GPOLL = Ring<true>::W0 + 3 * Ring<true>::WS;
static_assert(GPOLL + 512 <= LDS_BYTES && Ring<false>::W0 + 4 * Ring<false>::WS <= LDS_BYTES && UB <= LDS_BYTES, "LDS");
constexpr int L_MAX = 0, L_GMAX = 4;
constexpr size_t WS_CNT = 0;
constexpr size_t WS_TMO = WS_CNT + NWG * 128;
constexpr size_t WS_GRAN = WS_TMO + 1024;
constexpr size_t WS_GMAX = WS_GRAN + NWG * 8;
constexpr size_t WS_DONE = WS_GMAX + 2 * 8 * 128;
constexpr size_t WS_XFLG = WS_DONE + NR * 16 * 128;
constexpr size_t WS_YFLG = 1u << 20;
constexpr size_t WS_XST = 2u << 20;
constexpr size_t WS_YST = 16u << 20;
constexpr size_t WS_END = WS_YST + (size_t)2 * 7 * 16 * 65536;
static_assert(WS_XFLG + (NR - 1) * NU * 128 <= WS_YFLG && WS_YFLG + 2 * 7 * 16 * 4 * 128 <= WS_XST && WS_XST + (size_t)(NR - 1) * NU * UB <= WS_YST, "workspace map");
#define STAMP(i) do {} while (0)
constexpr u64 DEADLINE = 30000000ull;

struct Params {
  const void* x; const void* w;
  char* out[NR]; char* ws[NR];
  int rank; int pad;
};

#define WAITV(n) asm volatile("s_waitcnt vmcnt(%0)" ::"n"(n) : "memory")
#define BAR() asm volatile("s_barrier" ::: "memory")
#define LBAR() asm volatile("s_waitcnt lgkmcnt(0)\n\ts_barrier" ::: "memory")
#define SB0() __builtin_amdgcn_sched_barrier(0)
#define DMA(rs, dst, voff, soff, aux) __builtin_amdgcn_raw_ptr_buffer_load_lds(rs, (LAS void*)(dst), 16, voff, soff, 0, aux)

template <int NI> struct Frag { bf16x8 x[NI]; bf16x4 wl[4], wh[4]; };
struct Addr { LAS char* x0; LAS char* x1; LAS char* w[4]; int sx, sw, dx, dw; };

template <bool G, int KS, int R> DI void rd(Frag<Ring<G>::NI>& f, const Addr& a) {
  if constexpr (R < 8) {
    const bf16x4 v = __builtin_amdgcn_ds_read_tr16_b64_v4bf16((LAS bf16x4*)(a.w[R >> 1] + Ring<G>::WROW * (32 * KS + 4 * (R & 1))));
    if constexpr (R & 1) f.wh[R >> 1] = v; else f.wl[R >> 1] = v;
  } else if constexpr (R < 8 + Ring<G>::NI) f.x[R - 8] = *(LAS bf16x8*)((KS ? a.x1 : a.x0) + 2048 * (R - 8));
}
template <bool G, int Q, int KN> DI void gap(f32x4 (&acc)[Ring<G>::NI][4], const Frag<Ring<G>::NI>& fc, Frag<Ring<G>::NI>& fn, Addr& a) {
  using RG = Ring<G>;
  constexpr int I = Q >> 2, J = (I & 1) ? 3 - (Q & 3) : Q & 3;
  acc[I][J] = __builtin_amdgcn_mfma_f32_16x16x32_bf16(__builtin_shufflevector(fc.wl[J], fc.wh[J], 0, 1, 2, 3, 4, 5, 6, 7), fc.x[I], acc[I][J], 0, 0, 0);
  SB0();
  if constexpr (Q % 3 == 0) rd<G, KN, 2 * (Q / 3)>(fn, a);
  if constexpr (Q % 3 == 2) rd<G, KN, 2 * (Q / 3) + 1>(fn, a);
  if constexpr (KN == 1) {
    if constexpr (Q == 13) { a.w[0] += a.dw; asm volatile("" : "+v"(a.w[0])); }
    if constexpr (Q == 16) { a.w[1] += a.dw; asm volatile("" : "+v"(a.w[1])); }
    if constexpr (Q == 19) { a.w[2] += a.dw; asm volatile("" : "+v"(a.w[2])); }
    if constexpr (Q == 22) { a.w[3] += a.dw; asm volatile("" : "+v"(a.w[3])); }
    if constexpr (Q == 25) { a.x0 += a.dx; asm volatile("" : "+v"(a.x0)); }
    if constexpr (Q == 26) { a.x1 += a.dx; asm volatile("" : "+v"(a.x1)); }
  } else {
    if constexpr (Q == 25) { a.sx = a.sx == 2 ? 0 : a.sx + 1; asm volatile("" : "+s"(a.sx)); }
    if constexpr (Q == 26) { a.dx = a.sx == 2 ? -2 * RG::XS : RG::XS; asm volatile("" : "+s"(a.dx)); }
    if constexpr (Q == 27) { a.sw = a.sw == RG::NW - 1 ? 0 : a.sw + 1; asm volatile("" : "+s"(a.sw)); }
    if constexpr (Q == 28) { a.dw = a.sw == RG::NW - 1 ? -(RG::NW - 1) * RG::WS : RG::WS; asm volatile("" : "+s"(a.dw)); }
  }
  if constexpr (Q == 4 * RG::NI - 2) __builtin_amdgcn_s_waitcnt(0xC07F);
  SB0();
}
template <bool G, int KN, int... Qs> DI void kstep(f32x4 (&acc)[Ring<G>::NI][4], const Frag<Ring<G>::NI>& fc, Frag<Ring<G>::NI>& fn, Addr& a, std::integer_sequence<int, Qs...>) {
  (gap<G, Qs, KN>(acc, fc, fn, a), ...);
}
template <bool G, int... Rs> DI void rd_all(Frag<Ring<G>::NI>& f, const Addr& a, std::integer_sequence<int, Rs...>) { (rd<G, 0, Rs>(f, a), ...); }

DI unsigned wave_umax(unsigned v) {
  v = max(v, (unsigned)__builtin_amdgcn_update_dpp(0, (int)v, 0xB1, 0xf, 0xf, false));
  v = max(v, (unsigned)__builtin_amdgcn_update_dpp(0, (int)v, 0x4E, 0xf, 0xf, false));
  v = max(v, (unsigned)__builtin_amdgcn_update_dpp(0, (int)v, 0x141, 0xf, 0xf, false));
  v = max(v, (unsigned)__builtin_amdgcn_update_dpp(0, (int)v, 0x140, 0xf, 0xf, false));
  return max(max((unsigned)__builtin_amdgcn_readlane((int)v, 0), (unsigned)__builtin_amdgcn_readlane((int)v, 16)),
             max((unsigned)__builtin_amdgcn_readlane((int)v, 32), (unsigned)__builtin_amdgcn_readlane((int)v, 48)));
}
DI bool timed_out(unsigned* tmo, unsigned call, u64 t0) {
  if (__hip_atomic_load(tmo, __ATOMIC_RELAXED, __HIP_MEMORY_SCOPE_AGENT) == call || __builtin_amdgcn_s_memrealtime() - t0 > DEADLINE) {
    __hip_atomic_store(tmo, call, __ATOMIC_RELAXED, __HIP_MEMORY_SCOPE_AGENT);
    return true;
  }
  return false;
}
DI unsigned call_number(const char* myws, int b) {
  return __builtin_amdgcn_readfirstlane(__hip_atomic_load((const unsigned*)(myws + WS_CNT + b * 128), __ATOMIC_RELAXED, __HIP_MEMORY_SCOPE_AGENT)) + 1u;
}
DI int gperm(int u, int rt) { return (u & ~7) | ((u + rt) & 7); }
DI bool early_tile(int cl) { return (cl & 3) == 0; }
DI int unit_row(int U, int rank) { return U < 32 ? rank * M + (31 - U) * 16 : ((rank + (U >> 4) - 1) & 7) * M + (U & 15) * 16; }

template <bool G> DI void mfma_role(LAS char* lds, int wid, int lane, int b, char* myws, char* obase, int rank, int u0, char* yws) {
  using RG = Ring<G>;
  constexpr int NI = RG::NI;
  volatile LAS unsigned* misc = (volatile LAS unsigned*)lds;
  const int wm = G ? 0 : wid >> 1, wn = G ? wid : wid & 1;
  const int fr = lane & 15, g = lane >> 4, q = (lane & 15) >> 2, pp = lane & 3;
  const int xo0 = (wm * 128 + fr) * 128 + ((g ^ (fr >> 1)) << 4);
  const int fk = q | ((g & 1) << 2);
  const int wo0 = RG::W0 + RG::WROW * (8 * g + q) + ((wn >> 1) * 256) + ((((wn & 1) * 4) ^ fk) << 5) + 8 * pp;
  f32x4 acc[NI][4];
#pragma unroll
  for (int i = 0; i < NI; ++i)
#pragma unroll
    for (int j = 0; j < 4; ++j) acc[i][j] = (f32x4){0.f, 0.f, 0.f, 0.f};
  Frag<NI> f0, f1;
  Addr a = {lds + xo0, lds + (xo0 ^ 64), {lds + wo0, lds + (wo0 ^ 32), lds + (wo0 ^ 64), lds + (wo0 ^ 96)}, 0, 0, RG::XS, RG::WS};
  SB0();
  BAR();
  rd_all<G>(f0, a, std::make_integer_sequence<int, 8 + NI>{});
  __builtin_amdgcn_s_waitcnt(0xC07F);
  for (int s = 0; s < NT; ++s) {
    SB0();
    kstep<G, 1>(acc, f0, f1, a, std::make_integer_sequence<int, 4 * NI>{});
    BAR();
    SB0();
    kstep<G, 0>(acc, f1, f0, a, std::make_integer_sequence<int, 4 * NI>{});
  }
  SB0();
  float m = 0.f, m2 = 0.f;
#pragma unroll
  for (int i = 0; i < NI; ++i)
#pragma unroll
    for (int j = 0; j < 4; ++j) {
      m = fmaxf(fmaxf(m, __builtin_fabsf(acc[i][j][0])), __builtin_fabsf(acc[i][j][1]));
      m2 = fmaxf(fmaxf(m2, __builtin_fabsf(acc[i][j][2])), __builtin_fabsf(acc[i][j][3]));
    }
  const unsigned mu = wave_umax(__float_as_uint(fmaxf(m, m2)));
  if (lane == 0) misc[L_MAX + wid] = mu;
  LBAR();
  auto finish = [&](bool bad, float pre, auto val) {
    LBAR();
    if (wid == 0) STAMP(3);
    const unsigned gm = misc[L_GMAX];
    bad = bad || gm == 0xffffffffu;
    const float amax = __uint_as_float(gm);
    const float inv = amax > 0.f ? pre * (127.f / amax) : 0.f;
    char* orow = obase + (size_t)(wm * 128 + fr) * NS + wn * 64 + 16 * g;
#pragma unroll
    for (int i = 0; i < NI; ++i) {
      unsigned d4[4];
#pragma unroll
      for (int j = 0; j < 4; ++j) {
        unsigned a0 = __float_as_uint(__builtin_fmaf(val(i, j, 0), inv, 12582912.f)), a1 = __float_as_uint(__builtin_fmaf(val(i, j, 1), inv, 12582912.f));
        unsigned a2 = __float_as_uint(__builtin_fmaf(val(i, j, 2), inv, 12582912.f)), a3 = __float_as_uint(__builtin_fmaf(val(i, j, 3), inv, 12582912.f));
        d4[j] = __builtin_amdgcn_perm(a1, a0, 0x0c0c0400) | __builtin_amdgcn_perm(a3, a2, 0x04000c0c);
      }
      auto r0 = __builtin_amdgcn_permlane32_swap(d4[0], d4[2], false, false); d4[0] = r0[0]; d4[2] = r0[1];
      auto r1 = __builtin_amdgcn_permlane32_swap(d4[1], d4[3], false, false); d4[1] = r1[0]; d4[3] = r1[1];
      auto r2 = __builtin_amdgcn_permlane16_swap(d4[0], d4[1], false, false); d4[0] = r2[0]; d4[1] = r2[1];
      auto r3 = __builtin_amdgcn_permlane16_swap(d4[2], d4[3], false, false); d4[2] = r3[0]; d4[3] = r3[1];
      u32x4 v = {d4[0], d4[1], d4[2], d4[3]};
      if (bad) v = (u32x4){0x80808080u, 0x80808080u, 0x80808080u, 0x80808080u};
      const int r16 = G ? unit_row(u0 + i, rank) : i * 16;
      asm volatile("global_store_dwordx4 %0, %1, off sc0 sc1\n\ts_nop 1" ::"v"(orow + (size_t)r16 * NS), "v"(v) : "memory");
    }
    if (wid == 0) STAMP(4);
    WAITV(0);
    LBAR();
    if (wid == 0) STAMP(5);
  };
  if (G || !yws) {
    finish(false, 1.f, [&](int i, int j, int e) { return acc[i][j][e]; });
    return;
  }
  const unsigned call = call_number(myws, b);
  const size_t slot = (size_t)(call & 1) * (7 * 16 * 4) + u0 * 4 + wid;
  const float rs = mu ? 1.f / __uint_as_float(mu) : 0.f;
  char* yp = yws + WS_YST + slot * 16384 + lane * 16;
#pragma unroll
  for (int i = 0; i < NI; ++i)
#pragma unroll
    for (int h = 0; h < 2; ++h) {
      u32x4 v;
#pragma unroll
      for (int e = 0; e < 4; ++e) v[e] = __builtin_bit_cast(unsigned, __builtin_amdgcn_cvt_pknorm_i16(acc[i][2 * h + (e >> 1)][2 * (e & 1)] * rs, acc[i][2 * h + (e >> 1)][2 * (e & 1) + 1] * rs));
      asm volatile("s_waitcnt vmcnt(2)\n\tglobal_store_dwordx4 %0, %1, off sc0 sc1\n\ts_nop 1" ::"v"(yp + (i * 2 + h) * 1024), "v"(v) : "memory");
    }
  WAITV(0);
  if (lane == 0) __hip_atomic_store((u64*)(yws + WS_YFLG + slot * 128), ((u64)call << 32) | mu, __ATOMIC_RELAXED, __HIP_MEMORY_SCOPE_SYSTEM);
  if (wid == 0) STAMP(7);
  const u64 t0 = __builtin_amdgcn_s_memrealtime();
  bool bad = false;
  u64 f;
  for (unsigned np = 1;; ++np) {
    f = __hip_atomic_load((u64*)(myws + WS_YFLG + slot * 128), __ATOMIC_RELAXED, __HIP_MEMORY_SCOPE_SYSTEM);
    if (__all((unsigned)(f >> 32) == call)) break;
    if ((np & 63u) == 0 && __any(timed_out((unsigned*)(myws + WS_TMO), call, t0))) { bad = true; break; }
    __builtin_amdgcn_s_sleep(4);
  }
  u32x4 v[16];
  const char* yq = myws + WS_YST + slot * 16384 + lane * 16;
#define YL(n, a, o) "global_load_dwordx4 %" #n ", %" #a ", off offset:" #o " sc0 sc1\n\t"
  asm volatile(YL(0, 16, 0) YL(1, 16, 1024) YL(2, 16, 2048) YL(3, 16, 3072) YL(4, 17, 0) YL(5, 17, 1024) YL(6, 17, 2048) YL(7, 17, 3072)
               YL(8, 18, 0) YL(9, 18, 1024) YL(10, 18, 2048) YL(11, 18, 3072) YL(12, 19, 0) YL(13, 19, 1024) YL(14, 19, 2048) YL(15, 19, 3072) "s_waitcnt vmcnt(0)"
               : "=&v"(v[0]), "=&v"(v[1]), "=&v"(v[2]), "=&v"(v[3]), "=&v"(v[4]), "=&v"(v[5]), "=&v"(v[6]), "=&v"(v[7]),
                 "=&v"(v[8]), "=&v"(v[9]), "=&v"(v[10]), "=&v"(v[11]), "=&v"(v[12]), "=&v"(v[13]), "=&v"(v[14]), "=&v"(v[15])
               : "v"(yq), "v"(yq + 4096), "v"(yq + 8192), "v"(yq + 12288) : "memory");
  if (wid == 0) STAMP(8);
  finish(bad, __uint_as_float((unsigned)f) * (1.f / 32767.f), [&](int i, int j, int e) {
    const unsigned q2 = v[(i * 2 + (j >> 1)) & 15][(j & 1) * 2 + (e >> 1)];
    return (e & 1) ? (float)((int)q2 >> 16) : (float)(short)(q2 & 0xffffu); });
}

__global__ __launch_bounds__(512, 2) void gemm_a2a(const Params p) {
  extern __shared__ __attribute__((aligned(16))) char smem[];
  LAS char* lds = (LAS char*)smem;
  volatile LAS unsigned* misc = (volatile LAS unsigned*)lds;
  const int tid = threadIdx.x, lane = tid & 63;
  const int wid = __builtin_amdgcn_readfirstlane(tid >> 6);
  const int b = blockIdx.x, xcd = b & 7, idx = b >> 3;
  const int rank = p.rank & 7;
  char* myws = p.ws[rank];
  unsigned* cnt = (unsigned*)(myws + WS_CNT + b * 128);
  unsigned* tmo = (unsigned*)(myws + WS_TMO);
  if (wid == 0) STAMP(0);
  auto rx = __builtin_amdgcn_make_buffer_rsrc((void*)p.x, 0, M * K * 2, 0x00020000);
  const unsigned xv0 = (unsigned)((lane >> 3) * K * 2 + (((lane & 7) ^ (lane >> 4)) << 4)), xv1 = (xv0 ^ 64) + 8 * K * 2;

  const int x4 = xcd >> 1;
  const bool isG = xcd & 1;
  if (!isG && idx >= 28) {
    const int un = x4 * 4 + (idx & 3);
    auto stage = [&](int kt) { DMA(rx, lds + kt * 2048, xv0, un * UB + kt * 128, 0); DMA(rx, lds + kt * 2048 + 1024, xv1, un * UB + kt * 128, 0); };
    stage(wid);
    const int R = max(wid - 1, 0) * NU + un;
    const int grt = (32 + R) / 9;
    char* pw = p.ws[(rank - wid) & 7];
    char* dp = pw + WS_XST + (size_t)R * UB + lane * 16;
    char* fa = pw + WS_XFLG + (size_t)R * 128;
    unsigned c0;
    asm volatile("global_load_dword %0, %1, off sc1\n\ts_waitcnt vmcnt(0)" : "=&v"(c0) : "v"(cnt) : "memory");
    const unsigned call = __builtin_amdgcn_readfirstlane(c0) + 1u;
    unsigned fv = call * 64u - 1u;
    LAS char* la = lds + lane * 16;
#pragma unroll 1
    for (int r = 0; r < 8; ++r) {
      BAR();
      if (wid == 0) {
        if (r < 7) stage(8 * (r + 1));
        WAITV(0);
        continue;
      }
      u32x4 a0 = *(LAS u32x4*)(la + gperm(r * 8, grt) * 2048), a1 = *(LAS u32x4*)(la + gperm(r * 8, grt) * 2048 + 1024);
#pragma unroll
      for (int j = 0; j < 8; ++j) {
        u32x4 n0 = a0, n1 = a1;
        if (j < 7) { n0 = *(LAS u32x4*)(la + gperm(r * 8 + j + 1, grt) * 2048); n1 = *(LAS u32x4*)(la + gperm(r * 8 + j + 1, grt) * 2048 + 1024); }
        asm volatile("global_store_dwordx4 %0, %1, off sc0 sc1\n\t"
                     "global_store_dwordx4 %0, %2, off offset:1024 sc0 sc1\n\t"
                     "s_waitcnt vmcnt(3)\n\t"
                     "s_mov_b64 exec, 1\n\tglobal_store_dword %3, %4, off offset:%5 sc0 sc1\n\ts_mov_b64 exec, -1\n\ts_nop 0"
                     ::"v"(dp), "v"(a0), "v"(a1), "v"(fa), "v"(fv), "n"((j & 3) * 4) : "memory");
        fv += 1; dp += 2048;
        if (j == 2 && r < 7) stage(8 * (r + 1) + wid);
        a0 = n0; a1 = n1;
      }
    }
    WAITV(0);
    if (lane == 0 && wid) __hip_atomic_store((unsigned*)fa, call * 64u + 63u, __ATOMIC_RELAXED, __HIP_MEMORY_SCOPE_SYSTEM);
    if (wid == 1) STAMP(7);
    if (un == 0 && wid == 0) {
      const char* gp = myws + WS_GRAN + lane * 16;
      const bool skip = lane >= 56;
      unsigned gm = 0xffffffffu;
      const u64 t0 = __builtin_amdgcn_s_memrealtime();
      for (unsigned np = 1;; ++np) {
        u32x4 a, c;
        asm volatile("global_load_dwordx4 %0, %2, off sc1\n\tglobal_load_dwordx4 %1, %2, off offset:1024 sc1\n\ts_waitcnt vmcnt(0)"
                     : "=&v"(a), "=&v"(c) : "v"(gp) : "memory");
        if (__all(a[1] == call && a[3] == call && (skip || (c[1] == call && c[3] == call)))) {
          gm = wave_umax(skip ? max(a[0], a[2]) : max(max(a[0], a[2]), max(c[0], c[2])));
          break;
        }
        if ((np & 63u) == 0 && __any(timed_out(tmo, call, t0))) break;
      }
      __hip_atomic_store((u64*)(p.ws[lane & 7] + WS_GMAX + ((call & 1) * 8 + (lane >> 3)) * 128 + (SOLO ? lane & 7 : rank) * 8), ((u64)call << 32) | gm, __ATOMIC_RELAXED, __HIP_MEMORY_SCOPE_SYSTEM);
      STAMP(8);
    }
    const int fi = un * 8 + wid;
    if (!SOLO && lane == 0 && (fi >> 4) != rank && !early_tile(fi & 15)) {
      unsigned* fl = (unsigned*)(myws + WS_DONE + (size_t)fi * 128);
      const u64 t1 = __builtin_amdgcn_s_memrealtime();
      unsigned np = 0;
      while (__hip_atomic_load(fl, __ATOMIC_RELAXED, __HIP_MEMORY_SCOPE_SYSTEM) < call) {
        if ((++np & 63u) == 0 && timed_out(tmo, call, t1)) { p.out[rank][(size_t)rank * M * NS + fi] = (char)-128; break; }
        __builtin_amdgcn_s_sleep(4);
      }
    }
    if (tid == 0) __hip_atomic_store(cnt, call, __ATOMIC_RELAXED, __HIP_MEMORY_SCOPE_AGENT);
    if (wid == 1) STAMP(6);
    return;
  }

  const int rt = (x4 & 1) * 8 + (idx & 7), dj = idx >> 2;
  const int cl = isG ? (x4 >> 1) * 4 + (idx >> 3) : x4 * 4 + (idx & 3);
  const int dst = (rank + 1 + dj) & 7;
  const bool early = !isG && early_tile(cl);
  if (wid < 4) {
    __builtin_amdgcn_s_setprio(2);
    if (isG) mfma_role<true>(lds, wid, lane, b, myws, p.out[rank] + cl * 256, rank, rt * 9, nullptr);
    else mfma_role<false>(lds, wid, lane, b, myws, (early ? p.out[rank] + (size_t)(((rank - 1 - dj) & 7) * M + GR) * NS : p.out[dst] + (size_t)(rank * M + GR) * NS) + cl * 128,
                          rank, dj * 16 + cl, early ? p.ws[dst] : nullptr);
    return;
  }
  const int w = wid - 4;
  unsigned call = 0;
  auto rw = __builtin_amdgcn_make_buffer_rsrc((void*)p.w, 0, K * N * 2, 0x00020000);
  if (!isG) {
    using RG = Ring<false>;
    const unsigned v0 = (unsigned)(((lane >> 4) + w * 16) * N * 2 + (dst * 16 + cl) * 256 + (((lane & 15) ^ ((lane >> 4) << 1)) << 4)), v1 = (v0 ^ 128) + 8 * N * 2;
    auto ldx = [&](int t, int sx) {
#pragma unroll
      for (int i = 0; i < 4; ++i) {
        DMA(rx, lds + sx * RG::XS + (w * 4 + i) * 2048, xv0, (NU + w * 4 + i) * UB + t * 128, 0);
        DMA(rx, lds + sx * RG::XS + (w * 4 + i) * 2048 + 1024, xv1, (NU + w * 4 + i) * UB + t * 128, 0);
      }
    };
    auto ldw = [&](int t) {
#pragma unroll
      for (int j = 0; j < 4; ++j)
        DMA(rw, lds + RG::W0 + (t & 3) * RG::WS + (w * 4 + j) * 1024, (j & 2) ? v1 : v0, (t * 64 + (j & 1) * 4) * N * 2, 2);
    };
    const bool xpf = w == 3 && (cl & 3) != 1;
    const int xq = dj * 3 + max((cl & 3) - 1, 0);
    unsigned t0_ = 0;
    const int al = (int)((size_t)p.w >> 8) + dst * 16 + cl;
    const int pend_ = w != 0 || (al & 3) == 1 || (al & 31) == 31 ? 0 : early ? LT2E : LT2;
    u64 tp = __builtin_amdgcn_s_memrealtime() + 200 + LT1 * 69;
    int sx = 0;
    for (int t = 0; t < NT; t += 2) {
      ldx(t, sx); ldw(t); ldw(t + 1);
      sx = sx == 2 ? 0 : sx + 1;
      if (xpf) {
        const unsigned n = ((t >> 1) * 21 + xq) * 64 + lane, kt = (n >> 8) + 3;
        asm volatile("buffer_load_dword %0, %1, %2, 0 offen" : "+v"(t0_) : "v"(kt < NT ? (GR + (n & 255)) * K * 2 + kt * 128 : 0xfffff000u), "s"(rx) : "memory");
        WAITV(17);
      } else WAITV(16);
      if (t) BAR();
      ldx(t + 1, sx);
      sx = sx == 2 ? 0 : sx + 1;
      if (t >= LT1 && t < pend_) { while ((long long)(__builtin_amdgcn_s_memrealtime() - tp) < 0) __builtin_amdgcn_s_sleep(1); tp += 2 * LPACE; }
      if (xpf) WAITV(13); else WAITV(12);
      BAR();
    }
    asm volatile("s_waitcnt vmcnt(0)" : "+v"(t0_)::"memory");
    BAR();
    BAR();
  } else if (w == 0 || w == 3) {
    using RG = Ring<true>;
    const int i0 = w ? 5 : 0, u0 = rt * 9 + i0;
    const bool own = u0 < 32;
    auto rg = __builtin_amdgcn_make_buffer_rsrc((void*)(myws + WS_XST), 0, (NR - 1) * NU * UB, 0x00020000);
    const unsigned gv = lane * 16, gv1 = gv + 1024;
    const unsigned* fp = (const unsigned*)(myws + WS_XFLG + (size_t)(own ? 0 : u0 - 32 + min(lane >> 2, w ? 3 : 4)) * 128) + (lane & 3);
    LAS unsigned* pl = (LAS unsigned*)(lds + GPOLL + (w ? 256 : 0));
    unsigned rdy = own ? NT : 0, np = 0;
    if (!own) call = call_number(myws, b);
    const u64 t0 = __builtin_amdgcn_s_memrealtime();
    auto seen = [&](unsigned f) {
      int n = (int)(f - call * 64u) + 1;
      n = max(n, __builtin_amdgcn_update_dpp(0, n, 0xB1, 0xf, 0xf, false));
      n = max(n, __builtin_amdgcn_update_dpp(0, n, 0x4E, 0xf, 0xf, false));
      int m = __builtin_amdgcn_readlane(n, 0);
#pragma unroll
      for (int i = 1; i < 5; ++i) m = min(m, __builtin_amdgcn_readlane(n, 4 * i));
      rdy = max((int)rdy, m);
    };
    int so = i0 * 2048, pend = 0;
    for (int t = 0; t < NT + 2; ++t) {
      if (t < NT) {
        if (own) {
          const unsigned ox = (31 - u0) * UB + gperm(t, rt) * 128;
#pragma unroll
          for (int i = 0; i < 5; ++i)
            if (i < 4 || w == 0) { DMA(rx, lds + so + i * 2048, xv0, ox - i * UB, 0); DMA(rx, lds + so + i * 2048 + 1024, xv1, ox - i * UB, 0); }
        } else {
          if (pend && rdy <= (unsigned)t) {
            if (w == 0) WAITV(10); else WAITV(8);
            seen(*(volatile LAS unsigned*)(pl + lane)); pend = 0;
          }
          while (rdy <= (unsigned)t) {
            seen(__hip_atomic_load(fp, __ATOMIC_RELAXED, __HIP_MEMORY_SCOPE_SYSTEM));
            if (rdy <= (unsigned)t && (++np & 63u) == 0 && __any(timed_out(tmo, call, t0))) rdy = NT;
          }
          if (rdy < NT && rdy < (unsigned)t + 5 && pend == 0) {
            __builtin_amdgcn_global_load_lds(fp, pl, 4, 0, 17);
            pend = 2;
          }
          const unsigned og = (u0 - 32) * UB + t * 2048;
#pragma unroll
          for (int i = 0; i < 5; ++i)
            if (i < 4 || w == 0) { DMA(rg, lds + so + i * 2048, gv, og + i * UB, 17); DMA(rg, lds + so + i * 2048 + 1024, gv1, og + i * UB, 17); }
        }
        so = so >= 2 * RG::XS ? so - 2 * RG::XS : so + RG::XS;
        if (w == 0) { if (pend == 2) WAITV(11); else WAITV(10); } else { if (pend == 2) WAITV(9); else WAITV(8); }
        if (pend && --pend == 0) seen(*(volatile LAS unsigned*)(pl + lane));
      } else WAITV(0);
      if (t >= 1) BAR();
    }
  } else {
    using RG = Ring<true>;
    const unsigned v0 = (unsigned)(((lane >> 5) + (w - 1) * 32) * N * 2 + (rank * 8 + cl) * 512 + (((lane & 31) ^ ((lane >> 5) << 1)) << 4));
    const unsigned v1 = (v0 ^ 64) + 2 * N * 2, v2 = (v0 ^ 128) + 8 * N * 2, v3 = (v0 ^ 192) + 10 * N * 2;
    int so = 0;
    for (int t = 0; t < NT + 2; ++t) {
      if (t < NT) {
        const int kt = gperm(t, rt);
#pragma unroll
        for (int j = 0; j < 16; ++j)
          DMA(rw, lds + RG::W0 + so + (w - 1) * 16384 + j * 1024, (j & 4) ? ((j & 1) ? v3 : v2) : ((j & 1) ? v1 : v0), (kt * 64 + ((j & 2) << 1) + ((j & 8) << 1)) * N * 2, 0);
        so = so == 2 * RG::WS ? 0 : so + RG::WS;
        WAITV(16);
      } else WAITV(0);
      if (t >= 1) BAR();
    }
  }
  if (w == 0 && call == 0) call = call_number(myws, b);
  LBAR();
  if (w == 0) {
    const u32x4 mv = *(volatile LAS u32x4*)(misc + L_MAX);
    const unsigned m = max(max(mv[0], mv[1]), max(mv[2], mv[3]));
    const int gslot = isG ? rt * 8 + cl : 128 + dj * 16 + cl;
    if (lane == 0) __hip_atomic_store((u64*)(myws + WS_GRAN + gslot * 8), ((u64)call << 32) | m, __ATOMIC_RELAXED, __HIP_MEMORY_SCOPE_AGENT);
    const char* gp = myws + WS_GMAX + ((call & 1) * 8 + xcd) * 128 + (lane & 7) * 8;
    unsigned gm = 0xffffffffu;
    const u64 t0 = __builtin_amdgcn_s_memrealtime();
    for (;;) {
      u64 gq; unsigned tm;
      asm volatile("global_load_dwordx2 %0, %2, off sc0 sc1\n\tglobal_load_dword %1, %3, off sc1\n\ts_waitcnt vmcnt(0)"
                   : "=&v"(gq), "=&v"(tm) : "v"(gp), "v"(tmo) : "memory");
      if (__any(tm == call)) break;
      if (__all((unsigned)(gq >> 32) == call)) { gm = wave_umax((unsigned)gq); break; }
      if (__builtin_amdgcn_s_memrealtime() - t0 > DEADLINE) { __hip_atomic_store(tmo, call, __ATOMIC_RELAXED, __HIP_MEMORY_SCOPE_AGENT); break; }
      __builtin_amdgcn_s_sleep(2);
    }
    if (lane == 0) misc[L_GMAX] = gm;
  }
  LBAR();
  LBAR();
  if (w == 0 && lane == 0) {
    if (!isG && !early)
      __hip_atomic_store((unsigned*)(p.ws[dst] + WS_DONE + (size_t)(rank * 16 + cl) * 128), call, __ATOMIC_RELAXED, __HIP_MEMORY_SCOPE_SYSTEM);
    __hip_atomic_store(cnt, call, __ATOMIC_RELAXED, __HIP_MEMORY_SCOPE_AGENT);
    STAMP(6);
  }
}

extern "C" void kernel_launch(int rank, int nranks, ncclComm_t comm, hipStream_t stream,
                              void* const* d_in, const int* in_sizes, int n_in,
                              void* d_out, int out_size, const int* in_dtypes, int out_dtype,
                              void* const* d_out_all, void* const* d_ws_all, size_t ws_bytes) {
  if (nranks != NR || (unsigned)rank >= (unsigned)NR || n_in < 2 || ws_bytes < WS_END || in_sizes[0] != M * K || in_sizes[1] != K * N || out_size != NR * M * NS) {
    fprintf(stderr, "unexpected problem shape\n");
    return;
  }
  static bool init = false;
  if (!init) {
    HIP_CHECK(hipFuncSetAttribute((const void*)gemm_a2a, hipFuncAttributeMaxDynamicSharedMemorySize, LDS_BYTES));
    init = true;
  }
  Params p;
  p.x = d_in[0]; p.w = d_in[1];
  for (int i = 0; i < NR; ++i) { p.out[i] = (char*)d_out_all[i]; p.ws[i] = (char*)d_ws_all[i]; }
  p.rank = rank; p.pad = 0;
  gemm_a2a<<<NWG, 512, LDS_BYTES, stream>>>(p);
  HIP_CHECK(hipGetLastError());
}
```

```cpp
#include <hip/hip_runtime.h>
#include <rccl/rccl.h>
#include <cstdio>
#include <utility>

#define HIP_CHECK(expr) do { hipError_t _e = (expr); if (_e != hipSuccess) { \
    fprintf(stderr, "HIP error %s at %s:%d\n", hipGetErrorString(_e), __FILE__, __LINE__); \
    return; } } while (0)

typedef __bf16 bf16x4 __attribute__((ext_vector_type(4)));
typedef __bf16 bf16x8 __attribute__((ext_vector_type(8)));
typedef float f32x4 __attribute__((ext_vector_type(4)));
typedef unsigned u32x4 __attribute__((ext_vector_type(4)));
typedef unsigned long long u64;
#define LAS __attribute__((address_space(3)))
#define DI __device__ __forceinline__

#define LPACE 116
#define LT1 4
#define LT2 32
#define LT2E 20
#define SOLO 0

constexpr int NR = 8;
constexpr int M = 512, N = 16384, K = 4096, NT = K / 64;
constexpr int NS = N / NR;
constexpr int NWG = 256;
constexpr int NU = 16, GR = NU * 16;
constexpr int UB = NT * 2048;
constexpr int LDS_BYTES = 163840;
template <bool G> struct Ring;
template <> struct Ring<true> { static constexpr int NI = 9, WROW = 512, XS = 144 * 128, WS = 64 * 512, NW = 3, W0 = 3 * XS; };
template <> struct Ring<false> { static constexpr int NI = 8, WROW = 256, XS = 256 * 128, WS = 64 * 256, NW = 4, W0 = 3 * XS; };
constexpr int GPOLL = Ring<true>::W0 + 3 * Ring<true>::WS;
static_assert(GPOLL + 512 <= LDS_BYTES && Ring<false>::W0 + 4 * Ring<false>::WS <= LDS_BYTES && UB <= LDS_BYTES, "LDS");
constexpr int L_MAX = 0, L_GMAX = 4;
constexpr size_t WS_CNT = 0;
constexpr size_t WS_TMO = WS_CNT + NWG * 128;
constexpr size_t WS_GRAN = WS_TMO + 1024;
constexpr size_t WS_GMAX = WS_GRAN + NWG * 8;
constexpr size_t WS_DONE = WS_GMAX + 2 * 8 * 128;
constexpr size_t WS_XFLG = WS_DONE + NR * 16 * 128;
constexpr size_t WS_YFLG = 1u << 20;
constexpr size_t WS_XST = 2u << 20;
constexpr size_t WS_YST = 16u << 20;
constexpr size_t WS_END = WS_YST + (size_t)2 * 7 * 16 * 65536;
static_assert(WS_XFLG + (NR - 1) * NU * 128 <= WS_YFLG && WS_YFLG + 2 * 7 * 16 * 4 * 128 <= WS_XST && WS_XST + (size_t)(NR - 1) * NU * UB <= WS_YST, "workspace map");
#define STAMP(i) do {} while (0)
constexpr u64 DEADLINE = 30000000ull;

struct Params {
  const void* x; const void* w;
  char* out[NR]; char* ws[NR];
  int rank; int pad;
};

#define WAITV(n) asm volatile("s_waitcnt vmcnt(%0)" ::"n"(n) : "memory")
#define BAR() asm volatile("s_barrier" ::: "memory")
#define LBAR() asm volatile("s_waitcnt lgkmcnt(0)\n\ts_barrier" ::: "memory")
#define SB0() __builtin_amdgcn_sched_barrier(0)
#define DMA(rs, dst, voff, soff, aux) __builtin_amdgcn_raw_ptr_buffer_load_lds(rs, (LAS void*)(dst), 16, voff, soff, 0, aux)

template <int NI> struct Frag { bf16x8 x[NI]; bf16x4 wl[4], wh[4]; };
struct Addr { LAS char* x0; LAS char* x1; LAS char* w[4]; int sx, sw, dx, dw; };

template <bool G, int KS, int R> DI void rd(Frag<Ring<G>::NI>& f, const Addr& a) {
  if constexpr (R < 8) {
    const bf16x4 v = __builtin_amdgcn_ds_read_tr16_b64_v4bf16((LAS bf16x4*)(a.w[R >> 1] + Ring<G>::WROW * (32 * KS + 4 * (R & 1))));
    if constexpr (R & 1) f.wh[R >> 1] = v; else f.wl[R >> 1] = v;
  } else if constexpr (R < 8 + Ring<G>::NI) f.x[R - 8] = *(LAS bf16x8*)((KS ? a.x1 : a.x0) + 2048 * (R - 8));
}
template <bool G, int Q, int KN> DI void gap(f32x4 (&acc)[Ring<G>::NI][4], const Frag<Ring<G>::NI>& fc, Frag<Ring<G>::NI>& fn, Addr& a) {
  using RG = Ring<G>;
  constexpr int I = Q >> 2, J = (I & 1) ? 3 - (Q & 3) : Q & 3;
  acc[I][J] = __builtin_amdgcn_mfma_f32_16x16x32_bf16(__builtin_shufflevector(fc.wl[J], fc.wh[J], 0, 1, 2, 3, 4, 5, 6, 7), fc.x[I], acc[I][J], 0, 0, 0);
  SB0();
  if constexpr (Q % 3 == 0) rd<G, KN, 2 * (Q / 3)>(fn, a);
  if constexpr (Q % 3 == 2) rd<G, KN, 2 * (Q / 3) + 1>(fn, a);
  if constexpr (KN == 1) {
    if constexpr (Q == 13) { a.w[0] += a.dw; asm volatile("" : "+v"(a.w[0])); }
    if constexpr (Q == 16) { a.w[1] += a.dw; asm volatile("" : "+v"(a.w[1])); }
    if constexpr (Q == 19) { a.w[2] += a.dw; asm volatile("" : "+v"(a.w[2])); }
    if constexpr (Q == 22) { a.w[3] += a.dw; asm volatile("" : "+v"(a.w[3])); }
    if constexpr (Q == 25) { a.x0 += a.dx; asm volatile("" : "+v"(a.x0)); }
    if constexpr (Q == 26) { a.x1 += a.dx; asm volatile("" : "+v"(a.x1)); }
  } else {
    if constexpr (Q == 25) { a.sx = a.sx == 2 ? 0 : a.sx + 1; asm volatile("" : "+s"(a.sx)); }
    if constexpr (Q == 26) { a.dx = a.sx == 2 ? -2 * RG::XS : RG::XS; asm volatile("" : "+s"(a.dx)); }
    if constexpr (Q == 27) { a.sw = a.sw == RG::NW - 1 ? 0 : a.sw + 1; asm volatile("" : "+s"(a.sw)); }
    if constexpr (Q == 28) { a.dw = a.sw == RG::NW - 1 ? -(RG::NW - 1) * RG::WS : RG::WS; asm volatile("" : "+s"(a.dw)); }
  }
  if constexpr (Q == 4 * RG::NI - 2) __builtin_amdgcn_s_waitcnt(0xC07F);
  SB0();
}
template <bool G, int KN, int... Qs> DI void kstep(f32x4 (&acc)[Ring<G>::NI][4], const Frag<Ring<G>::NI>& fc, Frag<Ring<G>::NI>& fn, Addr& a, std::integer_sequence<int, Qs...>) {
  (gap<G, Qs, KN>(acc, fc, fn, a), ...);
}
template <bool G, int... Rs> DI void rd_all(Frag<Ring<G>::NI>& f, const Addr& a, std::integer_sequence<int, Rs...>) { (rd<G, 0, Rs>(f, a), ...); }

DI unsigned wave_umax(unsigned v) {
  v = max(v, (unsigned)__builtin_amdgcn_update_dpp(0, (int)v, 0xB1, 0xf, 0xf, false));
  v = max(v, (unsigned)__builtin_amdgcn_update_dpp(0, (int)v, 0x4E, 0xf, 0xf, false));
  v = max(v, (unsigned)__builtin_amdgcn_update_dpp(0, (int)v, 0x141, 0xf, 0xf, false));
  v = max(v, (unsigned)__builtin_amdgcn_update_dpp(0, (int)v, 0x140, 0xf, 0xf, false));
  return max(max((unsigned)__builtin_amdgcn_readlane((int)v, 0), (unsigned)__builtin_amdgcn_readlane((int)v, 16)),
             max((unsigned)__builtin_amdgcn_readlane((int)v, 32), (unsigned)__builtin_amdgcn_readlane((int)v, 48)));
}
DI bool timed_out(unsigned* tmo, unsigned call, u64 t0) {
  if (__hip_atomic_load(tmo, __ATOMIC_RELAXED, __HIP_MEMORY_SCOPE_AGENT) == call || __builtin_amdgcn_s_memrealtime() - t0 > DEADLINE) {
    __hip_atomic_store(tmo, call, __ATOMIC_RELAXED, __HIP_MEMORY_SCOPE_AGENT);
    return true;
  }
  return false;
}
DI unsigned call_number(const char* myws, int b) {
  return __builtin_amdgcn_readfirstlane(__hip_atomic_load((const unsigned*)(myws + WS_CNT + b * 128), __ATOMIC_RELAXED, __HIP_MEMORY_SCOPE_AGENT)) + 1u;
}
DI int gperm(int u, int rt) { return (u & ~7) | ((u + rt) & 7); }
DI bool early_tile(int cl) { return (cl & 3) == 0; }
DI int unit_row(int U, int rank) { return U < 32 ? rank * M + (31 - U) * 16 : ((rank + (U >> 4) - 1) & 7) * M + (U & 15) * 16; }

template <bool G> DI void mfma_role(LAS char* lds, int wid, int lane, int b, char* myws, char* obase, int rank, int u0, char* yws) {
  using RG = Ring<G>;
  constexpr int NI = RG::NI;
  volatile LAS unsigned* misc = (volatile LAS unsigned*)lds;
  const int wm = G ? 0 : wid >> 1, wn = G ? wid : wid & 1;
  const int fr = lane & 15, g = lane >> 4, q = (lane & 15) >> 2, pp = lane & 3;
  const int xo0 = (wm * 128 + fr) * 128 + ((g ^ (fr >> 1)) << 4);
  const int fk = q | ((g & 1) << 2);
  const int wo0 = RG::W0 + RG::WROW * (8 * g + q) + ((wn >> 1) * 256) + ((((wn & 1) * 4) ^ fk) << 5) + 8 * pp;
  f32x4 acc[NI][4];
#pragma unroll
  for (int i = 0; i < NI; ++i)
#pragma unroll
    for (int j = 0; j < 4; ++j) acc[i][j] = (f32x4){0.f, 0.f, 0.f, 0.f};
  Frag<NI> f0, f1;
  Addr a = {lds + xo0, lds + (xo0 ^ 64), {lds + wo0, lds + (wo0 ^ 32), lds + (wo0 ^ 64), lds + (wo0 ^ 96)}, 0, 0, RG::XS, RG::WS};
  SB0();
  BAR();
  rd_all<G>(f0, a, std::make_integer_sequence<int, 8 + NI>{});
  __builtin_amdgcn_s_waitcnt(0xC07F);
  for (int s = 0; s < NT; ++s) {
    SB0();
    kstep<G, 1>(acc, f0, f1, a, std::make_integer_sequence<int, 4 * NI>{});
    BAR();
    SB0();
    kstep<G, 0>(acc, f1, f0, a, std::make_integer_sequence<int, 4 * NI>{});
  }
  SB0();
  float m = 0.f, m2 = 0.f;
#pragma unroll
  for (int i = 0; i < NI; ++i)
#pragma unroll
    for (int j = 0; j < 4; ++j) {
      m = fmaxf(fmaxf(m, __builtin_fabsf(acc[i][j][0])), __builtin_fabsf(acc[i][j][1]));
      m2 = fmaxf(fmaxf(m2, __builtin_fabsf(acc[i][j][2])), __builtin_fabsf(acc[i][j][3]));
    }
  const unsigned mu = wave_umax(__float_as_uint(fmaxf(m, m2)));
  if (lane == 0) misc[L_MAX + wid] = mu;
  LBAR();
  auto finish = [&](bool bad, float pre, auto val) {
    LBAR();
    if (wid == 0) STAMP(3);
    const unsigned gm = misc[L_GMAX];
    bad = bad || gm == 0xffffffffu;
    const float amax = __uint_as_float(gm);
    const float inv = amax > 0.f ? pre * (127.f / amax) : 0.f;
    char* orow = obase + (size_t)(wm * 128 + fr) * NS + wn * 64 + 16 * g;
#pragma unroll
    for (int i = 0; i < NI; ++i) {
      unsigned d4[4];
#pragma unroll
      for (int j = 0; j < 4; ++j) {
        unsigned a0 = __float_as_uint(__builtin_fmaf(val(i, j, 0), inv, 12582912.f)), a1 = __float_as_uint(__builtin_fmaf(val(i, j, 1), inv, 12582912.f));
        unsigned a2 = __float_as_uint(__builtin_fmaf(val(i, j, 2), inv, 12582912.f)), a3 = __float_as_uint(__builtin_fmaf(val(i, j, 3), inv, 12582912.f));
        d4[j] = __builtin_amdgcn_perm(a1, a0, 0x0c0c0400) | __builtin_amdgcn_perm(a3, a2, 0x04000c0c);
      }
      auto r0 = __builtin_amdgcn_permlane32_swap(d4[0], d4[2], false, false); d4[0] = r0[0]; d4[2] = r0[1];
      auto r1 = __builtin_amdgcn_permlane32_swap(d4[1], d4[3], false, false); d4[1] = r1[0]; d4[3] = r1[1];
      auto r2 = __builtin_amdgcn_permlane16_swap(d4[0], d4[1], false, false); d4[0] = r2[0]; d4[1] = r2[1];
      auto r3 = __builtin_amdgcn_permlane16_swap(d4[2], d4[3], false, false); d4[2] = r3[0]; d4[3] = r3[1];
      u32x4 v = {d4[0], d4[1], d4[2], d4[3]};
      if (bad) v = (u32x4){0x80808080u, 0x80808080u, 0x80808080u, 0x80808080u};
      const int r16 = G ? unit_row(u0 + i, rank) : i * 16;
      asm volatile("global_store_dwordx4 %0, %1, off sc0 sc1\n\ts_nop 1" ::"v"(orow + (size_t)r16 * NS), "v"(v) : "memory");
    }
    if (wid == 0) STAMP(4);
    WAITV(0);
    LBAR();
    if (wid == 0) STAMP(5);
  };
  if (G || !yws) {
    finish(false, 1.f, [&](int i, int j, int e) { return acc[i][j][e]; });
    return;
  }
  const unsigned call = call_number(myws, b);
  const size_t slot = (size_t)(call & 1) * (7 * 16 * 4) + u0 * 4 + wid;
  const float rs = mu ? 1.f / __uint_as_float(mu) : 0.f;
  char* yp = yws + WS_YST + slot * 16384 + lane * 16;
#pragma unroll
  for (int i = 0; i < NI; ++i)
#pragma unroll
    for (int h = 0; h < 2; ++h) {
      u32x4 v;
#pragma unroll
      for (int e = 0; e < 4; ++e) v[e] = __builtin_bit_cast(unsigned, __builtin_amdgcn_cvt_pknorm_i16(acc[i][2 * h + (e >> 1)][2 * (e & 1)] * rs, acc[i][2 * h + (e >> 1)][2 * (e & 1) + 1] * rs));
      asm volatile("s_waitcnt vmcnt(2)\n\tglobal_store_dwordx4 %0, %1, off sc0 sc1\n\ts_nop 1" ::"v"(yp + (i * 2 + h) * 1024), "v"(v) : "memory");
    }
  WAITV(0);
  if (lane == 0) __hip_atomic_store((u64*)(yws + WS_YFLG + slot * 128), ((u64)call << 32) | mu, __ATOMIC_RELAXED, __HIP_MEMORY_SCOPE_SYSTEM);
  if (wid == 0) STAMP(7);
  const u64 t0 = __builtin_amdgcn_s_memrealtime();
  bool bad = false;
  u64 f;
  for (unsigned np = 1;; ++np) {
    f = __hip_atomic_load((u64*)(myws + WS_YFLG + slot * 128), __ATOMIC_RELAXED, __HIP_MEMORY_SCOPE_SYSTEM);
    if (__all((unsigned)(f >> 32) == call)) break;
    if ((np & 63u) == 0 && __any(timed_out((unsigned*)(myws + WS_TMO), call, t0))) { bad = true; break; }
    __builtin_amdgcn_s_sleep(4);
  }
  u32x4 v[16];
  const char* yq = myws + WS_YST + slot * 16384 + lane * 16;
#define YL(n, a, o) "global_load_dwordx4 %" #n ", %" #a ", off offset:" #o " sc0 sc1\n\t"
  asm volatile(YL(0, 16, 0) YL(1, 16, 1024) YL(2, 16, 2048) YL(3, 16, 3072) YL(4, 17, 0) YL(5, 17, 1024) YL(6, 17, 2048) YL(7, 17, 3072)
               YL(8, 18, 0) YL(9, 18, 1024) YL(10, 18, 2048) YL(11, 18, 3072) YL(12, 19, 0) YL(13, 19, 1024) YL(14, 19, 2048) YL(15, 19, 3072) "s_waitcnt vmcnt(0)"
               : "=&v"(v[0]), "=&v"(v[1]), "=&v"(v[2]), "=&v"(v[3]), "=&v"(v[4]), "=&v"(v[5]), "=&v"(v[6]), "=&v"(v[7]),
                 "=&v"(v[8]), "=&v"(v[9]), "=&v"(v[10]), "=&v"(v[11]), "=&v"(v[12]), "=&v"(v[13]), "=&v"(v[14]), "=&v"(v[15])
               : "v"(yq), "v"(yq + 4096), "v"(yq + 8192), "v"(yq + 12288) : "memory");
  if (wid == 0) STAMP(8);
  finish(bad, __uint_as_float((unsigned)f) * (1.f / 32767.f), [&](int i, int j, int e) {
    const unsigned q2 = v[(i * 2 + (j >> 1)) & 15][(j & 1) * 2 + (e >> 1)];
    return (e & 1) ? (float)((int)q2 >> 16) : (float)(short)(q2 & 0xffffu); });
}

__global__ __launch_bounds__(512, 2) void gemm_a2a(const Params p) {
  extern __shared__ __attribute__((aligned(16))) char smem[];
  LAS char* lds = (LAS char*)smem;
  volatile LAS unsigned* misc = (volatile LAS unsigned*)lds;
  const int tid = threadIdx.x, lane = tid & 63;
  const int wid = __builtin_amdgcn_readfirstlane(tid >> 6);
  const int b = blockIdx.x, xcd = b & 7, idx = b >> 3;
  const int rank = p.rank & 7;
  char* myws = p.ws[rank];
  unsigned* cnt = (unsigned*)(myws + WS_CNT + b * 128);
  unsigned* tmo = (unsigned*)(myws + WS_TMO);
  if (wid == 0) STAMP(0);
  auto rx = __builtin_amdgcn_make_buffer_rsrc((void*)p.x, 0, M * K * 2, 0x00020000);
  const unsigned xv0 = (unsigned)((lane >> 3) * K * 2 + (((lane & 7) ^ (lane >> 4)) << 4)), xv1 = (xv0 ^ 64) + 8 * K * 2;

  auto aggregate = [&](unsigned call) {
      const char* gp = myws + WS_GRAN + lane * 16;
      const bool skip = lane >= 56;
      unsigned gm = 0xffffffffu;
      const u64 t0 = __builtin_amdgcn_s_memrealtime();
      for (unsigned np = 1;; ++np) {
        u32x4 a, c;
        asm volatile("global_load_dwordx4 %0, %2, off sc1\n\tglobal_load_dwordx4 %1, %2, off offset:1024 sc1\n\ts_waitcnt vmcnt(0)"
                     : "=&v"(a), "=&v"(c) : "v"(gp) : "memory");
        if (__all(a[1] == call && a[3] == call && (skip || (c[1] == call && c[3] == call)))) {
          gm = wave_umax(skip ? max(a[0], a[2]) : max(max(a[0], a[2]), max(c[0], c[2])));
          break;
        }
        if ((np & 63u) == 0 && __any(timed_out(tmo, call, t0))) break;
      }
      __hip_atomic_store((u64*)(p.ws[lane & 7] + WS_GMAX + ((call & 1) * 8 + (lane >> 3)) * 128 + (SOLO ? lane & 7 : rank) * 8), ((u64)call << 32) | gm, __ATOMIC_RELAXED, __HIP_MEMORY_SCOPE_SYSTEM);
      STAMP(8);
    };

  const int x4 = xcd >> 1;
  const bool isG = xcd & 1;
  if (!isG && idx >= 28) {
    const int un = x4 * 4 + (idx & 3);
    auto stage = [&](int kt) { DMA(rx, lds + kt * 2048, xv0, un * UB + kt * 128, 0); DMA(rx, lds + kt * 2048 + 1024, xv1, un * UB + kt * 128, 0); };
    stage(wid);
    const int R = max(wid - 1, 0) * NU + un;
    const int grt = (32 + R) / 9;
    char* pw = p.ws[(rank - wid) & 7];
    char* dp = pw + WS_XST + (size_t)R * UB + lane * 16;
    char* fa = pw + WS_XFLG + (size_t)R * 128;
    unsigned c0;
    asm volatile("global_load_dword %0, %1, off sc1\n\ts_waitcnt vmcnt(0)" : "=&v"(c0) : "v"(cnt) : "memory");
    const unsigned call = __builtin_amdgcn_readfirstlane(c0) + 1u;
    unsigned fv = call * 64u - 1u;
    LAS char* la = lds + lane * 16;
#pragma unroll 1
    for (int r = 0; r < 8; ++r) {
      BAR();
      if (wid == 0) {
        if (r < 7) stage(8 * (r + 1));
        WAITV(0);
        continue;
      }
      u32x4 a0 = *(LAS u32x4*)(la + gperm(r * 8, grt) * 2048), a1 = *(LAS u32x4*)(la + gperm(r * 8, grt) * 2048 + 1024);
#pragma unroll
      for (int j = 0; j < 8; ++j) {
        u32x4 n0 = a0, n1 = a1;
        if (j < 7) { n0 = *(LAS u32x4*)(la + gperm(r * 8 + j + 1, grt) * 2048); n1 = *(LAS u32x4*)(la + gperm(r * 8 + j + 1, grt) * 2048 + 1024); }
        asm volatile("global_store_dwordx4 %0, %1, off sc0 sc1\n\t"
                     "global_store_dwordx4 %0, %2, off offset:1024 sc0 sc1\n\t"
                     "s_waitcnt vmcnt(3)\n\t"
                     "s_mov_b64 exec, 1\n\tglobal_store_dword %3, %4, off offset:%5 sc0 sc1\n\ts_mov_b64 exec, -1\n\ts_nop 0"
                     ::"v"(dp), "v"(a0), "v"(a1), "v"(fa), "v"(fv), "n"((j & 3) * 4) : "memory");
        fv += 1; dp += 2048;
        if (j == 2 && r < 7) stage(8 * (r + 1) + wid);
        a0 = n0; a1 = n1;
      }
    }
    WAITV(0);
    if (lane == 0 && wid) __hip_atomic_store((unsigned*)fa, call * 64u + 63u, __ATOMIC_RELAXED, __HIP_MEMORY_SCOPE_SYSTEM);
    if (wid == 1) STAMP(7);
    const int fi = un * 8 + wid;
    if (!SOLO && lane == 0 && (fi >> 4) != rank && !early_tile(fi & 15)) {
      unsigned* fl = (unsigned*)(myws + WS_DONE + (size_t)fi * 128);
      const u64 t1 = __builtin_amdgcn_s_memrealtime();
      unsigned np = 0;
      while (__hip_atomic_load(fl, __ATOMIC_RELAXED, __HIP_MEMORY_SCOPE_SYSTEM) < call) {
        if ((++np & 63u) == 0 && timed_out(tmo, call, t1)) { p.out[rank][(size_t)rank * M * NS + fi] = (char)-128; break; }
        __builtin_amdgcn_s_sleep(2);
      }
    }
    if (tid == 0) __hip_atomic_store(cnt, call, __ATOMIC_RELAXED, __HIP_MEMORY_SCOPE_AGENT);
    if (wid == 1) STAMP(6);
    return;
  }

  const int rt = (x4 & 1) * 8 + (idx & 7), dj = idx >> 2;
  const int cl = isG ? (x4 >> 1) * 4 + (idx >> 3) : x4 * 4 + (idx & 3);
  const int dst = (rank + 1 + dj) & 7;
  const bool early = !isG && early_tile(cl);
  if (wid < 4) {
    __builtin_amdgcn_s_setprio(2);
    if (isG) mfma_role<true>(lds, wid, lane, b, myws, p.out[rank] + cl * 256, rank, rt * 9, nullptr);
    else mfma_role<false>(lds, wid, lane, b, myws, (early ? p.out[rank] + (size_t)(((rank - 1 - dj) & 7) * M + GR) * NS : p.out[dst] + (size_t)(rank * M + GR) * NS) + cl * 128,
                          rank, dj * 16 + cl, early ? p.ws[dst] : nullptr);
    return;
  }
  const int w = wid - 4;
  unsigned call = 0;
  auto rw = __builtin_amdgcn_make_buffer_rsrc((void*)p.w, 0, K * N * 2, 0x00020000);
  if (!isG) {
    using RG = Ring<false>;
    const unsigned v0 = (unsigned)(((lane >> 4) + w * 16) * N * 2 + (dst * 16 + cl) * 256 + (((lane & 15) ^ ((lane >> 4) << 1)) << 4)), v1 = (v0 ^ 128) + 8 * N * 2;
    auto ldx = [&](int t, int sx) {
#pragma unroll
      for (int i = 0; i < 4; ++i) {
        DMA(rx, lds + sx * RG::XS + (w * 4 + i) * 2048, xv0, (NU + w * 4 + i) * UB + t * 128, 0);
        DMA(rx, lds + sx * RG::XS + (w * 4 + i) * 2048 + 1024, xv1, (NU + w * 4 + i) * UB + t * 128, 0);
      }
    };
    auto ldw = [&](int t) {
#pragma unroll
      for (int j = 0; j < 4; ++j)
        DMA(rw, lds + RG::W0 + (t & 3) * RG::WS + (w * 4 + j) * 1024, (j & 2) ? v1 : v0, (t * 64 + (j & 1) * 4) * N * 2, 2);
    };
    const bool xpf = w == 3 && (cl & 3) != 1;
    const int xq = dj * 3 + max((cl & 3) - 1, 0);
    unsigned t0_ = 0;
    const int al = (int)((size_t)p.w >> 8) + dst * 16 + cl;
    const int pend_ = w != 0 || (al & 3) == 1 || (al & 31) == 31 ? 0 : early ? LT2E : LT2;
    u64 tp = __builtin_amdgcn_s_memrealtime() + 200 + LT1 * 69;
    int sx = 0;
    for (int t = 0; t < NT; t += 2) {
      ldx(t, sx); ldw(t); ldw(t + 1);
      sx = sx == 2 ? 0 : sx + 1;
      if (xpf) {
        const unsigned n = ((t >> 1) * 21 + xq) * 64 + lane, kt = (n >> 8) + 3;
        asm volatile("buffer_load_dword %0, %1, %2, 0 offen" : "+v"(t0_) : "v"(kt < NT ? (GR + (n & 255)) * K * 2 + kt * 128 : 0xfffff000u), "s"(rx) : "memory");
        WAITV(17);
      } else WAITV(16);
      if (t) BAR();
      ldx(t + 1, sx);
      sx = sx == 2 ? 0 : sx + 1;
      if (t >= LT1 && t < pend_) { while ((long long)(__builtin_amdgcn_s_memrealtime() - tp) < 0) __builtin_amdgcn_s_sleep(1); tp += 2 * LPACE; }
      if (xpf) WAITV(13); else WAITV(12);
      BAR();
    }
    asm volatile("s_waitcnt vmcnt(0)" : "+v"(t0_)::"memory");
    BAR();
    BAR();
  } else if (w == 0 || w == 3) {
    using RG = Ring<true>;
    const int i0 = w ? 5 : 0, u0 = rt * 9 + i0;
    const bool own = u0 < 32;
    auto rg = __builtin_amdgcn_make_buffer_rsrc((void*)(myws + WS_XST), 0, (NR - 1) * NU * UB, 0x00020000);
    const unsigned gv = lane * 16, gv1 = gv + 1024;
    const unsigned* fp = (const unsigned*)(myws + WS_XFLG + (size_t)(own ? 0 : u0 - 32 + min(lane >> 2, w ? 3 : 4)) * 128) + (lane & 3);
    LAS unsigned* pl = (LAS unsigned*)(lds + GPOLL + (w ? 256 : 0));
    unsigned rdy = own ? NT : 0, np = 0;
    if (!own) call = call_number(myws, b);
    const u64 t0 = __builtin_amdgcn_s_memrealtime();
    auto seen = [&](unsigned f) {
      int n = (int)(f - call * 64u) + 1;
      n = max(n, __builtin_amdgcn_update_dpp(0, n, 0xB1, 0xf, 0xf, false));
      n = max(n, __builtin_amdgcn_update_dpp(0, n, 0x4E, 0xf, 0xf, false));
      int m = __builtin_amdgcn_readlane(n, 0);
#pragma unroll
      for (int i = 1; i < 5; ++i) m = min(m, __builtin_amdgcn_readlane(n, 4 * i));
      rdy = max((int)rdy, m);
    };
    int so = i0 * 2048, pend = 0;
    for (int t = 0; t < NT + 2; ++t) {
      if (t < NT) {
        if (own) {
          const unsigned ox = (31 - u0) * UB + gperm(t, rt) * 128;
#pragma unroll
          for (int i = 0; i < 5; ++i)
            if (i < 4 || w == 0) { DMA(rx, lds + so + i * 2048, xv0, ox - i * UB, 0); DMA(rx, lds + so + i * 2048 + 1024, xv1, ox - i * UB, 0); }
        } else {
          if (pend && rdy <= (unsigned)t) {
            if (w == 0) WAITV(10); else WAITV(8);
            seen(*(volatile LAS unsigned*)(pl + lane)); pend = 0;
          }
          while (rdy <= (unsigned)t) {
            seen(__hip_atomic_load(fp, __ATOMIC_RELAXED, __HIP_MEMORY_SCOPE_SYSTEM));
            if (rdy <= (unsigned)t && (++np & 63u) == 0 && __any(timed_out(tmo, call, t0))) rdy = NT;
          }
          if (rdy < NT && rdy < (unsigned)t + 5 && pend == 0) {
            __builtin_amdgcn_global_load_lds(fp, pl, 4, 0, 17);
            pend = 2;
          }
          const unsigned og = (u0 - 32) * UB + t * 2048;
#pragma unroll
          for (int i = 0; i < 5; ++i)
            if (i < 4 || w == 0) { DMA(rg, lds + so + i * 2048, gv, og + i * UB, 17); DMA(rg, lds + so + i * 2048 + 1024, gv1, og + i * UB, 17); }
        }
        so = so >= 2 * RG::XS ? so - 2 * RG::XS : so + RG::XS;
        if (w == 0) { if (pend == 2) WAITV(11); else WAITV(10); } else { if (pend == 2) WAITV(9); else WAITV(8); }
        if (pend && --pend == 0) seen(*(volatile LAS unsigned*)(pl + lane));
      } else WAITV(0);
      if (t >= 1) BAR();
    }
  } else {
    using RG = Ring<true>;
    const unsigned v0 = (unsigned)(((lane >> 5) + (w - 1) * 32) * N * 2 + (rank * 8 + cl) * 512 + (((lane & 31) ^ ((lane >> 5) << 1)) << 4));
    const unsigned v1 = (v0 ^ 64) + 2 * N * 2, v2 = (v0 ^ 128) + 8 * N * 2, v3 = (v0 ^ 192) + 10 * N * 2;
    int so = 0;
    for (int t = 0; t < NT + 2; ++t) {
      if (t < NT) {
        const int kt = gperm(t, rt);
#pragma unroll
        for (int j = 0; j < 16; ++j)
          DMA(rw, lds + RG::W0 + so + (w - 1) * 16384 + j * 1024, (j & 4) ? ((j & 1) ? v3 : v2) : ((j & 1) ? v1 : v0), (kt * 64 + ((j & 2) << 1) + ((j & 8) << 1)) * N * 2, 0);
        so = so == 2 * RG::WS ? 0 : so + RG::WS;
        WAITV(16);
      } else WAITV(0);
      if (t >= 1) BAR();
    }
  }
  if (w == 0 && call == 0) call = call_number(myws, b);
  LBAR();
  if (b == 1 && w) {
    if (w > 1) __builtin_amdgcn_s_sleep(6);
    if (w > 2) __builtin_amdgcn_s_sleep(6);
    aggregate(call_number(myws, b));
  }
  if (w == 0) {
    const u32x4 mv = *(volatile LAS u32x4*)(misc + L_MAX);
    const unsigned m = max(max(mv[0], mv[1]), max(mv[2], mv[3]));
    const int gslot = isG ? rt * 8 + cl : 128 + dj * 16 + cl;
    if (lane == 0) __hip_atomic_store((u64*)(myws + WS_GRAN + gslot * 8), ((u64)call << 32) | m, __ATOMIC_RELAXED, __HIP_MEMORY_SCOPE_AGENT);
    const char* gp = myws + WS_GMAX + ((call & 1) * 8 + xcd) * 128 + (lane & 7) * 8;
    unsigned gm = 0xffffffffu;
    const u64 t0 = __builtin_amdgcn_s_memrealtime();
    for (;;) {
      u64 gq; unsigned tm;
      asm volatile("global_load_dwordx2 %0, %2, off sc0 sc1\n\tglobal_load_dword %1, %3, off sc1\n\ts_waitcnt vmcnt(0)"
                   : "=&v"(gq), "=&v"(tm) : "v"(gp), "v"(tmo) : "memory");
      if (__any(tm == call)) break;
      if (__all((unsigned)(gq >> 32) == call)) { gm = wave_umax((unsigned)gq); break; }
      if (__builtin_amdgcn_s_memrealtime() - t0 > DEADLINE) { __hip_atomic_store(tmo, call, __ATOMIC_RELAXED, __HIP_MEMORY_SCOPE_AGENT); break; }
      __builtin_amdgcn_s_sleep(1);
    }
    if (lane == 0) misc[L_GMAX] = gm;
  }
  LBAR();
  LBAR();
  if (w == 0 && lane == 0) {
    if (!isG && !early)
      __hip_atomic_store((unsigned*)(p.ws[dst] + WS_DONE + (size_t)(rank * 16 + cl) * 128), call, __ATOMIC_RELAXED, __HIP_MEMORY_SCOPE_SYSTEM);
    __hip_atomic_store(cnt, call, __ATOMIC_RELAXED, __HIP_MEMORY_SCOPE_AGENT);
    STAMP(6);
  }
}

extern "C" void kernel_launch(int rank, int nranks, ncclComm_t comm, hipStream_t stream,
                              void* const* d_in, const int* in_sizes, int n_in,
                              void* d_out, int out_size, const int* in_dtypes, int out_dtype,
                              void* const* d_out_all, void* const* d_ws_all, size_t ws_bytes) {
  if (nranks != NR || (unsigned)rank >= (unsigned)NR || n_in < 2 || ws_bytes < WS_END || in_sizes[0] != M * K || in_sizes[1] != K * N || out_size != NR * M * NS) {
    fprintf(stderr, "unexpected problem shape\n");
    return;
  }
  static bool init = false;
  if (!init) {
    HIP_CHECK(hipFuncSetAttribute((const void*)gemm_a2a, hipFuncAttributeMaxDynamicSharedMemorySize, LDS_BYTES));
    init = true;
  }
  Params p;
  p.x = d_in[0]; p.w = d_in[1];
  for (int i = 0; i < NR; ++i) { p.out[i] = (char*)d_out_all[i]; p.ws[i] = (char*)d_ws_all[i]; }
  p.rank = rank; p.pad = 0;
  gemm_a2a<<<NWG, 512, LDS_BYTES, stream>>>(p);
  HIP_CHECK(hipGetLastError());
}
```

```cpp
#include <hip/hip_runtime.h>
#include <rccl/rccl.h>
#include <cstdio>
#include <utility>

#define HIP_CHECK(expr) do { hipError_t _e = (expr); if (_e != hipSuccess) { \
    fprintf(stderr, "HIP error %s at %s:%d\n", hipGetErrorString(_e), __FILE__, __LINE__); \
    return; } } while (0)

typedef __bf16 bf16x4 __attribute__((ext_vector_type(4)));
typedef __bf16 bf16x8 __attribute__((ext_vector_type(8)));
typedef float f32x4 __attribute__((ext_vector_type(4)));
typedef unsigned u32x4 __attribute__((ext_vector_type(4)));
typedef unsigned long long u64;
#define LAS __attribute__((address_space(3)))
#define DI __device__ __forceinline__

#define LPACE 116
#define LT1 4
#define LT2 30
#define LT2E 20
#define SOLO 0

constexpr int NR = 8;
constexpr int M = 512, N = 16384, K = 4096, NT = K / 64;
constexpr int NS = N / NR;
constexpr int NWG = 256;
constexpr int NU = 16, GR = NU * 16;
constexpr int UB = NT * 2048;
constexpr int LDS_BYTES = 163840;
template <bool G> struct Ring;
template <> struct Ring<true> { static constexpr int NI = 9, WROW = 512, XS = 144 * 128, WS = 64 * 512, NW = 3, W0 = 3 * XS; };
template <> struct Ring<false> { static constexpr int NI = 8, WROW = 256, XS = 256 * 128, WS = 64 * 256, NW = 4, W0 = 3 * XS; };
constexpr int GPOLL = Ring<true>::W0 + 3 * Ring<true>::WS;
static_assert(GPOLL + 512 <= LDS_BYTES && Ring<false>::W0 + 4 * Ring<false>::WS <= LDS_BYTES && UB <= LDS_BYTES, "LDS");
constexpr int L_MAX = 0, L_GMAX = 4;
constexpr size_t WS_CNT = 0;
constexpr size_t WS_TMO = WS_CNT + NWG * 128;
constexpr size_t WS_GRAN = WS_TMO + 1024;
constexpr size_t WS_GMAX = WS_GRAN + NWG * 8;
constexpr size_t WS_DONE = WS_GMAX + 2 * 8 * 128;
constexpr size_t WS_XFLG = WS_DONE + NR * 16 * 128;
constexpr size_t WS_YFLG = 1u << 20;
constexpr size_t WS_XST = 2u << 20;
constexpr size_t WS_YST = 16u << 20;
constexpr size_t WS_END = WS_YST + (size_t)2 * 7 * 16 * 65536;
static_assert(WS_XFLG + (NR - 1) * NU * 128 <= WS_YFLG && WS_YFLG + 2 * 7 * 16 * 4 * 128 <= WS_XST && WS_XST + (size_t)(NR - 1) * NU * UB <= WS_YST, "workspace map");
#define STAMP(i) do {} while (0)
constexpr u64 DEADLINE = 30000000ull;

struct Params {
  const void* x; const void* w;
  char* out[NR]; char* ws[NR];
  int rank; int pad;
};

#define WAITV(n) asm volatile("s_waitcnt vmcnt(%0)" ::"n"(n) : "memory")
#define BAR() asm volatile("s_barrier" ::: "memory")
#define LBAR() asm volatile("s_waitcnt lgkmcnt(0)\n\ts_barrier" ::: "memory")
#define SB0() __builtin_amdgcn_sched_barrier(0)
#define DMA(rs, dst, voff, soff, aux) __builtin_amdgcn_raw_ptr_buffer_load_lds(rs, (LAS void*)(dst), 16, voff, soff, 0, aux)

template <int NI> struct Frag { bf16x8 x[NI]; bf16x4 wl[4], wh[4]; };
struct Addr { LAS char* x0; LAS char* x1; LAS char* w[4]; int sx, sw, dx, dw; };

template <bool G, int KS, int R> DI void rd(Frag<Ring<G>::NI>& f, const Addr& a) {
  if constexpr (R < 8) {
    const bf16x4 v = __builtin_amdgcn_ds_read_tr16_b64_v4bf16((LAS bf16x4*)(a.w[R >> 1] + Ring<G>::WROW * (32 * KS + 4 * (R & 1))));
    if constexpr (R & 1) f.wh[R >> 1] = v; else f.wl[R >> 1] = v;
  } else if constexpr (R < 8 + Ring<G>::NI) f.x[R - 8] = *(LAS bf16x8*)((KS ? a.x1 : a.x0) + 2048 * (R - 8));
}
template <bool G, int Q, int KN> DI void gap(f32x4 (&acc)[Ring<G>::NI][4], const Frag<Ring<G>::NI>& fc, Frag<Ring<G>::NI>& fn, Addr& a) {
  using RG = Ring<G>;
  constexpr int I = Q >> 2, J = (I & 1) ? 3 - (Q & 3) : Q & 3;
  acc[I][J] = __builtin_amdgcn_mfma_f32_16x16x32_bf16(__builtin_shufflevector(fc.wl[J], fc.wh[J], 0, 1, 2, 3, 4, 5, 6, 7), fc.x[I], acc[I][J], 0, 0, 0);
  SB0();
  if constexpr (Q % 3 == 0) rd<G, KN, 2 * (Q / 3)>(fn, a);
  if constexpr (Q % 3 == 2) rd<G, KN, 2 * (Q / 3) + 1>(fn, a);
  if constexpr (KN == 1) {
    if constexpr (Q == 13) { a.w[0] += a.dw; asm volatile("" : "+v"(a.w[0])); }
    if constexpr (Q == 16) { a.w[1] += a.dw; asm volatile("" : "+v"(a.w[1])); }
    if constexpr (Q == 19) { a.w[2] += a.dw; asm volatile("" : "+v"(a.w[2])); }
    if constexpr (Q == 22) { a.w[3] += a.dw; asm volatile("" : "+v"(a.w[3])); }
    if constexpr (Q == 25) { a.x0 += a.dx; asm volatile("" : "+v"(a.x0)); }
    if constexpr (Q == 26) { a.x1 += a.dx; asm volatile("" : "+v"(a.x1)); }
  } else {
    if constexpr (Q == 25) { a.sx = a.sx == 2 ? 0 : a.sx + 1; asm volatile("" : "+s"(a.sx)); }
    if constexpr (Q == 26) { a.dx = a.sx == 2 ? -2 * RG::XS : RG::XS; asm volatile("" : "+s"(a.dx)); }
    if constexpr (Q == 27) { a.sw = a.sw == RG::NW - 1 ? 0 : a.sw + 1; asm volatile("" : "+s"(a.sw)); }
    if constexpr (Q == 28) { a.dw = a.sw == RG::NW - 1 ? -(RG::NW - 1) * RG::WS : RG::WS; asm volatile("" : "+s"(a.dw)); }
  }
  if constexpr (Q == 4 * RG::NI - 2) __builtin_amdgcn_s_waitcnt(0xC07F);
  SB0();
}
template <bool G, int KN, int... Qs> DI void kstep(f32x4 (&acc)[Ring<G>::NI][4], const Frag<Ring<G>::NI>& fc, Frag<Ring<G>::NI>& fn, Addr& a, std::integer_sequence<int, Qs...>) {
  (gap<G, Qs, KN>(acc, fc, fn, a), ...);
}
template <bool G, int... Rs> DI void rd_all(Frag<Ring<G>::NI>& f, const Addr& a, std::integer_sequence<int, Rs...>) { (rd<G, 0, Rs>(f, a), ...); }

DI unsigned wave_umax(unsigned v) {
  v = max(v, (unsigned)__builtin_amdgcn_update_dpp(0, (int)v, 0xB1, 0xf, 0xf, false));
  v = max(v, (unsigned)__builtin_amdgcn_update_dpp(0, (int)v, 0x4E, 0xf, 0xf, false));
  v = max(v, (unsigned)__builtin_amdgcn_update_dpp(0, (int)v, 0x141, 0xf, 0xf, false));
  v = max(v, (unsigned)__builtin_amdgcn_update_dpp(0, (int)v, 0x140, 0xf, 0xf, false));
  return max(max((unsigned)__builtin_amdgcn_readlane((int)v, 0), (unsigned)__builtin_amdgcn_readlane((int)v, 16)),
             max((unsigned)__builtin_amdgcn_readlane((int)v, 32), (unsigned)__builtin_amdgcn_readlane((int)v, 48)));
}
DI bool timed_out(unsigned* tmo, unsigned call, u64 t0) {
  if (__hip_atomic_load(tmo, __ATOMIC_RELAXED, __HIP_MEMORY_SCOPE_AGENT) == call || __builtin_amdgcn_s_memrealtime() - t0 > DEADLINE) {
    __hip_atomic_store(tmo, call, __ATOMIC_RELAXED, __HIP_MEMORY_SCOPE_AGENT);
    return true;
  }
  return false;
}
DI unsigned call_number(const char* myws, int b) {
  return __builtin_amdgcn_readfirstlane(__hip_atomic_load((const unsigned*)(myws + WS_CNT + b * 128), __ATOMIC_RELAXED, __HIP_MEMORY_SCOPE_AGENT)) + 1u;
}
DI int gperm(int u, int rt) { return (u & ~7) | ((u + rt) & 7); }
DI bool early_tile(int cl) { return (cl & 3) == 0; }
DI int unit_row(int U, int rank) { return U < 32 ? rank * M + (31 - U) * 16 : ((rank + (U >> 4) - 1) & 7) * M + (U & 15) * 16; }

template <bool G> DI void mfma_role(LAS char* lds, int wid, int lane, int b, char* myws, char* obase, int rank, int u0, char* yws) {
  using RG = Ring<G>;
  constexpr int NI = RG::NI;
  volatile LAS unsigned* misc = (volatile LAS unsigned*)lds;
  const int wm = G ? 0 : wid >> 1, wn = G ? wid : wid & 1;
  const int fr = lane & 15, g = lane >> 4, q = (lane & 15) >> 2, pp = lane & 3;
  const int xo0 = (wm * 128 + fr) * 128 + ((g ^ (fr >> 1)) << 4);
  const int fk = q | ((g & 1) << 2);
  const int wo0 = RG::W0 + RG::WROW * (8 * g + q) + ((wn >> 1) * 256) + ((((wn & 1) * 4) ^ fk) << 5) + 8 * pp;
  f32x4 acc[NI][4];
#pragma unroll
  for (int i = 0; i < NI; ++i)
#pragma unroll
    for (int j = 0; j < 4; ++j) acc[i][j] = (f32x4){0.f, 0.f, 0.f, 0.f};
  Frag<NI> f0, f1;
  Addr a = {lds + xo0, lds + (xo0 ^ 64), {lds + wo0, lds + (wo0 ^ 32), lds + (wo0 ^ 64), lds + (wo0 ^ 96)}, 0, 0, RG::XS, RG::WS};
  SB0();
  BAR();
  rd_all<G>(f0, a, std::make_integer_sequence<int, 8 + NI>{});
  __builtin_amdgcn_s_waitcnt(0xC07F);
  for (int s = 0; s < NT; ++s) {
    SB0();
    kstep<G, 1>(acc, f0, f1, a, std::make_integer_sequence<int, 4 * NI>{});
    BAR();
    SB0();
    kstep<G, 0>(acc, f1, f0, a, std::make_integer_sequence<int, 4 * NI>{});
  }
  SB0();
  float m = 0.f, m2 = 0.f;
#pragma unroll
  for (int i = 0; i < NI; ++i)
#pragma unroll
    for (int j = 0; j < 4; ++j) {
      m = fmaxf(fmaxf(m, __builtin_fabsf(acc[i][j][0])), __builtin_fabsf(acc[i][j][1]));
      m2 = fmaxf(fmaxf(m2, __builtin_fabsf(acc[i][j][2])), __builtin_fabsf(acc[i][j][3]));
    }
  const unsigned mu = wave_umax(__float_as_uint(fmaxf(m, m2)));
  if (lane == 0) misc[L_MAX + wid] = mu;
  LBAR();
  auto finish = [&](bool bad, float pre, auto val) {
    LBAR();
    if (wid == 0) STAMP(3);
    const unsigned gm = misc[L_GMAX];
    bad = bad || gm == 0xffffffffu;
    const float amax = __uint_as_float(gm);
    const float inv = amax > 0.f ? pre * (127.f / amax) : 0.f;
    char* orow = obase + (size_t)(wm * 128 + fr) * NS + wn * 64 + 16 * g;
#pragma unroll
    for (int i = 0; i < NI; ++i) {
      unsigned d4[4];
#pragma unroll
      for (int j = 0; j < 4; ++j) {
        unsigned a0 = __float_as_uint(__builtin_fmaf(val(i, j, 0), inv, 12582912.f)), a1 = __float_as_uint(__builtin_fmaf(val(i, j, 1), inv, 12582912.f));
        unsigned a2 = __float_as_uint(__builtin_fmaf(val(i, j, 2), inv, 12582912.f)), a3 = __float_as_uint(__builtin_fmaf(val(i, j, 3), inv, 12582912.f));
        d4[j] = __builtin_amdgcn_perm(a1, a0, 0x0c0c0400) | __builtin_amdgcn_perm(a3, a2, 0x04000c0c);
      }
      auto r0 = __builtin_amdgcn_permlane32_swap(d4[0], d4[2], false, false); d4[0] = r0[0]; d4[2] = r0[1];
      auto r1 = __builtin_amdgcn_permlane32_swap(d4[1], d4[3], false, false); d4[1] = r1[0]; d4[3] = r1[1];
      auto r2 = __builtin_amdgcn_permlane16_swap(d4[0], d4[1], false, false); d4[0] = r2[0]; d4[1] = r2[1];
      auto r3 = __builtin_amdgcn_permlane16_swap(d4[2], d4[3], false, false); d4[2] = r3[0]; d4[3] = r3[1];
      u32x4 v = {d4[0], d4[1], d4[2], d4[3]};
      if (bad) v = (u32x4){0x80808080u, 0x80808080u, 0x80808080u, 0x80808080u};
      const int r16 = G ? unit_row(u0 + i, rank) : i * 16;
      asm volatile("global_store_dwordx4 %0, %1, off sc0 sc1\n\ts_nop 1" ::"v"(orow + (size_t)r16 * NS), "v"(v) : "memory");
    }
    if (wid == 0) STAMP(4);
    WAITV(0);
    LBAR();
    if (wid == 0) STAMP(5);
  };
  if (G || !yws) {
    finish(false, 1.f, [&](int i, int j, int e) { return acc[i][j][e]; });
    return;
  }
  const unsigned call = call_number(myws, b);
  const size_t slot = (size_t)(call & 1) * (7 * 16 * 4) + u0 * 4 + wid;
  const float rs = mu ? 1.f / __uint_as_float(mu) : 0.f;
  char* yp = yws + WS_YST + slot * 16384 + lane * 16;
#pragma unroll
  for (int i = 0; i < NI; ++i)
#pragma unroll
    for (int h = 0; h < 2; ++h) {
      u32x4 v;
#pragma unroll
      for (int e = 0; e < 4; ++e) v[e] = __builtin_bit_cast(unsigned, __builtin_amdgcn_cvt_pknorm_i16(acc[i][2 * h + (e >> 1)][2 * (e & 1)] * rs, acc[i][2 * h + (e >> 1)][2 * (e & 1) + 1] * rs));
      asm volatile("s_waitcnt vmcnt(2)\n\tglobal_store_dwordx4 %0, %1, off sc0 sc1\n\ts_nop 1" ::"v"(yp + (i * 2 + h) * 1024), "v"(v) : "memory");
    }
  WAITV(0);
  if (lane == 0) __hip_atomic_store((u64*)(yws + WS_YFLG + slot * 128), ((u64)call << 32) | mu, __ATOMIC_RELAXED, __HIP_MEMORY_SCOPE_SYSTEM);
  if (wid == 0) STAMP(7);
  const u64 t0 = __builtin_amdgcn_s_memrealtime();
  bool bad = false;
  u64 f;
  for (unsigned np = 1;; ++np) {
    f = __hip_atomic_load((u64*)(myws + WS_YFLG + slot * 128), __ATOMIC_RELAXED, __HIP_MEMORY_SCOPE_SYSTEM);
    if (__all((unsigned)(f >> 32) == call)) break;
    if ((np & 63u) == 0 && __any(timed_out((unsigned*)(myws + WS_TMO), call, t0))) { bad = true; break; }
    __builtin_amdgcn_s_sleep(4);
  }
  u32x4 v[16];
  const char* yq = myws + WS_YST + slot * 16384 + lane * 16;
#define YL(n, a, o) "global_load_dwordx4 %" #n ", %" #a ", off offset:" #o " sc0 sc1\n\t"
  asm volatile(YL(0, 16, 0) YL(1, 16, 1024) YL(2, 16, 2048) YL(3, 16, 3072) YL(4, 17, 0) YL(5, 17, 1024) YL(6, 17, 2048) YL(7, 17, 3072)
               YL(8, 18, 0) YL(9, 18, 1024) YL(10, 18, 2048) YL(11, 18, 3072) YL(12, 19, 0) YL(13, 19, 1024) YL(14, 19, 2048) YL(15, 19, 3072) "s_waitcnt vmcnt(0)"
               : "=&v"(v[0]), "=&v"(v[1]), "=&v"(v[2]), "=&v"(v[3]), "=&v"(v[4]), "=&v"(v[5]), "=&v"(v[6]), "=&v"(v[7]),
                 "=&v"(v[8]), "=&v"(v[9]), "=&v"(v[10]), "=&v"(v[11]), "=&v"(v[12]), "=&v"(v[13]), "=&v"(v[14]), "=&v"(v[15])
               : "v"(yq), "v"(yq + 4096), "v"(yq + 8192), "v"(yq + 12288) : "memory");
  if (wid == 0) STAMP(8);
  finish(bad, __uint_as_float((unsigned)f) * (1.f / 32767.f), [&](int i, int j, int e) {
    const unsigned q2 = v[(i * 2 + (j >> 1)) & 15][(j & 1) * 2 + (e >> 1)];
    return (e & 1) ? (float)((int)q2 >> 16) : (float)(short)(q2 & 0xffffu); });
}

__global__ __launch_bounds__(512, 2) void gemm_a2a(const Params p) {
  extern __shared__ __attribute__((aligned(16))) char smem[];
  LAS char* lds = (LAS char*)smem;
  volatile LAS unsigned* misc = (volatile LAS unsigned*)lds;
  const int tid = threadIdx.x, lane = tid & 63;
  const int wid = __builtin_amdgcn_readfirstlane(tid >> 6);
  const int b = blockIdx.x, xcd = b & 7, idx = b >> 3;
  const int rank = p.rank & 7;
  char* myws = p.ws[rank];
  unsigned* cnt = (unsigned*)(myws + WS_CNT + b * 128);
  unsigned* tmo = (unsigned*)(myws + WS_TMO);
  if (wid == 0) STAMP(0);
  auto rx = __builtin_amdgcn_make_buffer_rsrc((void*)p.x, 0, M * K * 2, 0x00020000);
  const unsigned xv0 = (unsigned)((lane >> 3) * K * 2 + (((lane & 7) ^ (lane >> 4)) << 4)), xv1 = (xv0 ^ 64) + 8 * K * 2;

  auto aggregate = [&](unsigned call) {
      const char* gp = myws + WS_GRAN + lane * 16;
      const bool skip = lane >= 56;
      unsigned gm = 0xffffffffu;
      const u64 t0 = __builtin_amdgcn_s_memrealtime();
      for (unsigned np = 1;; ++np) {
        u32x4 a, c;
        asm volatile("global_load_dwordx4 %0, %2, off sc1\n\tglobal_load_dwordx4 %1, %2, off offset:1024 sc1\n\ts_waitcnt vmcnt(0)"
                     : "=&v"(a), "=&v"(c) : "v"(gp) : "memory");
        if (__all(a[1] == call && a[3] == call && (skip || (c[1] == call && c[3] == call)))) {
          gm = wave_umax(skip ? max(a[0], a[2]) : max(max(a[0], a[2]), max(c[0], c[2])));
          break;
        }
        if ((np & 63u) == 0 && __any(timed_out(tmo, call, t0))) break;
      }
      __hip_atomic_store((u64*)(p.ws[lane & 7] + WS_GMAX + ((call & 1) * 8 + (lane >> 3)) * 128 + (SOLO ? lane & 7 : rank) * 8), ((u64)call << 32) | gm, __ATOMIC_RELAXED, __HIP_MEMORY_SCOPE_SYSTEM);
      STAMP(8);
    };

  const int x4 = xcd >> 1;
  const bool isG = xcd & 1;
  if (!isG && idx >= 28) {
    const int un = x4 * 4 + (idx & 3);
    auto stage = [&](int kt) { DMA(rx, lds + kt * 2048, xv0, un * UB + kt * 128, 0); DMA(rx, lds + kt * 2048 + 1024, xv1, un * UB + kt * 128, 0); };
    stage(wid);
    const int R = max(wid - 1, 0) * NU + un;
    const int grt = (32 + R) / 9;
    char* pw = p.ws[(rank - wid) & 7];
    char* dp = pw + WS_XST + (size_t)R * UB + lane * 16;
    char* fa = pw + WS_XFLG + (size_t)R * 128;
    unsigned c0;
    asm volatile("global_load_dword %0, %1, off sc1\n\ts_waitcnt vmcnt(0)" : "=&v"(c0) : "v"(cnt) : "memory");
    const unsigned call = __builtin_amdgcn_readfirstlane(c0) + 1u;
    unsigned fv = call * 64u - 1u;
    LAS char* la = lds + lane * 16;
#pragma unroll 1
    for (int r = 0; r < 8; ++r) {
      BAR();
      if (wid == 0) {
        if (r < 7) stage(8 * (r + 1));
        WAITV(0);
        continue;
      }
      u32x4 a0 = *(LAS u32x4*)(la + gperm(r * 8, grt) * 2048), a1 = *(LAS u32x4*)(la + gperm(r * 8, grt) * 2048 + 1024);
#pragma unroll
      for (int j = 0; j < 8; ++j) {
        u32x4 n0 = a0, n1 = a1;
        if (j < 7) { n0 = *(LAS u32x4*)(la + gperm(r * 8 + j + 1, grt) * 2048); n1 = *(LAS u32x4*)(la + gperm(r * 8 + j + 1, grt) * 2048 + 1024); }
        asm volatile("global_store_dwordx4 %0, %1, off sc0 sc1\n\t"
                     "global_store_dwordx4 %0, %2, off offset:1024 sc0 sc1\n\t"
                     "s_waitcnt vmcnt(3)\n\t"
                     "s_mov_b64 exec, 1\n\tglobal_store_dword %3, %4, off offset:%5 sc0 sc1\n\ts_mov_b64 exec, -1\n\ts_nop 0"
                     ::"v"(dp), "v"(a0), "v"(a1), "v"(fa), "v"(fv), "n"((j & 3) * 4) : "memory");
        fv += 1; dp += 2048;
        if (j == 2 && r < 7) stage(8 * (r + 1) + wid);
        a0 = n0; a1 = n1;
      }
    }
    WAITV(0);
    if (lane == 0 && wid) __hip_atomic_store((unsigned*)fa, call * 64u + 63u, __ATOMIC_RELAXED, __HIP_MEMORY_SCOPE_SYSTEM);
    if (wid == 1) STAMP(7);
    const int fi = un * 8 + wid;
    if (!SOLO && lane == 0 && (fi >> 4) != rank && !early_tile(fi & 15)) {
      unsigned* fl = (unsigned*)(myws + WS_DONE + (size_t)fi * 128);
      const u64 t1 = __builtin_amdgcn_s_memrealtime();
      unsigned np = 0;
      while (__hip_atomic_load(fl, __ATOMIC_RELAXED, __HIP_MEMORY_SCOPE_SYSTEM) < call) {
        if ((++np & 63u) == 0 && timed_out(tmo, call, t1)) { p.out[rank][(size_t)rank * M * NS + fi] = (char)-128; break; }
        __builtin_amdgcn_s_sleep(2);
      }
    }
    if (tid == 0) __hip_atomic_store(cnt, call, __ATOMIC_RELAXED, __HIP_MEMORY_SCOPE_AGENT);
    if (wid == 1) STAMP(6);
    return;
  }

  const int rt = (x4 & 1) * 8 + (idx & 7), dj = idx >> 2;
  const int cl = isG ? (x4 >> 1) * 4 + (idx >> 3) : x4 * 4 + (idx & 3);
  const int dst = (rank + 1 + dj) & 7;
  const bool early = !isG && early_tile(cl);
  if (wid < 4) {
    __builtin_amdgcn_s_setprio(2);
    if (isG) mfma_role<true>(lds, wid, lane, b, myws, p.out[rank] + cl * 256, rank, rt * 9, nullptr);
    else mfma_role<false>(lds, wid, lane, b, myws, (early ? p.out[rank] + (size_t)(((rank - 1 - dj) & 7) * M + GR) * NS : p.out[dst] + (size_t)(rank * M + GR) * NS) + cl * 128,
                          rank, dj * 16 + cl, early ? p.ws[dst] : nullptr);
    return;
  }
  const int w = wid - 4;
  unsigned call = 0;
  auto rw = __builtin_amdgcn_make_buffer_rsrc((void*)p.w, 0, K * N * 2, 0x00020000);
  if (!isG) {
    using RG = Ring<false>;
    const unsigned v0 = (unsigned)(((lane >> 4) + w * 16) * N * 2 + (dst * 16 + cl) * 256 + (((lane & 15) ^ ((lane >> 4) << 1)) << 4)), v1 = (v0 ^ 128) + 8 * N * 2;
    auto ldx = [&](int t, int sx) {
#pragma unroll
      for (int i = 0; i < 4; ++i) {
        DMA(rx, lds + sx * RG::XS + (w * 4 + i) * 2048, xv0, (NU + w * 4 + i) * UB + t * 128, 0);
        DMA(rx, lds + sx * RG::XS + (w * 4 + i) * 2048 + 1024, xv1, (NU + w * 4 + i) * UB + t * 128, 0);
      }
    };
    auto ldw = [&](int t) {
#pragma unroll
      for (int j = 0; j < 4; ++j)
        DMA(rw, lds + RG::W0 + (t & 3) * RG::WS + (w * 4 + j) * 1024, (j & 2) ? v1 : v0, (t * 64 + (j & 1) * 4) * N * 2, 2);
    };
    const int al = (int)((size_t)p.w >> 8) + dst * 16 + cl;
    const int pend_ = w != 0 || (al & 3) == 1 || (al & 31) == 31 ? 0 : early ? LT2E : LT2;
    const bool xpf = (cl & 3) == 2 ? w == 3 || (w == 2 && (al & 31) == 30) : w == 3 && (cl & 3) != 1 && (al & 31) != 31;
    const int xq = dj * 3 + max((cl & 3) - 1, 0) + (w == 2);
    unsigned t0_ = 0;
    u64 tp = __builtin_amdgcn_s_memrealtime() + 200 + LT1 * 69;
    int sx = 0;
    for (int t = 0; t < NT; t += 2) {
      ldx(t, sx); ldw(t); ldw(t + 1);
      sx = sx == 2 ? 0 : sx + 1;
      if (xpf) {
        const unsigned n = ((t >> 1) * 21 + xq) * 64 + lane, kt = (n >> 8) + 3;
        asm volatile("buffer_load_dword %0, %1, %2, 0 offen" : "+v"(t0_) : "v"(kt < NT ? (GR + (n & 255)) * K * 2 + kt * 128 : 0xfffff000u), "s"(rx) : "memory");
        WAITV(17);
      } else WAITV(16);
      if (t) BAR();
      ldx(t + 1, sx);
      sx = sx == 2 ? 0 : sx + 1;
      if (t >= LT1 && t < pend_) { while ((long long)(__builtin_amdgcn_s_memrealtime() - tp) < 0) __builtin_amdgcn_s_sleep(1); tp += 2 * LPACE; }
      if (xpf) WAITV(13); else WAITV(12);
      BAR();
    }
    asm volatile("s_waitcnt vmcnt(0)" : "+v"(t0_)::"memory");
    BAR();
    BAR();
  } else if (w == 0 || w == 3) {
    using RG = Ring<true>;
    const int i0 = w ? 5 : 0, u0 = rt * 9 + i0;
    int so = i0 * 2048;
    if (u0 < 32) {
      for (int t = 0; t < NT + 2; ++t) {
        if (t < NT) {
          const unsigned ox = (31 - u0) * UB + gperm(t, rt) * 128;
#pragma unroll
          for (int i = 0; i < 5; ++i)
            if (i < 4 || w == 0) { DMA(rx, lds + so + i * 2048, xv0, ox - i * UB, 0); DMA(rx, lds + so + i * 2048 + 1024, xv1, ox - i * UB, 0); }
          so = so >= 2 * RG::XS ? so - 2 * RG::XS : so + RG::XS;
          if (w == 0) WAITV(10); else WAITV(8);
        } else WAITV(0);
        if (t >= 1) BAR();
      }
    } else {
      auto rg = __builtin_amdgcn_make_buffer_rsrc((void*)(myws + WS_XST), 0, (NR - 1) * NU * UB, 0x00020000);
      const unsigned gv = lane * 16, gv1 = gv + 1024;
      const unsigned* fp = (const unsigned*)(myws + WS_XFLG + (size_t)(u0 - 32 + min(lane >> 2, w ? 3 : 4)) * 128) + (lane & 3);
      LAS unsigned* pl = (LAS unsigned*)(lds + GPOLL + (w ? 256 : 0));
      unsigned rdy = 0, np = 0;
      call = call_number(myws, b);
      const u64 t0 = __builtin_amdgcn_s_memrealtime();
      auto seen = [&](unsigned f) {
        int n = (int)(f - call * 64u) + 1;
        n = max(n, __builtin_amdgcn_update_dpp(0, n, 0xB1, 0xf, 0xf, false));
        n = max(n, __builtin_amdgcn_update_dpp(0, n, 0x4E, 0xf, 0xf, false));
        int m = __builtin_amdgcn_readlane(n, 0);
#pragma unroll
        for (int i = 1; i < 5; ++i) m = min(m, __builtin_amdgcn_readlane(n, 4 * i));
        rdy = max((int)rdy, m);
      };
      int nx = 0;
      auto issue = [&]() {
        const unsigned og = (u0 - 32) * UB + nx * 2048;
#pragma unroll
        for (int i = 0; i < 5; ++i)
          if (i < 4 || w == 0) { DMA(rg, lds + so + i * 2048, gv, og + i * UB, 17); DMA(rg, lds + so + i * 2048 + 1024, gv1, og + i * UB, 17); }
        so = so >= 2 * RG::XS ? so - 2 * RG::XS : so + RG::XS;
        ++nx;
      };
      bool pend = false;
      for (int t = 1; t <= NT; ++t) {
        if (pend && rdy <= (unsigned)t) { WAITV(0); seen(*(volatile LAS unsigned*)(pl + lane)); pend = false; }
        if (nx < t) {
          while (rdy < (unsigned)t) {
            seen(__hip_atomic_load(fp, __ATOMIC_RELAXED, __HIP_MEMORY_SCOPE_SYSTEM));
            if (rdy < (unsigned)t && (++np & 63u) == 0 && __any(timed_out(tmo, call, t0))) rdy = NT;
          }
          issue();
        }
        const bool opt = t < NT && rdy > (unsigned)t;
        if (opt) issue();
        const bool poll = rdy < NT && rdy < (unsigned)t + 5 && !pend;
        if (poll) __builtin_amdgcn_global_load_lds(fp, pl, 4, 0, 17);
        if (!opt) { if (poll) WAITV(1); else WAITV(0); }
        else if (w == 0) { if (poll) WAITV(11); else WAITV(10); }
        else { if (poll) WAITV(9); else WAITV(8); }
        if (pend) seen(*(volatile LAS unsigned*)(pl + lane));
        pend = poll;
        BAR();
      }
      WAITV(0);
      BAR();
    }
  } else {
    using RG = Ring<true>;
    const unsigned v0 = (unsigned)(((lane >> 5) + (w - 1) * 32) * N * 2 + (rank * 8 + cl) * 512 + (((lane & 31) ^ ((lane >> 5) << 1)) << 4));
    const unsigned v1 = (v0 ^ 64) + 2 * N * 2, v2 = (v0 ^ 128) + 8 * N * 2, v3 = (v0 ^ 192) + 10 * N * 2;
    int so = 0;
    for (int t = 0; t < NT + 2; ++t) {
      if (t < NT) {
        const int kt = gperm(t, rt);
#pragma unroll
        for (int j = 0; j < 16; ++j)
          DMA(rw, lds + RG::W0 + so + (w - 1) * 16384 + j * 1024, (j & 4) ? ((j & 1) ? v3 : v2) : ((j & 1) ? v1 : v0), (kt * 64 + ((j & 2) << 1) + ((j & 8) << 1)) * N * 2, 0);
        so = so == 2 * RG::WS ? 0 : so + RG::WS;
        WAITV(16);
      } else WAITV(0);
      if (t >= 1) BAR();
    }
  }
  if (w == 0 && call == 0) call = call_number(myws, b);
  LBAR();
  if (b == 1 && w) {
    if (w > 1) __builtin_amdgcn_s_sleep(6);
    if (w > 2) __builtin_amdgcn_s_sleep(6);
    aggregate(call_number(myws, b));
  }
  if (w == 0) {
    const u32x4 mv = *(volatile LAS u32x4*)(misc + L_MAX);
    const unsigned m = max(max(mv[0], mv[1]), max(mv[2], mv[3]));
    const int gslot = isG ? rt * 8 + cl : 128 + dj * 16 + cl;
    if (lane == 0) __hip_atomic_store((u64*)(myws + WS_GRAN + gslot * 8), ((u64)call << 32) | m, __ATOMIC_RELAXED, __HIP_MEMORY_SCOPE_AGENT);
    const char* gp = myws + WS_GMAX + ((call & 1) * 8 + xcd) * 128 + (lane & 7) * 8;
    unsigned gm = 0xffffffffu;
    const u64 t0 = __builtin_amdgcn_s_memrealtime();
    for (;;) {
      u64 gq; unsigned tm;
      asm volatile("global_load_dwordx2 %0, %2, off sc0 sc1\n\tglobal_load_dword %1, %3, off sc1\n\ts_waitcnt vmcnt(0)"
                   : "=&v"(gq), "=&v"(tm) : "v"(gp), "v"(tmo) : "memory");
      if (__any(tm == call)) break;
      if (__all((unsigned)(gq >> 32) == call)) { gm = wave_umax((unsigned)gq); break; }
      if (__builtin_amdgcn_s_memrealtime() - t0 > DEADLINE) { __hip_atomic_store(tmo, call, __ATOMIC_RELAXED, __HIP_MEMORY_SCOPE_AGENT); break; }
      __builtin_amdgcn_s_sleep(1);
    }
    if (lane == 0) misc[L_GMAX] = gm;
  }
  LBAR();
  LBAR();
  if (w == 0 && lane == 0) {
    if (!isG && !early)
      __hip_atomic_store((unsigned*)(p.ws[dst] + WS_DONE + (size_t)(rank * 16 + cl) * 128), call, __ATOMIC_RELAXED, __HIP_MEMORY_SCOPE_SYSTEM);
    __hip_atomic_store(cnt, call, __ATOMIC_RELAXED, __HIP_MEMORY_SCOPE_AGENT);
    STAMP(6);
  }
}

extern "C" void kernel_launch(int rank, int nranks, ncclComm_t comm, hipStream_t stream,
                              void* const* d_in, const int* in_sizes, int n_in,
                              void* d_out, int out_size, const int* in_dtypes, int out_dtype,
                              void* const* d_out_all, void* const* d_ws_all, size_t ws_bytes) {
  if (nranks != NR || (unsigned)rank >= (unsigned)NR || n_in < 2 || ws_bytes < WS_END || in_sizes[0] != M * K || in_sizes[1] != K * N || out_size != NR * M * NS) {
    fprintf(stderr, "unexpected problem shape\n");
    return;
  }
  static bool init = false;
  if (!init) {
    HIP_CHECK(hipFuncSetAttribute((const void*)gemm_a2a, hipFuncAttributeMaxDynamicSharedMemorySize, LDS_BYTES));
    init = true;
  }
  Params p;
  p.x = d_in[0]; p.w = d_in[1];
  for (int i = 0; i < NR; ++i) { p.out[i] = (char*)d_out_all[i]; p.ws[i] = (char*)d_ws_all[i]; }
  p.rank = rank; p.pad = 0;
  gemm_a2a<<<NWG, 512, LDS_BYTES, stream>>>(p);
  HIP_CHECK(hipGetLastError());
}
```

```cpp
#include <hip/hip_runtime.h>
#include <rccl/rccl.h>
#include <cstdio>
#include <utility>

#define HIP_CHECK(expr) do { hipError_t _e = (expr); if (_e != hipSuccess) { \
    fprintf(stderr, "HIP error %s at %s:%d\n", hipGetErrorString(_e), __FILE__, __LINE__); \
    return; } } while (0)

typedef __bf16 bf16x4 __attribute__((ext_vector_type(4)));
typedef __bf16 bf16x8 __attribute__((ext_vector_type(8)));
typedef float f32x4 __attribute__((ext_vector_type(4)));
typedef unsigned u32x4 __attribute__((ext_vector_type(4)));
typedef unsigned long long u64;
#define LAS __attribute__((address_space(3)))
#define DI __device__ __forceinline__

constexpr int LPACE = 116, LT1 = 4, LT2 = 30, LT2E = 20, BP = 66, SP = 84;
#define SOLO 0

constexpr int NR = 8;
constexpr int M = 512, N = 16384, K = 4096, NT = K / 64;
constexpr int NS = N / NR;
constexpr int NWG = 256;
constexpr int NU = 16, GR = NU * 16;
constexpr int UB = NT * 2048;
constexpr int LDS_BYTES = 163840;
template <bool G> struct Ring;
template <> struct Ring<true> { static constexpr int NI = 9, WROW = 512, XS = 144 * 128, WS = 64 * 512, NW = 3, W0 = 3 * XS; };
template <> struct Ring<false> { static constexpr int NI = 8, WROW = 256, XS = 256 * 128, WS = 64 * 256, NW = 4, W0 = 3 * XS; };
constexpr int GPOLL = Ring<true>::W0 + 3 * Ring<true>::WS;
static_assert(GPOLL + 512 <= LDS_BYTES && Ring<false>::W0 + 4 * Ring<false>::WS <= LDS_BYTES && UB <= LDS_BYTES, "LDS");
constexpr int L_MAX = 0, L_GMAX = 4;
constexpr size_t WS_CNT = 0;
constexpr size_t WS_TMO = WS_CNT + NWG * 128;
constexpr size_t WS_T0 = WS_TMO + 512;
constexpr size_t WS_GRAN = WS_TMO + 1024;
constexpr size_t WS_GMAX = WS_GRAN + NWG * 8;
constexpr size_t WS_DONE = WS_GMAX + 2 * 8 * 128;
constexpr size_t WS_XFLG = WS_DONE + NR * 16 * 128;
constexpr size_t WS_YFLG = 1u << 20;
constexpr size_t WS_XST = 2u << 20;
constexpr size_t WS_YST = 16u << 20;
constexpr size_t WS_END = WS_YST + (size_t)2 * 7 * 16 * 65536;
static_assert(WS_XFLG + (NR - 1) * NU * 128 <= WS_YFLG && WS_YFLG + 2 * 7 * 16 * 4 * 128 <= WS_XST && WS_XST + (size_t)(NR - 1) * NU * UB <= WS_YST, "workspace map");
#define STAMP(i) do {} while (0)
constexpr u64 DEADLINE = 30000000ull;

struct Params {
  const void* x; const void* w;
  char* out[NR]; char* ws[NR];
  int rank; int pad;
};

#define WAITV(n) asm volatile("s_waitcnt vmcnt(%0)" ::"n"(n) : "memory")
#define BAR() asm volatile("s_barrier" ::: "memory")
#define LBAR() asm volatile("s_waitcnt lgkmcnt(0)\n\ts_barrier" ::: "memory")
#define SB0() __builtin_amdgcn_sched_barrier(0)
#define DMA(rs, dst, voff, soff, aux) __builtin_amdgcn_raw_ptr_buffer_load_lds(rs, (LAS void*)(dst), 16, voff, soff, 0, aux)

template <int NI> struct Frag { bf16x8 x[NI]; bf16x4 wl[4], wh[4]; };
struct Addr { LAS char* x0; LAS char* x1; LAS char* w[4]; int sx, sw, dx, dw; };

template <bool G, int KS, int R> DI void rd(Frag<Ring<G>::NI>& f, const Addr& a) {
  if constexpr (R < 8) {
    const bf16x4 v = __builtin_amdgcn_ds_read_tr16_b64_v4bf16((LAS bf16x4*)(a.w[R >> 1] + Ring<G>::WROW * (32 * KS + 4 * (R & 1))));
    if constexpr (R & 1) f.wh[R >> 1] = v; else f.wl[R >> 1] = v;
  } else if constexpr (R < 8 + Ring<G>::NI) f.x[R - 8] = *(LAS bf16x8*)((KS ? a.x1 : a.x0) + 2048 * (R - 8));
}
template <bool G, int Q, int KN> DI void gap(f32x4 (&acc)[Ring<G>::NI][4], const Frag<Ring<G>::NI>& fc, Frag<Ring<G>::NI>& fn, Addr& a) {
  using RG = Ring<G>;
  constexpr int I = Q >> 2, J = (I & 1) ? 3 - (Q & 3) : Q & 3;
  acc[I][J] = __builtin_amdgcn_mfma_f32_16x16x32_bf16(__builtin_shufflevector(fc.wl[J], fc.wh[J], 0, 1, 2, 3, 4, 5, 6, 7), fc.x[I], acc[I][J], 0, 0, 0);
  SB0();
  if constexpr (Q % 3 == 0) rd<G, KN, 2 * (Q / 3)>(fn, a);
  if constexpr (Q % 3 == 2) rd<G, KN, 2 * (Q / 3) + 1>(fn, a);
  if constexpr (KN == 1) {
    if constexpr (Q == 13) { a.w[0] += a.dw; asm volatile("" : "+v"(a.w[0])); }
    if constexpr (Q == 16) { a.w[1] += a.dw; asm volatile("" : "+v"(a.w[1])); }
    if constexpr (Q == 19) { a.w[2] += a.dw; asm volatile("" : "+v"(a.w[2])); }
    if constexpr (Q == 22) { a.w[3] += a.dw; asm volatile("" : "+v"(a.w[3])); }
    if constexpr (Q == 25) { a.x0 += a.dx; asm volatile("" : "+v"(a.x0)); }
    if constexpr (Q == 26) { a.x1 += a.dx; asm volatile("" : "+v"(a.x1)); }
  } else {
    if constexpr (Q == 25) { a.sx = a.sx == 2 ? 0 : a.sx + 1; asm volatile("" : "+s"(a.sx)); }
    if constexpr (Q == 26) { a.dx = a.sx == 2 ? -2 * RG::XS : RG::XS; asm volatile("" : "+s"(a.dx)); }
    if constexpr (Q == 27) { a.sw = a.sw == RG::NW - 1 ? 0 : a.sw + 1; asm volatile("" : "+s"(a.sw)); }
    if constexpr (Q == 28) { a.dw = a.sw == RG::NW - 1 ? -(RG::NW - 1) * RG::WS : RG::WS; asm volatile("" : "+s"(a.dw)); }
  }
  if constexpr (Q == 4 * RG::NI - 2) __builtin_amdgcn_s_waitcnt(0xC07F);
  SB0();
}
template <bool G, int KN, int... Qs> DI void kstep(f32x4 (&acc)[Ring<G>::NI][4], const Frag<Ring<G>::NI>& fc, Frag<Ring<G>::NI>& fn, Addr& a, std::integer_sequence<int, Qs...>) {
  (gap<G, Qs, KN>(acc, fc, fn, a), ...);
}
template <bool G, int... Rs> DI void rd_all(Frag<Ring<G>::NI>& f, const Addr& a, std::integer_sequence<int, Rs...>) { (rd<G, 0, Rs>(f, a), ...); }

DI unsigned wave_umax(unsigned v) {
  v = max(v, (unsigned)__builtin_amdgcn_update_dpp(0, (int)v, 0xB1, 0xf, 0xf, false));
  v = max(v, (unsigned)__builtin_amdgcn_update_dpp(0, (int)v, 0x4E, 0xf, 0xf, false));
  v = max(v, (unsigned)__builtin_amdgcn_update_dpp(0, (int)v, 0x141, 0xf, 0xf, false));
  v = max(v, (unsigned)__builtin_amdgcn_update_dpp(0, (int)v, 0x140, 0xf, 0xf, false));
  return max(max((unsigned)__builtin_amdgcn_readlane((int)v, 0), (unsigned)__builtin_amdgcn_readlane((int)v, 16)),
             max((unsigned)__builtin_amdgcn_readlane((int)v, 32), (unsigned)__builtin_amdgcn_readlane((int)v, 48)));
}
DI bool timed_out(unsigned* tmo, unsigned call, u64 t0) {
  if (__hip_atomic_load(tmo, __ATOMIC_RELAXED, __HIP_MEMORY_SCOPE_AGENT) == call || __builtin_amdgcn_s_memrealtime() - t0 > DEADLINE) {
    __hip_atomic_store(tmo, call, __ATOMIC_RELAXED, __HIP_MEMORY_SCOPE_AGENT);
    return true;
  }
  return false;
}
DI unsigned call_number(const char* myws, int b) {
  return __builtin_amdgcn_readfirstlane(__hip_atomic_load((const unsigned*)(myws + WS_CNT + b * 128), __ATOMIC_RELAXED, __HIP_MEMORY_SCOPE_AGENT)) + 1u;
}
DI int gperm(int u, int rt) { return (u & ~7) | ((u + rt) & 7); }
DI bool early_tile(int cl) { return (cl & 3) == 0; }
DI int unit_row(int U, int rank) { return U < 32 ? rank * M + (31 - U) * 16 : ((rank + (U >> 4) - 1) & 7) * M + (U & 15) * 16; }

template <bool G> DI void mfma_role(LAS char* lds, int wid, int lane, int b, char* myws, char* obase, int rank, int u0, char* yws) {
  using RG = Ring<G>;
  constexpr int NI = RG::NI;
  volatile LAS unsigned* misc = (volatile LAS unsigned*)lds;
  const int wm = G ? 0 : wid >> 1, wn = G ? wid : wid & 1;
  const int fr = lane & 15, g = lane >> 4, q = (lane & 15) >> 2, pp = lane & 3;
  const int xo0 = (wm * 128 + fr) * 128 + ((g ^ (fr >> 1)) << 4);
  const int fk = q | ((g & 1) << 2);
  const int wo0 = RG::W0 + RG::WROW * (8 * g + q) + ((wn >> 1) * 256) + ((((wn & 1) * 4) ^ fk) << 5) + 8 * pp;
  f32x4 acc[NI][4];
#pragma unroll
  for (int i = 0; i < NI; ++i)
#pragma unroll
    for (int j = 0; j < 4; ++j) acc[i][j] = (f32x4){0.f, 0.f, 0.f, 0.f};
  Frag<NI> f0, f1;
  Addr a = {lds + xo0, lds + (xo0 ^ 64), {lds + wo0, lds + (wo0 ^ 32), lds + (wo0 ^ 64), lds + (wo0 ^ 96)}, 0, 0, RG::XS, RG::WS};
  SB0();
  BAR();
  rd_all<G>(f0, a, std::make_integer_sequence<int, 8 + NI>{});
  __builtin_amdgcn_s_waitcnt(0xC07F);
  for (int s = 0; s < NT; ++s) {
    SB0();
    kstep<G, 1>(acc, f0, f1, a, std::make_integer_sequence<int, 4 * NI>{});
    BAR();
    SB0();
    kstep<G, 0>(acc, f1, f0, a, std::make_integer_sequence<int, 4 * NI>{});
  }
  SB0();
  float m = 0.f, m2 = 0.f;
#pragma unroll
  for (int i = 0; i < NI; ++i)
#pragma unroll
    for (int j = 0; j < 4; ++j) {
      m = fmaxf(fmaxf(m, __builtin_fabsf(acc[i][j][0])), __builtin_fabsf(acc[i][j][1]));
      m2 = fmaxf(fmaxf(m2, __builtin_fabsf(acc[i][j][2])), __builtin_fabsf(acc[i][j][3]));
    }
  const unsigned mu = wave_umax(__float_as_uint(fmaxf(m, m2)));
  if (lane == 0) misc[L_MAX + wid] = mu;
  LBAR();
  auto finish = [&](bool bad, float pre, auto val) {
    LBAR();
    if (wid == 0) STAMP(3);
    const unsigned gm = misc[L_GMAX];
    bad = bad || gm == 0xffffffffu;
    const float amax = __uint_as_float(gm);
    const float inv = amax > 0.f ? pre * (127.f / amax) : 0.f;
    char* orow = obase + (size_t)(wm * 128 + fr) * NS + wn * 64 + 16 * g;
#pragma unroll
    for (int i = 0; i < NI; ++i) {
      unsigned d4[4];
#pragma unroll
      for (int j = 0; j < 4; ++j) {
        unsigned a0 = __float_as_uint(__builtin_fmaf(val(i, j, 0), inv, 12582912.f)), a1 = __float_as_uint(__builtin_fmaf(val(i, j, 1), inv, 12582912.f));
        unsigned a2 = __float_as_uint(__builtin_fmaf(val(i, j, 2), inv, 12582912.f)), a3 = __float_as_uint(__builtin_fmaf(val(i, j, 3), inv, 12582912.f));
        d4[j] = __builtin_amdgcn_perm(a1, a0, 0x0c0c0400) | __builtin_amdgcn_perm(a3, a2, 0x04000c0c);
      }
      auto r0 = __builtin_amdgcn_permlane32_swap(d4[0], d4[2], false, false); d4[0] = r0[0]; d4[2] = r0[1];
      auto r1 = __builtin_amdgcn_permlane32_swap(d4[1], d4[3], false, false); d4[1] = r1[0]; d4[3] = r1[1];
      auto r2 = __builtin_amdgcn_permlane16_swap(d4[0], d4[1], false, false); d4[0] = r2[0]; d4[1] = r2[1];
      auto r3 = __builtin_amdgcn_permlane16_swap(d4[2], d4[3], false, false); d4[2] = r3[0]; d4[3] = r3[1];
      u32x4 v = {d4[0], d4[1], d4[2], d4[3]};
      if (bad) v = (u32x4){0x80808080u, 0x80808080u, 0x80808080u, 0x80808080u};
      const int r16 = G ? unit_row(u0 + i, rank) : i * 16;
      asm volatile("global_store_dwordx4 %0, %1, off sc0 sc1\n\ts_nop 1" ::"v"(orow + (size_t)r16 * NS), "v"(v) : "memory");
    }
    if (wid == 0) STAMP(4);
    WAITV(0);
    LBAR();
    if (wid == 0) STAMP(5);
  };
  if (G || !yws) {
    finish(false, 1.f, [&](int i, int j, int e) { return acc[i][j][e]; });
    return;
  }
  const unsigned call = call_number(myws, b);
  const size_t slot = (size_t)(call & 1) * (7 * 16 * 4) + u0 * 4 + wid;
  const float rs = mu ? 1.f / __uint_as_float(mu) : 0.f;
  char* yp = yws + WS_YST + slot * 16384 + lane * 16;
#pragma unroll
  for (int i = 0; i < NI; ++i)
#pragma unroll
    for (int h = 0; h < 2; ++h) {
      u32x4 v;
#pragma unroll
      for (int e = 0; e < 4; ++e) v[e] = __builtin_bit_cast(unsigned, __builtin_amdgcn_cvt_pknorm_i16(acc[i][2 * h + (e >> 1)][2 * (e & 1)] * rs, acc[i][2 * h + (e >> 1)][2 * (e & 1) + 1] * rs));
      asm volatile("s_waitcnt vmcnt(2)\n\tglobal_store_dwordx4 %0, %1, off sc0 sc1\n\ts_nop 1" ::"v"(yp + (i * 2 + h) * 1024), "v"(v) : "memory");
    }
  WAITV(0);
  if (lane == 0) __hip_atomic_store((u64*)(yws + WS_YFLG + slot * 128), ((u64)call << 32) | mu, __ATOMIC_RELAXED, __HIP_MEMORY_SCOPE_SYSTEM);
  if (wid == 0) STAMP(7);
  const u64 t0 = __builtin_amdgcn_s_memrealtime();
  bool bad = false;
  u64 f;
  for (unsigned np = 1;; ++np) {
    f = __hip_atomic_load((u64*)(myws + WS_YFLG + slot * 128), __ATOMIC_RELAXED, __HIP_MEMORY_SCOPE_SYSTEM);
    if (__all((unsigned)(f >> 32) == call)) break;
    if ((np & 63u) == 0 && __any(timed_out((unsigned*)(myws + WS_TMO), call, t0))) { bad = true; break; }
    __builtin_amdgcn_s_sleep(4);
  }
  u32x4 v[16];
  const char* yq = myws + WS_YST + slot * 16384 + lane * 16;
#define YL(n, a, o) "global_load_dwordx4 %" #n ", %" #a ", off offset:" #o " sc0 sc1\n\t"
  asm volatile(YL(0, 16, 0) YL(1, 16, 1024) YL(2, 16, 2048) YL(3, 16, 3072) YL(4, 17, 0) YL(5, 17, 1024) YL(6, 17, 2048) YL(7, 17, 3072)
               YL(8, 18, 0) YL(9, 18, 1024) YL(10, 18, 2048) YL(11, 18, 3072) YL(12, 19, 0) YL(13, 19, 1024) YL(14, 19, 2048) YL(15, 19, 3072) "s_waitcnt vmcnt(0)"
               : "=&v"(v[0]), "=&v"(v[1]), "=&v"(v[2]), "=&v"(v[3]), "=&v"(v[4]), "=&v"(v[5]), "=&v"(v[6]), "=&v"(v[7]),
                 "=&v"(v[8]), "=&v"(v[9]), "=&v"(v[10]), "=&v"(v[11]), "=&v"(v[12]), "=&v"(v[13]), "=&v"(v[14]), "=&v"(v[15])
               : "v"(yq), "v"(yq + 4096), "v"(yq + 8192), "v"(yq + 12288) : "memory");
  if (wid == 0) STAMP(8);
  finish(bad, __uint_as_float((unsigned)f) * (1.f / 32767.f), [&](int i, int j, int e) {
    const unsigned q2 = v[(i * 2 + (j >> 1)) & 15][(j & 1) * 2 + (e >> 1)];
    return (e & 1) ? (float)((int)q2 >> 16) : (float)(short)(q2 & 0xffffu); });
}

__global__ __launch_bounds__(512, 2) void gemm_a2a(const Params p) {
  extern __shared__ __attribute__((aligned(16))) char smem[];
  LAS char* lds = (LAS char*)smem;
  volatile LAS unsigned* misc = (volatile LAS unsigned*)lds;
  const int tid = threadIdx.x, lane = tid & 63;
  const int wid = __builtin_amdgcn_readfirstlane(tid >> 6);
  const int b = blockIdx.x, xcd = b & 7, idx = b >> 3;
  const int rank = p.rank & 7;
  char* myws = p.ws[rank];
  unsigned* cnt = (unsigned*)(myws + WS_CNT + b * 128);
  unsigned* tmo = (unsigned*)(myws + WS_TMO);
  if (wid == 0) STAMP(0);
  auto rx = __builtin_amdgcn_make_buffer_rsrc((void*)p.x, 0, M * K * 2, 0x00020000);
  const unsigned xv0 = (unsigned)((lane >> 3) * K * 2 + (((lane & 7) ^ (lane >> 4)) << 4)), xv1 = (xv0 ^ 64) + 8 * K * 2;

  auto aggregate = [&](unsigned call) {
      const char* gp = myws + WS_GRAN + lane * 16;
      const bool skip = lane >= 56;
      unsigned gm = 0xffffffffu;
      const u64 t0 = __builtin_amdgcn_s_memrealtime();
      for (unsigned np = 1;; ++np) {
        u32x4 a, c;
        asm volatile("global_load_dwordx4 %0, %2, off sc1\n\tglobal_load_dwordx4 %1, %2, off offset:1024 sc1\n\ts_waitcnt vmcnt(0)"
                     : "=&v"(a), "=&v"(c) : "v"(gp) : "memory");
        if (__all(a[1] == call && a[3] == call && (skip || (c[1] == call && c[3] == call)))) {
          gm = wave_umax(skip ? max(a[0], a[2]) : max(max(a[0], a[2]), max(c[0], c[2])));
          break;
        }
        if ((np & 63u) == 0 && __any(timed_out(tmo, call, t0))) break;
      }
      __hip_atomic_store((u64*)(p.ws[lane & 7] + WS_GMAX + ((call & 1) * 8 + (lane >> 3)) * 128 + (SOLO ? lane & 7 : rank) * 8), ((u64)call << 32) | gm, __ATOMIC_RELAXED, __HIP_MEMORY_SCOPE_SYSTEM);
      STAMP(8);
    };

  const int x4 = xcd >> 1;
  const bool isG = xcd & 1;
  if (!isG && idx >= 28) {
    const int un = x4 * 4 + (idx & 3);
    auto stage = [&](int kt) { DMA(rx, lds + kt * 2048, xv0, un * UB + kt * 128, 0); DMA(rx, lds + kt * 2048 + 1024, xv1, un * UB + kt * 128, 0); };
    stage(wid);
    const unsigned tstart = (unsigned)__builtin_amdgcn_s_memrealtime();
    const int R = max(wid - 1, 0) * NU + un;
    const int grt = (32 + R) / 9;
    char* pw = p.ws[(rank - wid) & 7];
    char* dp = pw + WS_XST + (size_t)R * UB + lane * 16;
    char* fa = pw + WS_XFLG + (size_t)R * 128;
    unsigned c0;
    asm volatile("global_load_dword %0, %1, off sc1\n\ts_waitcnt vmcnt(0)" : "=&v"(c0) : "v"(cnt) : "memory");
    const unsigned call = __builtin_amdgcn_readfirstlane(c0) + 1u;
    if (un == 0 && tid == 0) __hip_atomic_store((u64*)(myws + WS_T0), ((u64)call << 32) | tstart, __ATOMIC_RELAXED, __HIP_MEMORY_SCOPE_AGENT);
    unsigned fv = call * 64u - 1u;
    LAS char* la = lds + lane * 16;
#pragma unroll 1
    for (int r = 0; r < 8; ++r) {
      BAR();
      if (wid == 0) {
        if (r < 7) stage(8 * (r + 1));
        WAITV(0);
        continue;
      }
      u32x4 a0 = *(LAS u32x4*)(la + gperm(r * 8, grt) * 2048), a1 = *(LAS u32x4*)(la + gperm(r * 8, grt) * 2048 + 1024);
#pragma unroll
      for (int j = 0; j < 8; ++j) {
        u32x4 n0 = a0, n1 = a1;
        if (j < 7) { n0 = *(LAS u32x4*)(la + gperm(r * 8 + j + 1, grt) * 2048); n1 = *(LAS u32x4*)(la + gperm(r * 8 + j + 1, grt) * 2048 + 1024); }
        asm volatile("global_store_dwordx4 %0, %1, off sc0 sc1\n\t"
                     "global_store_dwordx4 %0, %2, off offset:1024 sc0 sc1\n\t"
                     "s_waitcnt vmcnt(3)\n\t"
                     "s_mov_b64 exec, 1\n\tglobal_store_dword %3, %4, off offset:%5 sc0 sc1\n\ts_mov_b64 exec, -1\n\ts_nop 0"
                     ::"v"(dp), "v"(a0), "v"(a1), "v"(fa), "v"(fv), "n"((j & 3) * 4) : "memory");
        fv += 1; dp += 2048;
        if (j == 2 && r < 7) stage(8 * (r + 1) + wid);
        a0 = n0; a1 = n1;
      }
    }
    WAITV(0);
    if (lane == 0 && wid) __hip_atomic_store((unsigned*)fa, call * 64u + 63u, __ATOMIC_RELAXED, __HIP_MEMORY_SCOPE_SYSTEM);
    if (wid == 1) STAMP(7);
    const int fi = un * 8 + wid;
    if (!SOLO && lane == 0 && (fi >> 4) != rank && !early_tile(fi & 15)) {
      unsigned* fl = (unsigned*)(myws + WS_DONE + (size_t)fi * 128);
      const u64 t1 = __builtin_amdgcn_s_memrealtime();
      unsigned np = 0;
      while (__hip_atomic_load(fl, __ATOMIC_RELAXED, __HIP_MEMORY_SCOPE_SYSTEM) < call) {
        if ((++np & 63u) == 0 && timed_out(tmo, call, t1)) { p.out[rank][(size_t)rank * M * NS + fi] = (char)-128; break; }
        __builtin_amdgcn_s_sleep(2);
      }
    }
    if (tid == 0) __hip_atomic_store(cnt, call, __ATOMIC_RELAXED, __HIP_MEMORY_SCOPE_AGENT);
    if (wid == 1) STAMP(6);
    return;
  }

  const int rt = (x4 & 1) * 8 + (idx & 7), dj = idx >> 2;
  const int cl = isG ? (x4 >> 1) * 4 + (idx >> 3) : x4 * 4 + (idx & 3);
  const int dst = (rank + 1 + dj) & 7;
  const bool early = !isG && early_tile(cl);
  if (wid < 4) {
    __builtin_amdgcn_s_setprio(2);
    if (isG) mfma_role<true>(lds, wid, lane, b, myws, p.out[rank] + cl * 256, rank, rt * 9, nullptr);
    else mfma_role<false>(lds, wid, lane, b, myws, (early ? p.out[rank] + (size_t)(((rank - 1 - dj) & 7) * M + GR) * NS : p.out[dst] + (size_t)(rank * M + GR) * NS) + cl * 128,
                          rank, dj * 16 + cl, early ? p.ws[dst] : nullptr);
    return;
  }
  const int w = wid - 4;
  unsigned call = 0;
  auto rw = __builtin_amdgcn_make_buffer_rsrc((void*)p.w, 0, K * N * 2, 0x00020000);
  if (!isG) {
    using RG = Ring<false>;
    const unsigned v0 = (unsigned)(((lane >> 4) + w * 16) * N * 2 + (dst * 16 + cl) * 256 + (((lane & 15) ^ ((lane >> 4) << 1)) << 4)), v1 = (v0 ^ 128) + 8 * N * 2;
    auto ldx = [&](int t, int sx) {
#pragma unroll
      for (int i = 0; i < 4; ++i) {
        DMA(rx, lds + sx * RG::XS + (w * 4 + i) * 2048, xv0, (NU + w * 4 + i) * UB + t * 128, 0);
        DMA(rx, lds + sx * RG::XS + (w * 4 + i) * 2048 + 1024, xv1, (NU + w * 4 + i) * UB + t * 128, 0);
      }
    };
    auto ldw = [&](int t) {
#pragma unroll
      for (int j = 0; j < 4; ++j)
        DMA(rw, lds + RG::W0 + (t & 3) * RG::WS + (w * 4 + j) * 1024, (j & 2) ? v1 : v0, (t * 64 + (j & 1) * 4) * N * 2, 2);
    };
    const int al = (int)((size_t)p.w >> 8) + dst * 16 + cl;
    const bool paced = w == 0 && (al & 31) != 31, slow = (al & 3) == 1;
    const int pend = slow ? NT : early ? LT2E : LT2, pace = slow ? SP : LPACE;
    const bool xpf = (cl & 3) == 2 ? w == 3 || (w == 2 && (al & 31) == 30) : w == 3 && (cl & 3) != 1 && (al & 31) != 31;
    const int xq = dj * 3 + max((cl & 3) - 1, 0) + (w == 2);
    unsigned t0_ = 0;
    u64 tp = __builtin_amdgcn_s_memrealtime() + 200 + LT1 * 69;
    int sx = 0;
    for (int t = 0; t < NT; t += 2) {
      ldx(t, sx); ldw(t); ldw(t + 1);
      sx = sx == 2 ? 0 : sx + 1;
      if (xpf) {
        const unsigned n = ((t >> 1) * 21 + xq) * 64 + lane, kt = (n >> 8) + 3;
        asm volatile("buffer_load_dword %0, %1, %2, 0 offen" : "+v"(t0_) : "v"(kt < NT ? (GR + (n & 255)) * K * 2 + kt * 128 : 0xfffff000u), "s"(rx) : "memory");
        WAITV(17);
      } else WAITV(16);
      if (t) BAR();
      ldx(t + 1, sx);
      sx = sx == 2 ? 0 : sx + 1;
      if (paced && t >= LT1) {
        if (t == LT1) {
          u64 an; unsigned c0;
          asm volatile("global_load_dwordx2 %0, %2, off sc1\n\tglobal_load_dword %1, %3, off sc1\n\ts_waitcnt vmcnt(0)" : "=&v"(an), "=&v"(c0) : "v"(myws + WS_T0), "v"(cnt) : "memory");
          call = __builtin_amdgcn_readfirstlane(c0) + 1u;
          const int dt = (int)(__builtin_amdgcn_readfirstlane((unsigned)an) + 200 + LT1 * 69 - (unsigned)tp);
          if (__builtin_amdgcn_readfirstlane((unsigned)(an >> 32)) == call && dt > -200 && dt < 200) tp += dt;
        }
        while ((long long)(__builtin_amdgcn_s_memrealtime() - tp) < 0) __builtin_amdgcn_s_sleep(1);
        tp += 2 * (t + 2 < pend ? pace : BP);
      }
      if (xpf) WAITV(13); else WAITV(12);
      BAR();
    }
    asm volatile("s_waitcnt vmcnt(0)" : "+v"(t0_)::"memory");
    BAR();
    BAR();
  } else if (w == 0 || w == 3) {
    using RG = Ring<true>;
    const int i0 = w ? 5 : 0, u0 = rt * 9 + i0;
    int so = i0 * 2048;
    if (u0 < 32) {
      for (int t = 0; t < NT + 2; ++t) {
        if (t < NT) {
          const unsigned ox = (31 - u0) * UB + gperm(t, rt) * 128;
#pragma unroll
          for (int i = 0; i < 5; ++i)
            if (i < 4 || w == 0) { DMA(rx, lds + so + i * 2048, xv0, ox - i * UB, 0); DMA(rx, lds + so + i * 2048 + 1024, xv1, ox - i * UB, 0); }
          so = so >= 2 * RG::XS ? so - 2 * RG::XS : so + RG::XS;
          if (w == 0) WAITV(10); else WAITV(8);
        } else WAITV(0);
        if (t >= 1) BAR();
      }
    } else {
      auto rg = __builtin_amdgcn_make_buffer_rsrc((void*)(myws + WS_XST), 0, (NR - 1) * NU * UB, 0x00020000);
      const unsigned gv = lane * 16, gv1 = gv + 1024;
      const unsigned* fp = (const unsigned*)(myws + WS_XFLG + (size_t)(u0 - 32 + min(lane >> 2, w ? 3 : 4)) * 128) + (lane & 3);
      LAS unsigned* pl = (LAS unsigned*)(lds + GPOLL + (w ? 256 : 0));
      unsigned rdy = 0, np = 0;
      call = call_number(myws, b);
      const u64 t0 = __builtin_amdgcn_s_memrealtime();
      auto seen = [&](unsigned f) {
        int n = (int)(f - call * 64u) + 1;
        n = max(n, __builtin_amdgcn_update_dpp(0, n, 0xB1, 0xf, 0xf, false));
        n = max(n, __builtin_amdgcn_update_dpp(0, n, 0x4E, 0xf, 0xf, false));
        int m = __builtin_amdgcn_readlane(n, 0);
#pragma unroll
        for (int i = 1; i < 5; ++i) m = min(m, __builtin_amdgcn_readlane(n, 4 * i));
        rdy = max((int)rdy, m);
      };
      int nx = 0;
      auto issue = [&]() {
        const unsigned og = (u0 - 32) * UB + nx * 2048;
#pragma unroll
        for (int i = 0; i < 5; ++i)
          if (i < 4 || w == 0) { DMA(rg, lds + so + i * 2048, gv, og + i * UB, 17); DMA(rg, lds + so + i * 2048 + 1024, gv1, og + i * UB, 17); }
        so = so >= 2 * RG::XS ? so - 2 * RG::XS : so + RG::XS;
        ++nx;
      };
      bool pend = false;
      for (int t = 1; t <= NT; ++t) {
        if (pend && rdy <= (unsigned)t) { WAITV(0); seen(*(volatile LAS unsigned*)(pl + lane)); pend = false; }
        if (nx < t) {
          while (rdy < (unsigned)t) {
            seen(__hip_atomic_load(fp, __ATOMIC_RELAXED, __HIP_MEMORY_SCOPE_SYSTEM));
            if (rdy < (unsigned)t && (++np & 63u) == 0 && __any(timed_out(tmo, call, t0))) rdy = NT;
          }
          issue();
        }
        const bool opt = t < NT && rdy > (unsigned)t;
        if (opt) issue();
        const bool poll = rdy < NT && rdy < (unsigned)t + 5 && !pend;
        if (poll) __builtin_amdgcn_global_load_lds(fp, pl, 4, 0, 17);
        if (!opt) { if (poll) WAITV(1); else WAITV(0); }
        else if (w == 0) { if (poll) WAITV(11); else WAITV(10); }
        else { if (poll) WAITV(9); else WAITV(8); }
        if (pend) seen(*(volatile LAS unsigned*)(pl + lane));
        pend = poll;
        BAR();
      }
      WAITV(0);
      BAR();
    }
  } else {
    using RG = Ring<true>;
    const unsigned v0 = (unsigned)(((lane >> 5) + (w - 1) * 32) * N * 2 + (rank * 8 + cl) * 512 + (((lane & 31) ^ ((lane >> 5) << 1)) << 4));
    const unsigned v1 = (v0 ^ 64) + 2 * N * 2, v2 = (v0 ^ 128) + 8 * N * 2, v3 = (v0 ^ 192) + 10 * N * 2;
    int so = 0;
    for (int t = 0; t < NT + 2; ++t) {
      if (t < NT) {
        const int kt = gperm(t, rt);
#pragma unroll
        for (int j = 0; j < 16; ++j)
          DMA(rw, lds + RG::W0 + so + (w - 1) * 16384 + j * 1024, (j & 4) ? ((j & 1) ? v3 : v2) : ((j & 1) ? v1 : v0), (kt * 64 + ((j & 2) << 1) + ((j & 8) << 1)) * N * 2, 0);
        so = so == 2 * RG::WS ? 0 : so + RG::WS;
        WAITV(16);
      } else WAITV(0);
      if (t >= 1) BAR();
    }
  }
  if (w == 0 && call == 0) call = call_number(myws, b);
  LBAR();
  if (b == 1 && w) {
    if (w > 1) __builtin_amdgcn_s_sleep(6);
    if (w > 2) __builtin_amdgcn_s_sleep(6);
    aggregate(call_number(myws, b));
  }
  if (w == 0) {
    const u32x4 mv = *(volatile LAS u32x4*)(misc + L_MAX);
    const unsigned m = max(max(mv[0], mv[1]), max(mv[2], mv[3]));
    const int gslot = isG ? rt * 8 + cl : 128 + dj * 16 + cl;
    if (lane == 0) __hip_atomic_store((u64*)(myws + WS_GRAN + gslot * 8), ((u64)call << 32) | m, __ATOMIC_RELAXED, __HIP_MEMORY_SCOPE_AGENT);
    const char* gp = myws + WS_GMAX + ((call & 1) * 8 + xcd) * 128 + (lane & 7) * 8;
    unsigned gm = 0xffffffffu;
    const u64 t0 = __builtin_amdgcn_s_memrealtime();
    for (;;) {
      u64 gq; unsigned tm;
      asm volatile("global_load_dwordx2 %0, %2, off sc0 sc1\n\tglobal_load_dword %1, %3, off sc1\n\ts_waitcnt vmcnt(0)"
                   : "=&v"(gq), "=&v"(tm) : "v"(gp), "v"(tmo) : "memory");
      if (__any(tm == call)) break;
      if (__all((unsigned)(gq >> 32) == call)) { gm = wave_umax((unsigned)gq); break; }
      if (__builtin_amdgcn_s_memrealtime() - t0 > DEADLINE) { __hip_atomic_store(tmo, call, __ATOMIC_RELAXED, __HIP_MEMORY_SCOPE_AGENT); break; }
      __builtin_amdgcn_s_sleep(1);
    }
    if (lane == 0) misc[L_GMAX] = gm;
  }
  LBAR();
  LBAR();
  if (w == 0 && lane == 0) {
    if (!isG && !early)
      __hip_atomic_store((unsigned*)(p.ws[dst] + WS_DONE + (size_t)(rank * 16 + cl) * 128), call, __ATOMIC_RELAXED, __HIP_MEMORY_SCOPE_SYSTEM);
    __hip_atomic_store(cnt, call, __ATOMIC_RELAXED, __HIP_MEMORY_SCOPE_AGENT);
    STAMP(6);
  }
}

extern "C" void kernel_launch(int rank, int nranks, ncclComm_t comm, hipStream_t stream,
                              void* const* d_in, const int* in_sizes, int n_in,
                              void* d_out, int out_size, const int* in_dtypes, int out_dtype,
                              void* const* d_out_all, void* const* d_ws_all, size_t ws_bytes) {
  if (nranks != NR || (unsigned)rank >= (unsigned)NR || n_in < 2 || ws_bytes < WS_END || in_sizes[0] != M * K || in_sizes[1] != K * N || out_size != NR * M * NS) {
    fprintf(stderr, "unexpected problem shape\n");
    return;
  }
  static bool init = false;
  if (!init) {
    HIP_CHECK(hipFuncSetAttribute((const void*)gemm_a2a, hipFuncAttributeMaxDynamicSharedMemorySize, LDS_BYTES));
    init = true;
  }
  Params p;
  p.x = d_in[0]; p.w = d_in[1];
  for (int i = 0; i < NR; ++i) { p.out[i] = (char*)d_out_all[i]; p.ws[i] = (char*)d_ws_all[i]; }
  p.rank = rank; p.pad = 0;
  gemm_a2a<<<NWG, 512, LDS_BYTES, stream>>>(p);
  HIP_CHECK(hipGetLastError());
}
```
